```python
import numpy as np

kernel_name = "gpu_gemm_rs_m4096_k4096_n4096_f32_silu"


ABSMAX_RTOL = 0.001

COMPUTE_DTYPE = "float32"
EPILOGUE = "silu"
IN_QUANT = None
OUT_QUANT = None
IN_DTYPES = {"x": "float32", "w_mat": "float32"}
WORLD_SIZE = 8

def setup_inputs():

    x = np.random.randn(4096, 4096).astype(np.float32)
    w_mat = (np.random.randn(4096, 4096) / np.sqrt(4096)).astype(np.float32)
    return {"x": x, "w_mat": w_mat}

def _epilogue(y):
    return (y / (1.0 + np.exp(-np.clip(y, -60.0, 60.0)))).astype(np.float32)

def reference(x, w_mat):
    y = (x @ w_mat).astype(np.float32)
    return _epilogue(y)
```

```cpp
#include <hip/hip_runtime.h>
#include <rccl/rccl.h>
#include <cstdio>

#define HIP_CHECK(expr) do { hipError_t _e = (expr); if (_e != hipSuccess) { \
    fprintf(stderr, "HIP error %s at %s:%d\n", hipGetErrorString(_e), __FILE__, __LINE__); \
    return; } } while (0)

typedef unsigned u32x4 __attribute__((ext_vector_type(4)));
typedef float f32x4 __attribute__((ext_vector_type(4)));
typedef float f32x16 __attribute__((ext_vector_type(16)));
typedef _Float16 f16x8 __attribute__((ext_vector_type(8)));
typedef __attribute__((address_space(1))) unsigned gu32;

#define NSUB 4
#define STAGGER_TICKS 0
#define POLL_SLEEP 8

constexpr int NR = 8;
constexpr int NN = 4096;
constexpr int KS = 512;
constexpr int CH = 64;
constexpr int NCH = KS / CH;
constexpr int NSLOT = 24;
constexpr unsigned PIECE = 16384;
constexpr unsigned BUF_BYTES = NR * NCH * NSLOT * PIECE;
constexpr unsigned IN_BYTES = 4096u * 512u * 4u;
constexpr int NWG = 256;
constexpr unsigned OFF_SEQ = 0;
constexpr unsigned OFF_INF = 128u << 10;
constexpr unsigned OFF_OUTF = 256u << 10;
constexpr unsigned OFF_TMO = 384u << 10;
constexpr unsigned OFF_DBG = 512u << 10;
constexpr unsigned OFF_DATA = 1u << 20;
constexpr unsigned WS_USED = OFF_DATA + 2 * BUF_BYTES;
constexpr unsigned long long DEADLINE_TICKS = 50000000ull;
constexpr int AUX_SYS = 17;
constexpr int RSRC_W3 = 0x00020000;

struct Params {
    const float* x;
    const float* w;
    char* ws[NR];
    char* out[NR];
    int rank;
    int pad;
};

#define STAMP_AT(idx) do {} while (0)

__device__ __forceinline__ u32x4 pack8(float a0, float a1, float a2, float a3,
                                       float a4, float a5, float a6, float a7) {
    f16x8 h = {(_Float16)a0, (_Float16)a1, (_Float16)a2, (_Float16)a3,
               (_Float16)a4, (_Float16)a5, (_Float16)a6, (_Float16)a7};
    return __builtin_bit_cast(u32x4, h);
}

__device__ __forceinline__ void wait_flags(gu32* addr, unsigned& seen, unsigned target, unsigned call,
                                           gu32* tmo, int lane) {
    if (__all(lane == 63 || seen >= target)) return;
    const unsigned long long t0 = __builtin_amdgcn_s_memrealtime();
    for (;;) {
        seen = __hip_atomic_load(addr, __ATOMIC_RELAXED, __HIP_MEMORY_SCOPE_SYSTEM);
        if (__all(lane == 63 || seen >= target)) break;
        if (__any(lane == 63 && seen == call) || __builtin_amdgcn_s_memrealtime() - t0 > DEADLINE_TICKS) {
            if (lane == 0) __hip_atomic_store(tmo, call, __ATOMIC_RELAXED, __HIP_MEMORY_SCOPE_AGENT);
            break;
        }
        __builtin_amdgcn_s_sleep(POLL_SLEEP);
    }
    asm volatile("" ::: "memory");
}

__global__ void __launch_bounds__(512, 2) fused_gemm_silu(Params p) {
    __shared__ f32x4 red[4 * 32 * 64 + 1];
    const int lane = threadIdx.x & 63;
    const int wave = __builtin_amdgcn_readfirstlane(threadIdx.x >> 6);
    const int b = blockIdx.x;
    const int rank = p.rank;
    char* const myws = p.ws[rank];
    STAMP_AT(0);

    if (threadIdx.x == 0) red[4 * 32 * 64] = f32x4{0.f, 0.f, 0.f, 0.f};
    __syncthreads();

    const bool sender = wave >= 4;
    if (sender && (wave - 4 >= NSUB || b >= NR * NSLOT)) return;

    unsigned call = 0;
    if (lane == 0)
        call = __hip_atomic_fetch_add((gu32*)(myws + OFF_SEQ + (b * 8 + wave) * 64), 1u,
                                      __ATOMIC_RELAXED, __HIP_MEMORY_SCOPE_AGENT) + 1u;

    if (sender) {
        constexpr int NG = 4 / NSUB;
        const int sub = wave - 4;
        const int u = b % NSLOT;
        const unsigned d = (unsigned)(rank + 1 + b / NSLOT) & (NR - 1);
        const bool isx = u < 8;
        const __amdgpu_buffer_rsrc_t src =
            __builtin_amdgcn_make_buffer_rsrc((void*)(isx ? p.x : p.w), 0, IN_BYTES, RSRC_W3);
        const __amdgpu_buffer_rsrc_t dst =
            __builtin_amdgcn_make_buffer_rsrc((void*)p.ws[d], 0, WS_USED, RSRC_W3);
        gu32* const flag = (gu32*)(p.ws[d] + OFF_INF + ((rank * NSLOT + u) * NSUB + sub) * 128);
        const int r = lane & 31, h = lane >> 5;
        f32x4 v[NG][8];
        if (isx) {
            const unsigned voff = ((u * 512 + (d >> 1) * 128 + sub * NG * 32 + r) * KS + h * 8) * 4;
#pragma unroll
            for (int g = 0; g < NG; ++g)
#pragma unroll
                for (int t = 0; t < 8; ++t)
                    v[g][t] = __builtin_bit_cast(f32x4, __builtin_amdgcn_raw_buffer_load_b128(
                        src, voff + (t >> 1) * 64 + (t & 1) * 16, g * 32 * KS * 4, 0));
            call = __builtin_amdgcn_readfirstlane(call);
            const unsigned dbase = OFF_DATA + (call & 1) * BUF_BYTES + (rank * NCH * NSLOT + u) * PIECE
                                   + sub * NG * 4096;
            for (int c = 0; c < NCH; ++c) {
                u32x4 f[NG][4];
#pragma unroll
                for (int g = 0; g < NG; ++g)
#pragma unroll
                    for (int ks = 0; ks < 4; ++ks) {
                        const f32x4 a = v[g][2 * ks], bb = v[g][2 * ks + 1];
                        f[g][ks] = pack8(a.x, a.y, a.z, a.w, bb.x, bb.y, bb.z, bb.w);
                    }
                if (c + 1 < NCH) {
#pragma unroll
                    for (int g = 0; g < NG; ++g)
#pragma unroll
                        for (int t = 0; t < 8; ++t)
                            v[g][t] = __builtin_bit_cast(f32x4, __builtin_amdgcn_raw_buffer_load_b128(
                                src, voff + (t >> 1) * 64 + (t & 1) * 16, g * 32 * KS * 4 + (c + 1) * CH * 4, 0));
                }
#pragma unroll
                for (int g = 0; g < NG; ++g)
#pragma unroll
                    for (int ks = 0; ks < 4; ++ks)
                        __builtin_amdgcn_raw_buffer_store_b128(f[g][ks], dst,
                            dbase + c * NSLOT * PIECE + g * 4096 + lane * 16 + ks * 1024, 0, AUX_SYS);
                asm volatile("s_waitcnt vmcnt(0)" ::: "memory");
                if (lane == 0)
                    __hip_atomic_store(flag, call * NCH + c + 1, __ATOMIC_RELAXED, __HIP_MEMORY_SCOPE_SYSTEM);
                STAMP_AT(1 + c);
            }
        } else {
            const int cb = u - 8;
            const unsigned voff = ((sub * NG * 16 + h * 8) * NN + (d & 1) * 2048 + cb * 128 + r * 4) * 4;
#pragma unroll
            for (int g = 0; g < NG; ++g)
#pragma unroll
                for (int t = 0; t < 8; ++t)
                    v[g][t] = __builtin_bit_cast(f32x4, __builtin_amdgcn_raw_buffer_load_b128(
                        src, voff, (g * 16 + t) * NN * 4, 0));
            call = __builtin_amdgcn_readfirstlane(call);
            const unsigned dbase = OFF_DATA + (call & 1) * BUF_BYTES + (rank * NCH * NSLOT + u) * PIECE
                                   + sub * NG * 4096;
            if (STAGGER_TICKS > 0 && d != (unsigned)rank) {
                const unsigned long long t0 = __builtin_amdgcn_s_memrealtime();
                const unsigned long long dt = (unsigned long long)((cb & 3) * STAGGER_TICKS);
                while (__builtin_amdgcn_s_memrealtime() - t0 < dt) __builtin_amdgcn_s_sleep(8);
            }
            for (int c = 0; c < NCH; ++c) {
                u32x4 f[NG][4];
#pragma unroll
                for (int g = 0; g < NG; ++g)
#pragma unroll
                    for (int e = 0; e < 4; ++e)
                        f[g][e] = pack8(v[g][0][e], v[g][1][e], v[g][2][e], v[g][3][e],
                                        v[g][4][e], v[g][5][e], v[g][6][e], v[g][7][e]);
                if (c + 1 < NCH) {
#pragma unroll
                    for (int g = 0; g < NG; ++g)
#pragma unroll
                        for (int t = 0; t < 8; ++t)
                            v[g][t] = __builtin_bit_cast(f32x4, __builtin_amdgcn_raw_buffer_load_b128(
                                src, voff, ((c + 1) * CH + g * 16 + t) * NN * 4, 0));
                }
#pragma unroll
                for (int g = 0; g < NG; ++g)
#pragma unroll
                    for (int e = 0; e < 4; ++e)
                        __builtin_amdgcn_raw_buffer_store_b128(f[g][e], dst,
                            dbase + c * NSLOT * PIECE + g * 4096 + lane * 16 + e * 1024, 0, AUX_SYS);
                asm volatile("s_waitcnt vmcnt(0)" ::: "memory");
                if (lane == 0)
                    __hip_atomic_store(flag, call * NCH + c + 1, __ATOMIC_RELAXED, __HIP_MEMORY_SCOPE_SYSTEM);
                STAMP_AT(1 + c);
            }
        }
        return;
    }

    call = __builtin_amdgcn_readfirstlane(call);
    const int xcd = b & 7, idx = b >> 3;
    const int rb = (xcd & 1) * 8 + (idx & 7);
    const int cb = (xcd >> 1) * 4 + (idx >> 3);
    const unsigned owner = (unsigned)rb >> 1;
    gu32* const tmo = (gu32*)(myws + OFF_TMO);
    const __amdgpu_buffer_rsrc_t buf = __builtin_amdgcn_make_buffer_rsrc((void*)myws, 0, WS_USED, RSRC_W3);

    const int s0 = (rank + wave) & 7, s1 = (rank + wave + 4) & 7;
    constexpr int NXF = NSUB >= 2 ? NSUB / 2 : 1;
    const int fl = lane < NSUB ? (8 + cb) * NSUB + lane
                               : (int)owner * NSUB + (NSUB >= 2 ? (rb & 1) * NXF + (lane - NSUB) % NXF : 0);
    gu32* const fa0 = lane == 63 ? tmo : (gu32*)(myws + OFF_INF + (s0 * NSLOT * NSUB + fl) * 128);
    gu32* const fa1 = lane == 63 ? tmo : (gu32*)(myws + OFF_INF + (s1 * NSLOT * NSUB + fl) * 128);
    unsigned seen0 = 0, seen1 = 0;

    const unsigned pbase = OFF_DATA + (call & 1) * BUF_BYTES;
    const unsigned xo = owner * PIECE + (rb & 1) * 8192, wo = (8 + cb) * PIECE;
    const unsigned vo = lane * 16;

    f32x16 acc[2][4];
#pragma unroll
    for (int mt = 0; mt < 2; ++mt)
#pragma unroll
        for (int e = 0; e < 4; ++e)
#pragma unroll
            for (int q = 0; q < 16; ++q) acc[mt][e][q] = 0.f;

    u32x4 A0[2][2], B0[2][4], A1[2][2], B1[2][4];
#define LOAD_HALF(A, B, base, hh) do { \
        _Pragma("unroll") for (int ks = 0; ks < 2; ++ks) { \
            _Pragma("unroll") for (int mt = 0; mt < 2; ++mt) \
                A[ks][mt] = __builtin_amdgcn_raw_buffer_load_b128(buf, vo + ((hh) * 2 + ks) * 1024, \
                                                                  (base) + xo + mt * 4096, AUX_SYS); \
            _Pragma("unroll") for (int e = 0; e < 4; ++e) \
                B[ks][e] = __builtin_amdgcn_raw_buffer_load_b128(buf, vo + e * 1024, \
                                                                 (base) + wo + ((hh) * 2 + ks) * 4096, AUX_SYS); \
        } } while (0)
#define MFMA_HALF(A, B) do { \
        _Pragma("unroll") for (int ks = 0; ks < 2; ++ks) \
            _Pragma("unroll") for (int mt = 0; mt < 2; ++mt) \
                _Pragma("unroll") for (int e = 0; e < 4; ++e) \
                    acc[mt][e] = __builtin_amdgcn_mfma_f32_32x32x16_f16( \
                        __builtin_bit_cast(f16x8, A[ks][mt]), __builtin_bit_cast(f16x8, B[ks][e]), \
                        acc[mt][e], 0, 0, 0); \
        } while (0)

    const unsigned sb0 = pbase + s0 * NCH * NSLOT * PIECE, sb1 = pbase + s1 * NCH * NSLOT * PIECE;
    wait_flags(fa0, seen0, call * NCH + 1, call, tmo, lane);
    STAMP_AT(1);
    LOAD_HALF(A0, B0, sb0, 0);
    for (int c = 0; c < NCH; ++c) {
        const unsigned co = c * NSLOT * PIECE;
        LOAD_HALF(A1, B1, sb0 + co, 1);
        MFMA_HALF(A0, B0);
        wait_flags(fa1, seen1, call * NCH + c + 1, call, tmo, lane);
        STAMP_AT(2 + 2 * c);
        LOAD_HALF(A0, B0, sb1 + co, 0);
        MFMA_HALF(A1, B1);
        LOAD_HALF(A1, B1, sb1 + co, 1);
        MFMA_HALF(A0, B0);
        if (c + 1 < NCH) {
            wait_flags(fa0, seen0, call * NCH + c + 2, call, tmo, lane);
            STAMP_AT(3 + 2 * c);
            LOAD_HALF(A0, B0, sb0 + co + NSLOT * PIECE, 0);
        }
        MFMA_HALF(A1, B1);
    }
    STAMP_AT(17);

#pragma unroll
    for (int mt = 0; mt < 2; ++mt)
#pragma unroll
        for (int q = 0; q < 16; ++q)
            red[(wave * 32 + mt * 16 + q) * 64 + lane] =
                f32x4{acc[mt][0][q], acc[mt][1][q], acc[mt][2][q], acc[mt][3][q]};
    {
        unsigned* cnt = (unsigned*)&red[4 * 32 * 64];
        __builtin_amdgcn_fence(__ATOMIC_RELEASE, "workgroup");
        if (lane == 0) __hip_atomic_fetch_add(cnt, 1u, __ATOMIC_RELAXED, __HIP_MEMORY_SCOPE_WORKGROUP);
        while (__hip_atomic_load(cnt, __ATOMIC_RELAXED, __HIP_MEMORY_SCOPE_WORKGROUP) < 4u)
            __builtin_amdgcn_s_sleep(1);
        __builtin_amdgcn_fence(__ATOMIC_ACQUIRE, "workgroup");
    }
    STAMP_AT(18);
    const bool dead = __hip_atomic_load(tmo, __ATOMIC_RELAXED, __HIP_MEMORY_SCOPE_AGENT) == call;
    const __amdgpu_buffer_rsrc_t od =
        __builtin_amdgcn_make_buffer_rsrc((void*)p.out[owner & (NR - 1)], 0, IN_BYTES, RSRC_W3);
    const unsigned orow = (rank >> 1) * 128 + (rb & 1) * 64 + (wave >> 1) * 32 + (wave & 1) * 16 + (lane >> 5) * 4;
    const unsigned ooff = (orow * NN + (rank & 1) * 2048 + cb * 128 + (lane & 31) * 4) * 4;
#pragma unroll
    for (int t = 0; t < 8; ++t) {
        const int e = (wave * 8 + t) * 64 + lane;
        f32x4 y = (red[e] + red[e + 32 * 64]) + (red[e + 2 * 32 * 64] + red[e + 3 * 32 * 64]);
        f32x4 o;
#pragma unroll
        for (int k = 0; k < 4; ++k) {
            const float z = fminf(fmaxf(y[k], -60.f), 60.f);
            o[k] = dead ? __builtin_nanf("") : y[k] * __builtin_amdgcn_rcpf(1.f + __expf(-z));
        }
        __builtin_amdgcn_raw_buffer_store_b128(__builtin_bit_cast(u32x4, o), od,
                                               ooff + ((t & 3) + (t >> 2) * 8) * NN * 4, 0, AUX_SYS);
    }
    STAMP_AT(19);
    asm volatile("s_waitcnt vmcnt(0)" ::: "memory");
    if (lane == 0)
        __hip_atomic_store((gu32*)(p.ws[owner & (NR - 1)] + OFF_OUTF + (((rank * 2 + (rb & 1)) * 16 + cb) * 4 + wave) * 128),
                           call, __ATOMIC_RELAXED, __HIP_MEMORY_SCOPE_SYSTEM);
    STAMP_AT(20);
    {
        gu32* const fo = lane == 63 ? tmo : (gu32*)(myws + OFF_OUTF + (b * 4 + wave) * 128);
        unsigned seen = 0;
        wait_flags(fo, seen, call, call, tmo, lane);
        if (!__all(lane == 63 || seen >= call) && lane == 0)
            *(float*)(p.out[rank] + (size_t)((rank >> 1) * 128 * NN + (rank & 1) * 2048) * 4) = __builtin_nanf("");
    }
    STAMP_AT(21);
}

extern "C" void kernel_launch(int rank, int nranks, ncclComm_t comm,
                              hipStream_t stream,
                              void* const* d_in, const int* in_sizes, int n_in,
                              void* d_out, int out_size,
                              const int* in_dtypes, int out_dtype,
                              void* const* d_out_all,
                              void* const* d_ws_all, size_t ws_bytes) {
    if (nranks != NR || (unsigned)rank >= (unsigned)NR || ws_bytes < WS_USED || n_in != 2 ||
        in_sizes[0] != 4096 * 512 || in_sizes[1] != 4096 * 512 || out_size != 4096 * 512) {
        fprintf(stderr, "unexpected problem shape\n");
        return;
    }
    Params p;
    p.x = (const float*)d_in[0];
    p.w = (const float*)d_in[1];
    for (int r = 0; r < NR; ++r) {
        p.ws[r] = (char*)d_ws_all[r];
        p.out[r] = (char*)d_out_all[r];
    }
    p.rank = rank;
    p.pad = 0;
    fused_gemm_silu<<<NWG, 512, 0, stream>>>(p);
    HIP_CHECK(hipGetLastError());
}
```

```cpp
#include <hip/hip_runtime.h>
#include <rccl/rccl.h>
#include <cstdio>

#define HIP_CHECK(expr) do { hipError_t _e = (expr); if (_e != hipSuccess) { \
    fprintf(stderr, "HIP error %s at %s:%d\n", hipGetErrorString(_e), __FILE__, __LINE__); \
    return; } } while (0)

typedef unsigned u32x4 __attribute__((ext_vector_type(4)));
typedef float f32x4 __attribute__((ext_vector_type(4)));
typedef float f32x16 __attribute__((ext_vector_type(16)));
typedef _Float16 f16x8 __attribute__((ext_vector_type(8)));
typedef __attribute__((address_space(1))) unsigned gu32;
typedef unsigned long long u64;

#define POLL_SLEEP 4
#define CONV_DELAY 8
#define COLO 0
#define SPREAD 1

constexpr int NR = 8;
constexpr int NN = 4096;
constexpr int KS = 512;
constexpr unsigned UNIT = 4096;
constexpr unsigned BLOCK = 32 * UNIT;
constexpr int NSLOT = 24;
constexpr unsigned IN_BYTES = 4096u * 512u * 4u;
constexpr int NWG = 256;
constexpr unsigned OFF_SEQ = 0;
constexpr unsigned OFF_EF = 128u << 10;
constexpr unsigned OFF_SF = 192u << 10;
constexpr unsigned OFF_RF = 224u << 10;
constexpr unsigned OFF_TMO = 240u << 10;
constexpr unsigned OFF_DUMMY = 244u << 10;
constexpr unsigned OFF_YF = 256u << 10;
constexpr unsigned OFF_DBG = 512u << 10;
constexpr unsigned OFF_DATA = 1u << 20;
constexpr unsigned P_EXP = 0;
constexpr unsigned P_STG = 64 * BLOCK;
constexpr unsigned P_Y = P_STG + NR * NSLOT * BLOCK;
constexpr unsigned P_BYTES = P_Y + 128 * 32768;
constexpr unsigned WS_USED = OFF_DATA + 2 * P_BYTES;
constexpr u64 DEADLINE_TICKS = 50000000ull;
constexpr int AUX_SYS = 17;
constexpr int RSRC_W3 = 0x00020000;

struct Params {
    const float* x;
    const float* w;
    char* ws[NR];
    char* out;
    int rank;
    int pad;
};

#define STAMP_AT(idx) do {} while (0)

__device__ __forceinline__ u32x4 pack8(float a0, float a1, float a2, float a3,
                                       float a4, float a5, float a6, float a7) {
    f16x8 h = {(_Float16)a0, (_Float16)a1, (_Float16)a2, (_Float16)a3,
               (_Float16)a4, (_Float16)a5, (_Float16)a6, (_Float16)a7};
    return __builtin_bit_cast(u32x4, h);
}
__device__ __forceinline__ unsigned poll(gu32* a) {
    return __hip_atomic_load(a, __ATOMIC_RELAXED, __HIP_MEMORY_SCOPE_SYSTEM);
}
__device__ __forceinline__ void post(gu32* a, unsigned v) {
    __hip_atomic_store(a, v, __ATOMIC_RELAXED, __HIP_MEMORY_SCOPE_SYSTEM);
}
__device__ __forceinline__ u64 full_groups(u64 m) {
    m |= 1ull << 63;
    m &= m >> 4; m &= m >> 2; m &= m >> 1;
    return m & 0x0101010101010101ull;
}

__device__ __forceinline__ void conv_load(f32x4 (&v)[8], __amdgpu_buffer_rsrc_t src, bool isx, unsigned voff, int m) {
    if (isx) {
#pragma unroll
        for (int t = 0; t < 8; ++t)
            v[t] = __builtin_bit_cast(f32x4, __builtin_amdgcn_raw_buffer_load_b128(
                src, voff + (t & 1) * 16, (t >> 1) * 32 * KS * 4 + m * 64, 0));
    } else {
#pragma unroll
        for (int t = 0; t < 8; ++t)
            v[t] = __builtin_bit_cast(f32x4, __builtin_amdgcn_raw_buffer_load_b128(
                src, voff, (m * 16 + t) * NN * 4, 0));
    }
}
__device__ __forceinline__ void conv_store(const f32x4 (&v)[8], __amdgpu_buffer_rsrc_t dst, bool isx, unsigned off) {
    u32x4 f[4];
    if (isx) {
#pragma unroll
        for (int e = 0; e < 4; ++e) {
            const f32x4 a = v[2 * e], b = v[2 * e + 1];
            f[e] = pack8(a.x, a.y, a.z, a.w, b.x, b.y, b.z, b.w);
        }
    } else {
#pragma unroll
        for (int e = 0; e < 4; ++e)
            f[e] = pack8(v[0][e], v[1][e], v[2][e], v[3][e], v[4][e], v[5][e], v[6][e], v[7][e]);
    }
#pragma unroll
    for (int e = 0; e < 4; ++e)
        __builtin_amdgcn_raw_buffer_store_b128(f[e], dst, off + e * 1024, 0, AUX_SYS);
}

struct Desc { const char* src; char* dst; unsigned foff, fval, fmask; bool isy; };

__global__ void __launch_bounds__(512, 2) fused_gemm_silu(Params p) {
    __shared__ f32x4 red[4 * 32 * 64 + 1];
    const int lane = threadIdx.x & 63;
    const int wave = __builtin_amdgcn_readfirstlane(threadIdx.x >> 6);
    const int b = blockIdx.x;
    const int rank = p.rank, ri = rank >> 1, rj = rank & 1;
    char* const myws = p.ws[rank];
    unsigned* const cnt = (unsigned*)&red[4 * 32 * 64];
    const int xcd = b & 7, idx = b >> 3;
    STAMP_AT(0);

    if (threadIdx.x == 0) red[4 * 32 * 64] = f32x4{0.f, 0.f, 0.f, 0.f};
    __syncthreads();

    constexpr int C0 = COLO ? 0 : 20, NG = COLO ? 32 : 20;
    const int link = SPREAD ? idx - C0 + 1 : xcd;
    const bool copier = wave == 4 || wave == 5;
    if (copier && (idx < C0 || idx >= C0 + 8 || link < 1 || link > 7)) return;
    if (wave == 3 && idx >= NG) return;

    gu32* const seq = (gu32*)(myws + OFF_SEQ + (b * 8 + wave) * 64);
    unsigned call = poll(seq);
#define GET_CALL() do { call = __builtin_amdgcn_readfirstlane(call) + 1u; if (lane == 0) post(seq, call); } while (0)
    gu32* const tmo = (gu32*)(myws + OFF_TMO);
    const __amdgpu_buffer_rsrc_t buf = __builtin_amdgcn_make_buffer_rsrc((void*)myws, 0, WS_USED, RSRC_W3);

    if (copier) {
        const unsigned q = (unsigned)(rank ^ link) & (NR - 1);
        const int v = (SPREAD ? xcd : idx - C0) * 2 + (wave - 4);
        const bool same = (link & 1) == 0;
        const int nu = same ? 56 : 58;
        const int sub = v & 3, a = v >> 2;
        const int gb1 = v < 4 ? (2 * v + 1 - rj) * 4 + ri : 32 + rj * 16 + v - 4;
        const int sp = 2 * a + 1 - rj, cq = 12 + (q >> 1);
        const int ux = same ? 48 : 34;
        unsigned poff = OFF_TMO, gate = 0;
        bool used = true;
        if (lane < 8) { poff = OFF_EF + (q * 64 + gb1) * 128 + lane * 4; gate = lane; }
        else if (lane == 8) { poff = OFF_EF + (q * 64 + 32 + rj * 16 + 12 + (same ? a : ri)) * 128 + (4 + sub) * 4; gate = 32; }
        else if (lane < 13 && same) { poff = OFF_RF + (q * 8 + sp) * 128 + (sub + 4 * (lane - 9)) * 4; gate = 40; }
        else if (lane == 13) { poff = OFF_EF + (q * 64 + rank * 4 + a) * 128 + (4 + sub) * 4; gate = ux; }
        else if (lane >= 16 && lane < 32 && !same) {
            poff = OFF_YF + ((q * 32 + (v >> 3) * 16 + lane - 16) * 4 + ((v >> 1) & 3)) * 128; gate = 42 + lane - 16;
        } else used = false;
        gu32* const pa = (gu32*)(myws + poff);
        char* const rfa = p.ws[(rank ^ (2 * (lane & 3))) & (NR - 1)] + OFF_RF + (rank * 8 + q) * 128 + v * 4;
        const unsigned lo = lane * 16, yo = (lane >> 5) * NN * 4 + (lane & 31) * 16;

        GET_CALL();
        const unsigned fb = call * 64, thr = used ? fb + 32 : 0;
        const unsigned par = OFF_DATA + (call & 1) * P_BYTES;
        const char* const qp = p.ws[q] + par;
        char* const mp = myws + par;
        auto desc = [&](int u) {
            Desc d;
            d.fmask = 1; d.isy = false;
            unsigned so, slot, src = q, k, fs;
            if (u < 32) { so = P_EXP + gb1 * BLOCK; slot = v < 4 ? v : v + 4; k = u; fs = 0; }
            else if (u >= ux && u < ux + 8) { so = P_EXP + (rank * 4 + a) * BLOCK; slot = 4 + a; k = 4 * (u - ux) + sub; fs = sub; }
            else if (!same) { so = P_EXP + (32 + rj * 16 + 12 + ri) * BLOCK; slot = 20 + ri; k = v + 16 * (u - 32); fs = v; }
            else if (u < 40) { so = P_EXP + (32 + rj * 16 + 12 + a) * BLOCK; slot = 20 + a; k = 4 * (u - 32) + sub; fs = sub; }
            else { src = sp; slot = 8 + cq; so = P_STG + (sp * NSLOT + slot) * BLOCK; k = 4 * (u - 40) + sub; fs = sub; }
            d.src = qp + so + k * UNIT;
            d.dst = mp + P_STG + (src * NSLOT + slot) * BLOCK + k * UNIT;
            d.foff = OFF_SF + (src * NSLOT + slot) * 128 + fs * 4;
            d.fval = fb + k + 1;
            if (!same && u >= 32) {
                if (u == 32) d.fval = fb;
                if (u == 33) { d.fval = fb + 32; d.fmask = 15; }
                if (u >= 42) {
                    const int n = u - 42;
                    d.src = qp + P_Y + ((ri * 2 + (v >> 3)) * 16 + n) * 32768 + (v & 7) * UNIT;
                    d.dst = p.out + (((q >> 1) * 128 + v * 8) * NN + (q & 1) * 2048 + n * 128) * 4;
                    d.foff = OFF_DUMMY + (link * 16 + v) * 4;
                    d.isy = true;
                }
            }
            return d;
        };
        auto cleared = [&](unsigned seen) {
            const u64 m = __ballot(seen < thr);
            return m ? (int)__builtin_amdgcn_readlane(gate, __builtin_ctzll(m)) : nu;
        };
        u32x4 q0, q1, q2, q3;
        unsigned pend = 0, seen = 0;
        int lim = 0, u = 0;
        Desc cur = desc(0);
        unsigned pfo = OFF_DUMMY + (link * 16 + v) * 4, pfv = 0, pfm = 1;
        while (u < nu) {
            if (lim <= u) {
                const u64 t0 = __builtin_amdgcn_s_memrealtime();
                bool dead = false;
                for (;;) {
                    seen = poll(pa);
                    lim = cleared(seen);
                    if (lim > u) break;
                    if (__builtin_amdgcn_readlane(seen, 63) == call || __builtin_amdgcn_s_memrealtime() - t0 > DEADLINE_TICKS) {
                        dead = true;
                        break;
                    }
                    __builtin_amdgcn_s_sleep(2);
                }
                if (dead) {
                    if (lane == 0) {
                        __hip_atomic_store(tmo, call, __ATOMIC_RELAXED, __HIP_MEMORY_SCOPE_AGENT);
                        if (!same) *(float*)desc(57).dst = __builtin_nanf("");
                    }
                    return;
                }
                STAMP_AT(u == 0 ? 1 : 12 + (u >> 3));
            }
            pend = seen;
            asm volatile("s_nop 4\n\t"
                         "global_load_dwordx4 %0, %4, %5 sc0 sc1\n\t"
                         "global_load_dwordx4 %1, %4, %5 offset:1024 sc0 sc1\n\t"
                         "global_load_dwordx4 %2, %4, %5 offset:2048 sc0 sc1\n\t"
                         "global_load_dwordx4 %3, %4, %5 offset:3072 sc0 sc1\n\t"
                         "s_waitcnt vmcnt(0)"
                         : "=&v"(q0), "=&v"(q1), "=&v"(q2), "=&v"(q3) : "v"(lo), "s"(cur.src) : "memory");
            while (u + 1 < lim) {
                const Desc nxt = desc(u + 1);
                const unsigned o0 = cur.isy ? yo : lo, st = cur.isy ? 2 * NN * 4 : 1024;
                char* const fa = lane == 0 ? myws + pfo : rfa;
                asm volatile("s_nop 4\n\t"
                             "s_waitcnt vmcnt(8)\n\t"
                             "global_store_dwordx4 %6, %0, %11 sc0 sc1\n\t"
                             "global_load_dwordx4 %0, %10, %12 sc0 sc1\n\t"
                             "s_waitcnt vmcnt(8)\n\t"
                             "global_store_dwordx4 %7, %1, %11 sc0 sc1\n\t"
                             "global_load_dwordx4 %1, %10, %12 offset:1024 sc0 sc1\n\t"
                             "s_waitcnt vmcnt(8)\n\t"
                             "global_store_dwordx4 %8, %2, %11 sc0 sc1\n\t"
                             "global_load_dwordx4 %2, %10, %12 offset:2048 sc0 sc1\n\t"
                             "s_waitcnt vmcnt(6)\n\t"
                             "v_mov_b32 %5, %4\n\t"
                             "global_store_dwordx4 %9, %3, %11 sc0 sc1\n\t"
                             "s_mov_b64 exec, %15\n\t"
                             "global_store_dword %13, %14, off sc0 sc1\n\t"
                             "s_mov_b64 exec, -1\n\t"
                             "global_load_dword %4, %16, off sc0 sc1\n\t"
                             "global_load_dwordx4 %3, %10, %12 offset:3072 sc0 sc1"
                             : "+v"(q0), "+v"(q1), "+v"(q2), "+v"(q3), "+v"(pend), "=&v"(seen)
                             : "v"(o0), "v"(o0 + st), "v"(o0 + 2 * st), "v"(o0 + 3 * st), "v"(lo),
                               "s"(cur.dst), "s"(nxt.src), "v"(fa), "v"(pfv), "s"((u64)pfm), "v"(pa)
                             : "memory");
                pfo = cur.foff; pfv = cur.fval; pfm = cur.fmask;
                cur = nxt;
                ++u;
                if (lim < nu) lim = cleared(seen);
                if ((u & 7) == 0) STAMP_AT(2 + (u >> 3));
            }
            {
                const unsigned o0 = cur.isy ? yo : lo, st = cur.isy ? 2 * NN * 4 : 1024;
                char* const fa = lane == 0 ? myws + pfo : rfa;
                char* const fc = lane == 0 ? myws + cur.foff : rfa;
                asm volatile("s_nop 4\n\t"
                             "s_waitcnt vmcnt(0)\n\t"
                             "global_store_dwordx4 %5, %0, %9 sc0 sc1\n\t"
                             "global_store_dwordx4 %6, %1, %9 sc0 sc1\n\t"
                             "global_store_dwordx4 %7, %2, %9 sc0 sc1\n\t"
                             "global_store_dwordx4 %8, %3, %9 sc0 sc1\n\t"
                             "s_mov_b64 exec, %12\n\t"
                             "global_store_dword %10, %11, off sc0 sc1\n\t"
                             "s_mov_b64 exec, %15\n\t"
                             "s_waitcnt vmcnt(0)\n\t"
                             "global_store_dword %13, %14, off sc0 sc1\n\t"
                             "s_mov_b64 exec, -1"
                             : "+v"(q0), "+v"(q1), "+v"(q2), "+v"(q3), "+v"(pend)
                             : "v"(o0), "v"(o0 + st), "v"(o0 + 2 * st), "v"(o0 + 3 * st), "s"(cur.dst),
                               "v"(fa), "v"(pfv), "s"((u64)pfm), "v"(fc), "v"(cur.fval), "s"((u64)cur.fmask)
                             : "memory");
                pfo = cur.foff; pfv = cur.fval; pfm = 1;
                ++u;
                if (u < nu) cur = desc(u);
            }
        }
        STAMP_AT(11);
        return;
    }

    if (wave != 3) {
        const int gb = b >> 2, sub = b & 3;
        const bool isx = gb < 32;
        const __amdgpu_buffer_rsrc_t src =
            __builtin_amdgcn_make_buffer_rsrc((void*)(isx ? p.x : p.w), 0, IN_BYTES, RSRC_W3);
        const int r = lane & 31, h = lane >> 5;
        const unsigned voff = isx ? ((gb * 128 + r) * KS + sub * 128 + h * 8) * 4
                                  : ((sub * 128 + h * 8) * NN + (gb - 32) * 128 + r * 4) * 4;
        const unsigned dl = isx ? ((lane & 1) ? (gb & 3) * 2 + 1 - ((gb >> 2) & 1) : gb >> 2)
                                : (lane & 3) * 2 + ((gb >> 4) & 1);
        gu32* const ef = (gu32*)(p.ws[dl & (NR - 1)] + OFF_EF + (rank * 64 + gb) * 128 + sub * 4);
        const bool efl = lane < (isx ? 2 : 4);
        const int m0 = wave >= 6 ? wave - 6 : 2 + 2 * wave;
        f32x4 v0[8], v1[8];
        if (CONV_DELAY > 0 && wave != 6) __builtin_amdgcn_s_sleep(CONV_DELAY);
        conv_load(v0, src, isx, voff, m0);
        if (wave < 6) conv_load(v1, src, isx, voff, m0 + 1);
        GET_CALL();
        const unsigned eo = OFF_DATA + (call & 1) * P_BYTES + P_EXP + gb * BLOCK + (4 * m0 + sub) * UNIT + lane * 16;
        conv_store(v0, buf, isx, eo);
        if (wave < 6) conv_store(v1, buf, isx, eo + 4 * UNIT);
        asm volatile("s_waitcnt vmcnt(0)" ::: "memory");
        if (wave == 6 && efl) post(ef, call * 64 + 32);
        unsigned n = 0;
        if (lane == 0) n = __hip_atomic_fetch_add(cnt + 2, 1u, __ATOMIC_RELAXED, __HIP_MEMORY_SCOPE_WORKGROUP);
        if (__builtin_amdgcn_readfirstlane(n) == 4 && efl) post(ef + 4, call * 64 + 32);
        STAMP_AT(1);
    } else {
        GET_CALL();
    }
    if (wave >= 4 || idx >= NG) return;

    call = __builtin_amdgcn_readfirstlane(call);
    const unsigned fb = call * 64;
    const unsigned par = OFF_DATA + (call & 1) * P_BYTES;
    const int rh = (xcd & 1) * 4 + (idx & 3);
    const int cb = idx < (COLO ? 24 : 12) ? (xcd >> 1) * 3 + (idx >> 2) % 3 : 12 + (xcd >> 1);
    const int njob = !COLO && idx < 12 ? 2 : 1;
    unsigned vo = lane * 16;
    for (int job = 0; job < njob; ++job) {
        const bool own = COLO ? (idx >= 12 && idx < 24) || idx >= 28 : job == 1 || idx >= 16;
        const int a = rh >> 1;
        const int s = lane >> 3, jj = lane & 7;
        const int gx = own ? rank * 4 + a : (2 * a + 1 - rj) * 4 + ri, gw = 32 + rj * 16 + cb;
        const int sx = own ? 4 + a : a;
        const bool isw = jj >= 1 && jj <= 4;
        unsigned poff;
        if (s == rank) poff = OFF_EF + (rank * 64 + (isw ? gw : gx)) * 128 + (4 + wave) * 4;
        else if (!isw) poff = OFF_SF + (s * NSLOT + sx) * 128 + (own ? wave : 0) * 4;
        else poff = OFF_SF + (s * NSLOT + 8 + cb) * 128 +
                    (cb < 12 ? 0 : ((s ^ rank) & 1) && cb == 12 + ri ? wave + 4 * (jj - 1) : wave) * 4;
        if (lane == 63) poff = OFF_TMO;
        gu32* const pa = (gu32*)(myws + poff);
        unsigned xoff = par + (s == rank ? P_EXP + gx * BLOCK : P_STG + (s * NSLOT + sx) * BLOCK) + wave * UNIT + (rh & 1) * 2048;
        const unsigned wd = par + (s == rank ? P_EXP + gw * BLOCK : P_STG + (s * NSLOT + 8 + cb) * BLOCK) + wave * UNIT - xoff;
        unsigned thr = fb + wave + 1, seen = 0;

        f32x16 acc[2][4];
#pragma unroll
        for (int mt = 0; mt < 2; ++mt)
#pragma unroll
            for (int e = 0; e < 4; ++e)
#pragma unroll
                for (int k = 0; k < 16; ++k) acc[mt][e][k] = 0.f;
        u32x4 A[4][2], B[4][4];
#define FETCH(r) do { \
            u64 t = full_groups(__ballot(seen >= thr)); \
            if (!t) { \
                const u64 t0 = __builtin_amdgcn_s_memrealtime(); \
                for (;;) { \
                    seen = poll(pa); \
                    t = full_groups(__ballot(seen >= thr)); \
                    if (t) break; \
                    if (__builtin_amdgcn_readlane(seen, 63) == call || __builtin_amdgcn_s_memrealtime() - t0 > DEADLINE_TICKS) { \
                        if (lane == 0) __hip_atomic_store(tmo, call, __ATOMIC_RELAXED, __HIP_MEMORY_SCOPE_AGENT); \
                        seen = fb + 32; \
                        t = full_groups(__ballot(seen >= thr)); \
                        break; \
                    } \
                    __builtin_amdgcn_s_sleep(POLL_SLEEP); \
                } \
                asm volatile("" ::: "memory"); \
            } \
            const int g = __builtin_ctzll(t); \
            const unsigned xo = __builtin_amdgcn_readlane(xoff, g), wo = xo + __builtin_amdgcn_readlane(wd, g); \
            if ((lane & 56) == g) { thr += 4; xoff += 4 * UNIT; } \
            asm volatile("" : "+v"(vo)); \
            _Pragma("unroll") for (int mt = 0; mt < 2; ++mt) \
                A[r][mt] = __builtin_amdgcn_raw_buffer_load_b128(buf, vo + mt * 1024, xo, AUX_SYS); \
            _Pragma("unroll") for (int e = 0; e < 4; ++e) \
                B[r][e] = __builtin_amdgcn_raw_buffer_load_b128(buf, vo + e * 1024, wo, AUX_SYS); \
            __builtin_amdgcn_sched_barrier(0); \
        } while (0)
#define MFMA(r) do { \
            _Pragma("unroll") for (int mt = 0; mt < 2; ++mt) \
                _Pragma("unroll") for (int e = 0; e < 4; ++e) \
                    acc[mt][e] = __builtin_amdgcn_mfma_f32_32x32x16_f16( \
                        __builtin_bit_cast(f16x8, A[r][mt]), __builtin_bit_cast(f16x8, B[r][e]), acc[mt][e], 0, 0, 0); \
        } while (0)
        FETCH(0); FETCH(1); FETCH(2);
        for (int it = 0; it < 15; ++it) {
            FETCH(3); MFMA(0);
            FETCH(0); MFMA(1);
            FETCH(1); MFMA(2);
            FETCH(2); MFMA(3);
            if ((it & 1) == 1) STAMP_AT(2 + job * 12 + (it >> 1));
        }
        FETCH(3); MFMA(0); MFMA(1); MFMA(2); MFMA(3);
        STAMP_AT(9 + job * 12);

        if (job == 1) {
            while (__hip_atomic_load(cnt + 1, __ATOMIC_RELAXED, __HIP_MEMORY_SCOPE_WORKGROUP) < 4u)
                __builtin_amdgcn_s_sleep(1);
            __builtin_amdgcn_fence(__ATOMIC_ACQUIRE, "workgroup");
        }
#pragma unroll
        for (int mt = 0; mt < 2; ++mt)
#pragma unroll
            for (int k = 0; k < 16; ++k)
                red[(wave * 32 + mt * 16 + k) * 64 + lane] =
                    f32x4{acc[mt][0][k], acc[mt][1][k], acc[mt][2][k], acc[mt][3][k]};
        __builtin_amdgcn_fence(__ATOMIC_RELEASE, "workgroup");
        if (lane == 0) __hip_atomic_fetch_add(cnt, 1u, __ATOMIC_RELAXED, __HIP_MEMORY_SCOPE_WORKGROUP);
        while (__hip_atomic_load(cnt, __ATOMIC_RELAXED, __HIP_MEMORY_SCOPE_WORKGROUP) < 4u * (job + 1))
            __builtin_amdgcn_s_sleep(1);
        __builtin_amdgcn_fence(__ATOMIC_ACQUIRE, "workgroup");
        STAMP_AT(10 + job * 12);
        const bool dead = __hip_atomic_load(tmo, __ATOMIC_RELAXED, __HIP_MEMORY_SCOPE_AGENT) == call;
        const __amdgpu_buffer_rsrc_t od = own ? __builtin_amdgcn_make_buffer_rsrc((void*)p.out, 0, IN_BYTES, RSRC_W3) : buf;
        const unsigned pitch = own ? NN * 4 : 512;
        const unsigned obase = own ? (rh * 64 * NN + rj * 2048 + cb * 128) * 4 : par + P_Y + (rh * 16 + cb) * 32768;
        const unsigned ooff = obase + (wave * 16 + (lane >> 5) * 4) * pitch + (lane & 31) * 16;
#pragma unroll
        for (int t = 0; t < 8; ++t) {
            const int e = (wave * 8 + t) * 64 + lane;
            f32x4 y = (red[e] + red[e + 32 * 64]) + (red[e + 2 * 32 * 64] + red[e + 3 * 32 * 64]);
            f32x4 o;
#pragma unroll
            for (int k = 0; k < 4; ++k) {
                const float z = fminf(fmaxf(y[k], -60.f), 60.f);
                o[k] = dead ? __builtin_nanf("") : y[k] * __builtin_amdgcn_rcpf(1.f + __expf(-z));
            }
            __builtin_amdgcn_raw_buffer_store_b128(__builtin_bit_cast(u32x4, o), od,
                                                   ooff + ((t & 3) + (t >> 2) * 8) * pitch, 0, AUX_SYS);
        }
        if (!own) {
            __builtin_amdgcn_fence(__ATOMIC_RELEASE, "workgroup");
            if (lane == 0) __hip_atomic_fetch_add(cnt + 1, 1u, __ATOMIC_RELAXED, __HIP_MEMORY_SCOPE_WORKGROUP);
            asm volatile("s_waitcnt vmcnt(0)" ::: "memory");
            if (lane == 0)
                post((gu32*)(p.ws[(2 * a + 1 - rj) & (NR - 1)] + OFF_YF + ((rank * 32 + (rh & 1) * 16 + cb) * 4 + wave) * 128), fb + 32);
        }
        STAMP_AT(11 + job * 12);
    }
}

extern "C" void kernel_launch(int rank, int nranks, ncclComm_t comm,
                              hipStream_t stream,
                              void* const* d_in, const int* in_sizes, int n_in,
                              void* d_out, int out_size,
                              const int* in_dtypes, int out_dtype,
                              void* const* d_out_all,
                              void* const* d_ws_all, size_t ws_bytes) {
    if (nranks != NR || (unsigned)rank >= (unsigned)NR || ws_bytes < WS_USED || n_in != 2 ||
        in_sizes[0] != 4096 * 512 || in_sizes[1] != 4096 * 512 || out_size != 4096 * 512) {
        fprintf(stderr, "unexpected problem shape\n");
        return;
    }
    Params p;
    p.x = (const float*)d_in[0];
    p.w = (const float*)d_in[1];
    for (int r = 0; r < NR; ++r) p.ws[r] = (char*)d_ws_all[r];
    p.out = (char*)d_out;
    p.rank = rank;
    p.pad = 0;
    fused_gemm_silu<<<NWG, 512, 0, stream>>>(p);
    HIP_CHECK(hipGetLastError());
}
```

```cpp
#include <hip/hip_runtime.h>
#include <rccl/rccl.h>
#include <cstdio>

#define HIP_CHECK(expr) do { hipError_t _e = (expr); if (_e != hipSuccess) { \
    fprintf(stderr, "HIP error %s at %s:%d\n", hipGetErrorString(_e), __FILE__, __LINE__); \
    return; } } while (0)

typedef unsigned u32x4 __attribute__((ext_vector_type(4)));
typedef float f32x4 __attribute__((ext_vector_type(4)));
typedef float f32x16 __attribute__((ext_vector_type(16)));
typedef _Float16 f16x8 __attribute__((ext_vector_type(8)));
typedef __attribute__((address_space(1))) unsigned gu32;
typedef unsigned long long u64;

#define POLL_SLEEP 4
#define DELAY1 40
#define DELAY2 40
#define RD 4
#define HACK 0

constexpr int NR = 8;
constexpr int NN = 4096;
constexpr int KS = 512;
constexpr unsigned UNIT = 4096;
constexpr unsigned BLOCK = 32 * UNIT;
constexpr int NSLOT = 24;
constexpr unsigned IN_BYTES = 4096u * 512u * 4u;
constexpr int NWG = 256;
constexpr unsigned OFF_SEQ = 0;
constexpr unsigned OFF_EF = 64u << 10;
constexpr unsigned OFF_SF = 384u << 10;
constexpr unsigned OFF_RF = 224u << 10;
constexpr unsigned OFF_TMO = 240u << 10;
constexpr unsigned OFF_DUMMY = 192u << 10;
constexpr unsigned OFF_YF = 256u << 10;
constexpr unsigned OFF_DBG = 512u << 10;
constexpr unsigned OFF_DATA = 1u << 20;
constexpr unsigned P_EXP = 0;
constexpr unsigned P_STG = 64 * BLOCK;
constexpr unsigned P_Y = P_STG + NR * NSLOT * BLOCK;
constexpr unsigned P_BYTES = P_Y + 128 * 32768;
constexpr unsigned WS_USED = OFF_DATA + 2 * P_BYTES;
constexpr u64 DEADLINE_TICKS = 50000000ull;
constexpr int AUX_SYS = 17;
constexpr int RSRC_W3 = 0x00020000;

struct Params {
    const float* x;
    const float* w;
    char* ws[NR];
    char* out;
    int rank;
    int pad;
};

#define STAMP_AT(idx) do {} while (0)

__device__ __forceinline__ u32x4 pack8(float a0, float a1, float a2, float a3,
                                       float a4, float a5, float a6, float a7) {
    f16x8 h = {(_Float16)a0, (_Float16)a1, (_Float16)a2, (_Float16)a3,
               (_Float16)a4, (_Float16)a5, (_Float16)a6, (_Float16)a7};
    return __builtin_bit_cast(u32x4, h);
}
__device__ __forceinline__ unsigned poll(gu32* a) {
    return __hip_atomic_load(a, __ATOMIC_RELAXED, __HIP_MEMORY_SCOPE_SYSTEM);
}
__device__ __forceinline__ void post(gu32* a, unsigned v) {
    __hip_atomic_store(a, v, __ATOMIC_RELAXED, __HIP_MEMORY_SCOPE_SYSTEM);
}
__device__ __forceinline__ u64 full_groups(u64 m) {
    m |= 1ull << 63;
    m &= m >> 4; m &= m >> 2; m &= m >> 1;
    return m & 0x0101010101010101ull;
}

__device__ __forceinline__ void conv_load(f32x4 (&v)[8], __amdgpu_buffer_rsrc_t src, bool isx, unsigned voff, int m) {
    if (isx) {
#pragma unroll
        for (int t = 0; t < 8; ++t)
            v[t] = __builtin_bit_cast(f32x4, __builtin_amdgcn_raw_buffer_load_b128(
                src, voff + (t & 1) * 16, (t >> 1) * 32 * KS * 4 + m * 64, 0));
    } else {
#pragma unroll
        for (int t = 0; t < 8; ++t)
            v[t] = __builtin_bit_cast(f32x4, __builtin_amdgcn_raw_buffer_load_b128(
                src, voff, (m * 16 + t) * NN * 4, 0));
    }
}
__device__ __forceinline__ void conv_store(const f32x4 (&v)[8], __amdgpu_buffer_rsrc_t dst, bool isx, unsigned off) {
    u32x4 f[4];
    if (isx) {
#pragma unroll
        for (int e = 0; e < 4; ++e) {
            const f32x4 a = v[2 * e], b = v[2 * e + 1];
            f[e] = pack8(a.x, a.y, a.z, a.w, b.x, b.y, b.z, b.w);
        }
    } else {
#pragma unroll
        for (int e = 0; e < 4; ++e)
            f[e] = pack8(v[0][e], v[1][e], v[2][e], v[3][e], v[4][e], v[5][e], v[6][e], v[7][e]);
    }
#pragma unroll
    for (int e = 0; e < 4; ++e)
        __builtin_amdgcn_raw_buffer_store_b128(f[e], dst, off + e * 1024, 0, AUX_SYS);
}

struct Desc { const char* src; char* dst; unsigned foff, fval, fmask; bool isy; };

__global__ void __launch_bounds__(256, 1) fused_gemm_silu(Params p) {
    __shared__ f32x4 red[4 * 32 * 64 + 1];
    const int lane = threadIdx.x & 63;
    const int wave = __builtin_amdgcn_readfirstlane(threadIdx.x >> 6);
    const int b = blockIdx.x;
    const int rank = p.rank, ri = rank >> 1, rj = rank & 1;
    char* const myws = p.ws[rank];
    unsigned* const cnt = (unsigned*)&red[4 * 32 * 64];
    const int xcd = b & 7, idx = b >> 3;
    STAMP_AT(0);

    if (threadIdx.x == 0) red[4 * 32 * 64] = f32x4{0.f, 0.f, 0.f, 0.f};
    __syncthreads();

    const int link = xcd;
    const bool copier = idx >= 28;
    if (copier && link == 0) return;

    gu32* const seq = (gu32*)(myws + OFF_SEQ + (b * 4 + wave) * 64);
    unsigned call = poll(seq);
#define GET_CALL() do { call = __builtin_amdgcn_readfirstlane(call) + 1u; if (lane == 0) post(seq, call); } while (0)
    gu32* const tmo = (gu32*)(myws + OFF_TMO);
    const __amdgpu_buffer_rsrc_t buf = __builtin_amdgcn_make_buffer_rsrc((void*)myws, 0, WS_USED, RSRC_W3);

    if (copier) {
        const unsigned q = (unsigned)(rank ^ link) & (NR - 1);
        const int v = (idx & 3) * 4 + wave;
        const bool same = (link & 1) == 0;
        const int nu = same ? 56 : 58;
        const int sub = v & 3, a = v >> 2;
        const int gb1 = v < 4 ? (2 * v + 1 - rj) * 4 + ri : 32 + rj * 16 + v - 4;
        const int sp = 2 * a + 1 - rj, cq = 12 + (q >> 1);
        const int ux = same ? 48 : 34;
        auto ycol = [&](int n) { return n < 12 ? n : 12 + ((n + (q >> 1)) & 3); };
        unsigned poff = OFF_TMO, gate = 0;
        bool used = true;
        if (lane < 32) { poff = OFF_EF + (q * 64 + gb1) * 256 + lane * 4; gate = lane; }
        else if (lane == 32) { poff = OFF_EF + (q * 64 + 32 + rj * 16 + 12 + (same ? a : ri)) * 256 + (32 + sub) * 4; gate = 32; }
        else if (lane < 37 && same) { poff = OFF_RF + (q * 8 + sp) * 128 + (sub + 4 * (lane - 33)) * 4; gate = 40; }
        else if (lane == 37) { poff = OFF_EF + (q * 64 + rank * 4 + a) * 256 + (32 + sub) * 4; gate = ux; }
        else if (lane >= 40 && lane < 56 && !same) {
            poff = OFF_YF + ((q * 32 + (v >> 3) * 16 + ycol(lane - 40)) * 4 + ((v >> 1) & 3)) * 128; gate = 42 + lane - 40;
        } else used = false;
        gu32* const pa = (gu32*)(myws + poff);
        char* const rfa = p.ws[(rank ^ (2 * (lane & 3))) & (NR - 1)] + OFF_RF + (rank * 8 + q) * 128 + v * 4;
        const unsigned lo = lane * 16, yo = (lane >> 5) * NN * 4 + (lane & 31) * 16;

        GET_CALL();
        const unsigned fb = call * 64, thr = used ? fb + 32 : 0;
        const unsigned par = OFF_DATA + (call & 1) * P_BYTES;
        const char* const qp = p.ws[q] + par;
        char* const mp = myws + par;
        auto desc = [&](int u) {
            Desc d;
            d.fmask = 1; d.isy = false;
            unsigned so, slot, src = q, k, fs = 0;
            if (u < 32) { so = P_EXP + gb1 * BLOCK; slot = v < 4 ? v : v + 4; k = u; }
            else if (u >= ux && u < ux + 8) { so = P_EXP + (rank * 4 + a) * BLOCK; slot = 4 + a; k = 4 * (u - ux) + sub; }
            else if (!same) { so = P_EXP + (32 + rj * 16 + 12 + ri) * BLOCK; slot = 20 + ri; k = v + 16 * (u - 32); fs = a; }
            else if (u < 40) { so = P_EXP + (32 + rj * 16 + 12 + a) * BLOCK; slot = 20 + a; k = 4 * (u - 32) + sub; }
            else { src = sp; slot = 8 + cq; so = P_STG + (sp * NSLOT + slot) * BLOCK; k = 4 * (u - 40) + sub; }
            d.src = qp + so + k * UNIT;
            d.dst = mp + P_STG + (src * NSLOT + slot) * BLOCK + k * UNIT;
            d.foff = OFF_SF + ((src * NSLOT + slot) * 4 + (k & 3)) * 128 + fs * 4;
            d.fval = fb + k + 1;
            if (!same && u >= 32) {
                if (u == 32) d.foff = OFF_DUMMY + (link * 16 + v) * 128;
                if (u == 33) { d.fval = fb + 32; d.fmask = 15; }
                if (u >= 42) {
                    const int n = ycol(u - 42);
                    d.src = qp + P_Y + ((ri * 2 + (v >> 3)) * 16 + n) * 32768 + (v & 7) * UNIT;
                    d.dst = p.out + (((q >> 1) * 128 + v * 8) * NN + (q & 1) * 2048 + n * 128) * 4;
                    d.foff = OFF_DUMMY + (link * 16 + v) * 128;
                    d.isy = true;
                }
            }
            return d;
        };
        auto cleared = [&](unsigned seen) {
            const u64 m = __ballot(seen < thr);
            return m ? (int)__builtin_amdgcn_readlane(gate, __builtin_ctzll(m)) : nu;
        };
        u32x4 q0, q1, q2, q3;
        unsigned pend = 0, seen = 0;
        int lim = 0, u = 0;
        Desc cur = desc(0);
        unsigned pfo = OFF_DUMMY + (link * 16 + v) * 128, pfv = 0, pfm = 1;
        while (u < nu) {
            if (lim <= u) {
                const u64 t0 = __builtin_amdgcn_s_memrealtime();
                bool dead = false;
                for (;;) {
                    seen = poll(pa);
                    lim = cleared(seen);
                    if (lim > u) break;
                    if (__builtin_amdgcn_readlane(seen, 63) == call || __builtin_amdgcn_s_memrealtime() - t0 > DEADLINE_TICKS) {
                        dead = true;
                        break;
                    }
                    __builtin_amdgcn_s_sleep(2);
                }
                if (dead) {
                    if (lane == 0) {
                        __hip_atomic_store(tmo, call, __ATOMIC_RELAXED, __HIP_MEMORY_SCOPE_AGENT);
                        if (!same) *(float*)desc(57).dst = __builtin_nanf("");
                    }
                    return;
                }
                STAMP_AT(u == 0 ? 1 : 12 + (u >> 3));
            }
            pend = seen;
            asm volatile("s_nop 4\n\t"
                         "global_load_dwordx4 %0, %4, %5 sc0 sc1\n\t"
                         "global_load_dwordx4 %1, %4, %5 offset:1024 sc0 sc1\n\t"
                         "global_load_dwordx4 %2, %4, %5 offset:2048 sc0 sc1\n\t"
                         "global_load_dwordx4 %3, %4, %5 offset:3072 sc0 sc1\n\t"
                         "s_waitcnt vmcnt(0)"
                         : "=&v"(q0), "=&v"(q1), "=&v"(q2), "=&v"(q3) : "v"(lo), "s"(cur.src) : "memory");
            while (u + 1 < lim) {
                const Desc nxt = desc(u + 1);
                const unsigned o0 = cur.isy ? yo : lo, st = cur.isy ? 2 * NN * 4 : 1024;
                char* const fa = lane == 0 ? myws + pfo : rfa;
                asm volatile("s_nop 4\n\t"
                             "s_waitcnt vmcnt(8)\n\t"
                             "global_store_dwordx4 %6, %0, %11 sc0 sc1\n\t"
                             "global_load_dwordx4 %0, %10, %12 sc0 sc1\n\t"
                             "s_waitcnt vmcnt(8)\n\t"
                             "global_store_dwordx4 %7, %1, %11 sc0 sc1\n\t"
                             "global_load_dwordx4 %1, %10, %12 offset:1024 sc0 sc1\n\t"
                             "s_waitcnt vmcnt(8)\n\t"
                             "global_store_dwordx4 %8, %2, %11 sc0 sc1\n\t"
                             "global_load_dwordx4 %2, %10, %12 offset:2048 sc0 sc1\n\t"
                             "s_waitcnt vmcnt(6)\n\t"
                             "v_mov_b32 %5, %4\n\t"
                             "global_store_dwordx4 %9, %3, %11 sc0 sc1\n\t"
                             "s_mov_b64 exec, %15\n\t"
                             "global_store_dword %13, %14, off sc0 sc1\n\t"
                             "s_mov_b64 exec, -1\n\t"
                             "global_load_dword %4, %16, off sc0 sc1\n\t"
                             "global_load_dwordx4 %3, %10, %12 offset:3072 sc0 sc1"
                             : "+v"(q0), "+v"(q1), "+v"(q2), "+v"(q3), "+v"(pend), "=&v"(seen)
                             : "v"(o0), "v"(o0 + st), "v"(o0 + 2 * st), "v"(o0 + 3 * st), "v"(lo),
                               "s"(cur.dst), "s"(nxt.src), "v"(fa), "v"(pfv), "s"((u64)pfm), "v"(pa)
                             : "memory");
                pfo = cur.foff; pfv = cur.fval; pfm = cur.fmask;
                cur = nxt;
                ++u;
                if (lim < nu) lim = cleared(seen);
                if ((u & 7) == 0) STAMP_AT(2 + (u >> 3));
            }
            {
                const unsigned o0 = cur.isy ? yo : lo, st = cur.isy ? 2 * NN * 4 : 1024;
                char* const fa = lane == 0 ? myws + pfo : rfa;
                char* const fc = lane == 0 ? myws + cur.foff : rfa;
                asm volatile("s_nop 4\n\t"
                             "s_waitcnt vmcnt(0)\n\t"
                             "global_store_dwordx4 %5, %0, %9 sc0 sc1\n\t"
                             "global_store_dwordx4 %6, %1, %9 sc0 sc1\n\t"
                             "global_store_dwordx4 %7, %2, %9 sc0 sc1\n\t"
                             "global_store_dwordx4 %8, %3, %9 sc0 sc1\n\t"
                             "s_mov_b64 exec, %12\n\t"
                             "global_store_dword %10, %11, off sc0 sc1\n\t"
                             "s_mov_b64 exec, %15\n\t"
                             "s_waitcnt vmcnt(0)\n\t"
                             "global_store_dword %13, %14, off sc0 sc1\n\t"
                             "s_mov_b64 exec, -1"
                             : "+v"(q0), "+v"(q1), "+v"(q2), "+v"(q3), "+v"(pend)
                             : "v"(o0), "v"(o0 + st), "v"(o0 + 2 * st), "v"(o0 + 3 * st), "s"(cur.dst),
                               "v"(fa), "v"(pfv), "s"((u64)pfm), "v"(fc), "v"(cur.fval), "s"((u64)cur.fmask)
                             : "memory");
                pfo = cur.foff; pfv = cur.fval; pfm = 1;
                ++u;
                if (u < nu) cur = desc(u);
            }
        }
        STAMP_AT(11);
        return;
    }

    for (int share = 0; share < (idx >= 24 ? 2 : 1); ++share) {
        const int gb = share ? 44 + ((b >> 2) & 3) + ((b >> 4) & 1) * 16 : (b >> 2) + (b >= 176 ? 4 : 0), sub = b & 3;
        const bool isx = gb < 32;
        const __amdgpu_buffer_rsrc_t src =
            __builtin_amdgcn_make_buffer_rsrc((void*)(isx ? p.x : p.w), 0, IN_BYTES, RSRC_W3);
        const int r = lane & 31, h = lane >> 5;
        const unsigned voff = isx ? ((gb * 128 + r) * KS + sub * 128 + h * 8) * 4
                                  : ((sub * 128 + h * 8) * NN + (gb - 32) * 128 + r * 4) * 4;
        const unsigned dl = isx ? ((lane & 1) ? (gb & 3) * 2 + 1 - ((gb >> 2) & 1) : gb >> 2)
                                : (lane & 3) * 2 + ((gb >> 4) & 1);
        gu32* const ef = (gu32*)(p.ws[dl & (NR - 1)] + OFF_EF + (rank * 64 + gb) * 256 + sub * 4);
        const bool efl = lane < (isx ? 2 : 4);
        f32x4 v0[8];
        if (share == 0)
            for (int i = 0; i < wave; ++i) __builtin_amdgcn_s_sleep(DELAY1);
        for (int m = wave; m < 8; m += 4) {
            conv_load(v0, src, isx, voff, m);
            if (share == 0 && m < 4) GET_CALL();
            conv_store(v0, buf, isx, OFF_DATA + (call & 1) * P_BYTES + P_EXP + gb * BLOCK + (4 * m + sub) * UNIT + lane * 16);
            asm volatile("s_waitcnt vmcnt(0)" ::: "memory");
            if (efl && share == 0 && dl != rank && !(isx && lane == 0)) post(ef + 4 * m, call * 64 + 32);
            if (m < 4) { STAMP_AT(1); __builtin_amdgcn_s_sleep(DELAY2); }
        }
        unsigned n = 0;
        if (lane == 0) n = __hip_atomic_fetch_add(cnt + 2, 1u, __ATOMIC_RELAXED, __HIP_MEMORY_SCOPE_WORKGROUP);
        if ((__builtin_amdgcn_readfirstlane(n) & 3) == 3 && efl) post(ef + 32, call * 64 + 32);
        if (share == 0 && idx >= 24) __syncthreads();
        STAMP_AT(12);
    }

    call = __builtin_amdgcn_readfirstlane(call);
    const unsigned fb = call * 64;
    const unsigned par = OFF_DATA + (call & 1) * P_BYTES;
    const int rh = (xcd & 1) * 4 + (idx & 3);
    const int njob = idx < 4 ? 2 : 1;
    unsigned vo = lane * 16;
    for (int job = 0; job < njob; ++job) {
        const bool own = job == 1 || (idx >= 12 && idx < 24);
        const int cb = idx < 24 && job == 0 ? (xcd >> 1) * 3 + (idx >> 2) % 3 : 12 + (xcd >> 1);
        const int a = rh >> 1;
        const int s = lane >> 3, jj = lane & 7;
        const int gx = own ? rank * 4 + a : (2 * a + 1 - rj) * 4 + ri, gw = 32 + rj * 16 + cb;
        const int sx = own ? 4 + a : a;
        const bool isw = jj >= 1 && jj <= 4;
        unsigned poff;
        if (s == rank) poff = OFF_EF + (rank * 64 + (isw ? gw : gx)) * 256 + (32 + wave) * 4;
        else if (!isw) poff = OFF_SF + ((s * NSLOT + sx) * 4 + wave) * 128;
        else poff = OFF_SF + ((s * NSLOT + 8 + cb) * 4 + wave) * 128 + (((s ^ rank) & 1) && cb == 12 + ri ? jj - 1 : 0) * 4;
        if (lane == 63) poff = OFF_TMO;
        gu32* const pa = (gu32*)(myws + poff);
        unsigned xoff = par + (s == rank ? P_EXP + gx * BLOCK : P_STG + (s * NSLOT + sx) * BLOCK) + wave * UNIT + (rh & 1) * 2048;
        const unsigned wd = par + (s == rank ? P_EXP + gw * BLOCK : P_STG + (s * NSLOT + 8 + cb) * BLOCK) + wave * UNIT - xoff;
        unsigned thr = fb + wave + 1, seen = 0;
        const unsigned fin = lane == 63 ? ~0u : fb + wave + 29;

        f32x16 acc[2][4];
#pragma unroll
        for (int mt = 0; mt < 2; ++mt)
#pragma unroll
            for (int e = 0; e < 4; ++e)
#pragma unroll
                for (int k = 0; k < 16; ++k) acc[mt][e][k] = 0.f;
        u32x4 A[RD][2], B[RD][4];
        unsigned full = 0;
        int left = 64;
        unsigned pend = 0;
#pragma unroll
        for (int r = 0; r < RD; ++r) {
#pragma unroll
            for (int mt = 0; mt < 2; ++mt) A[r][mt] = u32x4{0, 0, 0, 0};
#pragma unroll
            for (int e = 0; e < 4; ++e) B[r][e] = u32x4{0, 0, 0, 0};
        }
        do {
            seen = seen > pend ? seen : pend;
            pend = poll(pa);
#pragma unroll
            for (int r = 0; r < RD; ++r) {
                u64 t = full_groups(__ballot(seen >= thr));
                if (!t && !full && left) {
                    const u64 t0 = __builtin_amdgcn_s_memrealtime();
                    for (;;) {
                        if (seen < fin) seen = poll(pa);
                        t = full_groups(__ballot(seen >= thr));
                        if (t) break;
                        if (__builtin_amdgcn_readlane(seen, 63) == call || __builtin_amdgcn_s_memrealtime() - t0 > DEADLINE_TICKS) {
                            if (lane == 0) __hip_atomic_store(tmo, call, __ATOMIC_RELAXED, __HIP_MEMORY_SCOPE_AGENT);
                            seen = fb + 32;
                            t = full_groups(__ballot(seen >= thr));
                            break;
                        }
                        __builtin_amdgcn_s_sleep(POLL_SLEEP);
                    }
                    asm volatile("" ::: "memory");
                }
                const int g = __builtin_ctzll(t | (1ull << 63));
                unsigned xo = __builtin_amdgcn_readlane(xoff, g), wo = xo + __builtin_amdgcn_readlane(wd, g);
                if (!t || HACK == 2) xo = wo = WS_USED;
                if ((lane & 56) == g) { thr += 4; xoff += 4 * UNIT; }
                if (t) { full |= 1u << r; --left; if ((left & 7) == 0) STAMP_AT(9 + job * 12 - (left >> 3)); }
                asm volatile("" : "+v"(vo));
#pragma unroll
                for (int mt = 0; mt < 2; ++mt)
                    A[r][mt] = __builtin_amdgcn_raw_buffer_load_b128(buf, vo + mt * 1024, xo, AUX_SYS);
#pragma unroll
                for (int e = 0; e < 4; ++e)
                    B[r][e] = __builtin_amdgcn_raw_buffer_load_b128(buf, vo + e * 1024, wo, AUX_SYS);
                __builtin_amdgcn_sched_barrier(0);
                const int c = (r + 1) % RD;
                if (full & (1u << c)) {
                    full &= ~(1u << c);
#pragma unroll
                    for (int mt = 0; mt < 2; ++mt)
#pragma unroll
                        for (int e = 0; e < 4; ++e)
                            acc[mt][e] = __builtin_amdgcn_mfma_f32_32x32x16_f16(
                                __builtin_bit_cast(f16x8, A[c][mt]), __builtin_bit_cast(f16x8, B[c][e]), acc[mt][e], 0, 0, 0);
                }
                __builtin_amdgcn_sched_barrier(0);
            }
        } while (left | full);
        STAMP_AT(13 + job * 12);

        if (job == 1) {
            while (__hip_atomic_load(cnt + 1, __ATOMIC_RELAXED, __HIP_MEMORY_SCOPE_WORKGROUP) < 4u)
                __builtin_amdgcn_s_sleep(1);
            __builtin_amdgcn_fence(__ATOMIC_ACQUIRE, "workgroup");
        }
#pragma unroll
        for (int mt = 0; mt < 2; ++mt)
#pragma unroll
            for (int k = 0; k < 16; ++k)
                red[(wave * 32 + mt * 16 + k) * 64 + lane] =
                    f32x4{acc[mt][0][k], acc[mt][1][k], acc[mt][2][k], acc[mt][3][k]};
        __builtin_amdgcn_fence(__ATOMIC_RELEASE, "workgroup");
        if (lane == 0) __hip_atomic_fetch_add(cnt, 1u, __ATOMIC_RELAXED, __HIP_MEMORY_SCOPE_WORKGROUP);
        while (__hip_atomic_load(cnt, __ATOMIC_RELAXED, __HIP_MEMORY_SCOPE_WORKGROUP) < 4u * (job + 1))
            __builtin_amdgcn_s_sleep(1);
        __builtin_amdgcn_fence(__ATOMIC_ACQUIRE, "workgroup");
        STAMP_AT(10 + job * 12);
        const bool dead = __hip_atomic_load(tmo, __ATOMIC_RELAXED, __HIP_MEMORY_SCOPE_AGENT) == call;
        const __amdgpu_buffer_rsrc_t od = own ? __builtin_amdgcn_make_buffer_rsrc((void*)p.out, 0, IN_BYTES, RSRC_W3) : buf;
        const unsigned pitch = own ? NN * 4 : 512;
        const unsigned obase = own ? (rh * 64 * NN + rj * 2048 + cb * 128) * 4 : par + P_Y + (rh * 16 + cb) * 32768;
        const unsigned ooff = obase + (wave * 16 + (lane >> 5) * 4) * pitch + (lane & 31) * 16;
#pragma unroll
        for (int t = 0; t < 8; ++t) {
            const int e = (wave * 8 + t) * 64 + lane;
            f32x4 y = (red[e] + red[e + 32 * 64]) + (red[e + 2 * 32 * 64] + red[e + 3 * 32 * 64]);
            f32x4 o;
#pragma unroll
            for (int k = 0; k < 4; ++k) {
                const float z = fminf(fmaxf(y[k], -60.f), 60.f);
                o[k] = dead ? __builtin_nanf("") : y[k] * __builtin_amdgcn_rcpf(1.f + __expf(-z));
            }
            __builtin_amdgcn_raw_buffer_store_b128(__builtin_bit_cast(u32x4, o), od,
                                                   ooff + ((t & 3) + (t >> 2) * 8) * pitch, 0, AUX_SYS);
        }
        if (!own) {
            __builtin_amdgcn_fence(__ATOMIC_RELEASE, "workgroup");
            if (lane == 0) __hip_atomic_fetch_add(cnt + 1, 1u, __ATOMIC_RELAXED, __HIP_MEMORY_SCOPE_WORKGROUP);
            asm volatile("s_waitcnt vmcnt(0)" ::: "memory");
            if (lane == 0)
                post((gu32*)(p.ws[(2 * a + 1 - rj) & (NR - 1)] + OFF_YF + ((rank * 32 + (rh & 1) * 16 + cb) * 4 + wave) * 128), fb + 32);
        }
        STAMP_AT(11 + job * 12);
    }
}

extern "C" void kernel_launch(int rank, int nranks, ncclComm_t comm,
                              hipStream_t stream,
                              void* const* d_in, const int* in_sizes, int n_in,
                              void* d_out, int out_size,
                              const int* in_dtypes, int out_dtype,
                              void* const* d_out_all,
                              void* const* d_ws_all, size_t ws_bytes) {
    if (nranks != NR || (unsigned)rank >= (unsigned)NR || ws_bytes < WS_USED || n_in != 2 ||
        in_sizes[0] != 4096 * 512 || in_sizes[1] != 4096 * 512 || out_size != 4096 * 512) {
        fprintf(stderr, "unexpected problem shape\n");
        return;
    }
    Params p;
    p.x = (const float*)d_in[0];
    p.w = (const float*)d_in[1];
    for (int r = 0; r < NR; ++r) p.ws[r] = (char*)d_ws_all[r];
    p.out = (char*)d_out;
    p.rank = rank;
    p.pad = 0;
    fused_gemm_silu<<<NWG, 256, 0, stream>>>(p);
    HIP_CHECK(hipGetLastError());
}
```

```cpp
#include <hip/hip_runtime.h>
#include <rccl/rccl.h>
#include <cstdio>

#define HIP_CHECK(expr) do { hipError_t _e = (expr); if (_e != hipSuccess) { \
    fprintf(stderr, "HIP error %s at %s:%d\n", hipGetErrorString(_e), __FILE__, __LINE__); \
    return; } } while (0)

typedef unsigned u32x4 __attribute__((ext_vector_type(4)));
typedef float f32x4 __attribute__((ext_vector_type(4)));
typedef float f32x16 __attribute__((ext_vector_type(16)));
typedef _Float16 f16x8 __attribute__((ext_vector_type(8)));
typedef __attribute__((address_space(1))) unsigned gu32;
typedef unsigned long long u64;

#define POLL_SLEEP 4
#define DELAY1 40
#define DELAY2 40

constexpr int NR = 8;
constexpr int NN = 4096;
constexpr int KS = 512;
constexpr unsigned UNIT = 4096;
constexpr unsigned BLOCK = 32 * UNIT;
constexpr int NSLOT = 24;
constexpr unsigned IN_BYTES = 4096u * 512u * 4u;
constexpr int NWG = 256;
constexpr unsigned OFF_SEQ = 0;
constexpr unsigned OFF_EF = 64u << 10;
constexpr unsigned OFF_SF = 384u << 10;
constexpr unsigned OFF_RF = 224u << 10;
constexpr unsigned OFF_NONE = 240u << 10;
constexpr unsigned OFF_DUMMY = 192u << 10;
constexpr unsigned OFF_YF = 256u << 10;
constexpr unsigned OFF_DBG = 512u << 10;
constexpr unsigned OFF_DATA = 1u << 20;
constexpr unsigned P_EXP = 0;
constexpr unsigned P_STG = 64 * BLOCK;
constexpr unsigned P_Y = P_STG + NR * NSLOT * BLOCK;
constexpr unsigned P_BYTES = P_Y + 128 * 32768;
constexpr unsigned WS_USED = OFF_DATA + 2 * P_BYTES;
constexpr u64 DEADLINE_TICKS = 50000000ull;
constexpr int AUX_SYS = 17;
constexpr int RSRC_W3 = 0x00020000;

struct Params {
    const float* x;
    const float* w;
    char* ws[NR];
    char* out;
    int rank;
    int pad;
};

#define STAMP_AT(idx) do {} while (0)

__device__ __forceinline__ u32x4 pack8(float a0, float a1, float a2, float a3,
                                       float a4, float a5, float a6, float a7) {
    f16x8 h = {(_Float16)a0, (_Float16)a1, (_Float16)a2, (_Float16)a3,
               (_Float16)a4, (_Float16)a5, (_Float16)a6, (_Float16)a7};
    return __builtin_bit_cast(u32x4, h);
}
__device__ __forceinline__ unsigned poll(gu32* a) {
    return __hip_atomic_load(a, __ATOMIC_RELAXED, __HIP_MEMORY_SCOPE_SYSTEM);
}
__device__ __forceinline__ void post(gu32* a, unsigned v) {
    __hip_atomic_store(a, v, __ATOMIC_RELAXED, __HIP_MEMORY_SCOPE_SYSTEM);
}

__device__ __forceinline__ void conv_load(f32x4 (&v)[8], __amdgpu_buffer_rsrc_t src, bool isx, unsigned voff, int m) {
    if (isx) {
#pragma unroll
        for (int t = 0; t < 8; ++t)
            v[t] = __builtin_bit_cast(f32x4, __builtin_amdgcn_raw_buffer_load_b128(
                src, voff + (t & 1) * 16, (t >> 1) * 32 * KS * 4 + m * 64, 0));
    } else {
#pragma unroll
        for (int t = 0; t < 8; ++t)
            v[t] = __builtin_bit_cast(f32x4, __builtin_amdgcn_raw_buffer_load_b128(
                src, voff, (m * 16 + t) * NN * 4, 0));
    }
}
__device__ __forceinline__ void conv_store(const f32x4 (&v)[8], __amdgpu_buffer_rsrc_t dst, bool isx, unsigned off) {
    u32x4 f[4];
    if (isx) {
#pragma unroll
        for (int e = 0; e < 4; ++e) {
            const f32x4 a = v[2 * e], b = v[2 * e + 1];
            f[e] = pack8(a.x, a.y, a.z, a.w, b.x, b.y, b.z, b.w);
        }
    } else {
#pragma unroll
        for (int e = 0; e < 4; ++e)
            f[e] = pack8(v[0][e], v[1][e], v[2][e], v[3][e], v[4][e], v[5][e], v[6][e], v[7][e]);
    }
#pragma unroll
    for (int e = 0; e < 4; ++e)
        __builtin_amdgcn_raw_buffer_store_b128(f[e], dst, off + e * 1024, 0, AUX_SYS);
}

#define CLOBBER_V \
    "v58", "v59", "v60", "v61", "v62", "v63", "v64", "v65", "v66", "v67", "v68", "v69", "v70", "v71", "v72", \
    "v73", "v74", "v75", "v76", "v77", "v78", "v79", "v80", "v81", "v82", "v83", "v84", "v85", "v86", "v87", \
    "v88", "v89", "v90", "v91", "v92", "v93", "v94", "v95", "v96", "v97", "v98", "v99", "v100", "v101", "v102", \
    "v103", "v104", "v105", "v106", "v107", "v108", "v109", "v110", "v111", "v112", "v113", "v114", "v115", \
    "v116", "v117", "v118", "v119", "v120", "v121", "v122", "v123", "v124", "v125", "v126", "v127", "v128", \
    "v129", "v130", "v131", "v132", "v133", "v134", "v135", "v136", "v137", "v138", "v139", "v140", "v141", \
    "v142", "v143", "v144", "v145", "v146", "v147", "v148", "v149", "v150", "v151", "v152", "v153", "v154", \
    "v155", "v156", "v157", "v158", "v159", "v160", "v161", "v162", "v163", "v164", "v165", "v166", "v167", \
    "v168", "v169", "v170", "v171", "v172", "v173", "v174", "v175", "v176", "v177", "v178", "v179", "v180", \
    "v181", "v182", "v183", "v184", "v185", "v186", "v187", "v188", "v189", "v190", "v191", "v192", "v193", \
    "v194", "v195", "v196", "v197", "v198", "v199", "v200", "v201", "v202", "v203", "v204", "v205", "v206", \
    "v207", "v208", "v209", "v210", "v211", "v212", "v213", "v214", "v215", "v216", "v217", "v218", "v219", \
    "v220", "v221", "v222", "v223", "v224", "v225", "v226", "v227", "v228", "v229", "v230", "v231", "v232", \
    "v233", "v234", "v235", "v236", "v237", "v238", "v239", "v240", "v241", "v242", "v243", "v244", "v245", \
    "v246", "v247", "v248", "v249", "v250", "v251", "v252", "v253", "v254", "v255"
#define CLOBBER_A \
    "a0", "a1", "a2", "a3", "a4", "a5", "a6", "a7", "a8", "a9", "a10", "a11", "a12", "a13", "a14", "a15", "a16", \
    "a17", "a18", "a19", "a20", "a21", "a22", "a23", "a24", "a25", "a26", "a27", "a28", "a29", "a30", "a31", \
    "a32", "a33", "a34", "a35", "a36", "a37", "a38", "a39", "a40", "a41", "a42", "a43", "a44", "a45", "a46", \
    "a47", "a48", "a49", "a50", "a51", "a52", "a53", "a54", "a55", "a56", "a57", "a58", "a59", "a60", "a61", \
    "a62", "a63", "a64", "a65", "a66", "a67", "a68", "a69", "a70", "a71", "a72", "a73", "a74", "a75", "a76", \
    "a77", "a78", "a79", "a80", "a81", "a82", "a83", "a84", "a85", "a86", "a87", "a88", "a89", "a90", "a91", \
    "a92", "a93", "a94", "a95", "a96", "a97", "a98", "a99", "a100", "a101", "a102", "a103", "a104", "a105", \
    "a106", "a107", "a108", "a109", "a110", "a111", "a112", "a113", "a114", "a115", "a116", "a117", "a118", \
    "a119", "a120", "a121", "a122", "a123", "a124", "a125", "a126", "a127"

struct Desc { const char* src; char* dst; unsigned foff, fval, fmask; bool isy; };

__global__ void __launch_bounds__(256, 1) fused_gemm_silu(Params p) {
    __shared__ f32x4 red[4 * 32 * 64 + 1];
    const int lane = threadIdx.x & 63;
    const int wave = __builtin_amdgcn_readfirstlane(threadIdx.x >> 6);
    const int b = blockIdx.x;
    const int rank = p.rank, ri = rank >> 1, rj = rank & 1;
    char* const myws = p.ws[rank];
    unsigned* const cnt = (unsigned*)&red[4 * 32 * 64];
    const int xcd = b & 7, idx = b >> 3;
    STAMP_AT(0);

    if (threadIdx.x == 0) red[4 * 32 * 64] = f32x4{0.f, 0.f, 0.f, 0.f};
    __syncthreads();

    const int link = xcd;
    const bool copier = idx >= 28;
    if (copier && link == 0) return;

    gu32* const seq = (gu32*)(myws + OFF_SEQ + (b * 4 + wave) * 64);
    unsigned call = poll(seq);
#define GET_CALL() do { call = __builtin_amdgcn_readfirstlane(call) + 1u; if (lane == 0) post(seq, call); } while (0)
    const __amdgpu_buffer_rsrc_t buf = __builtin_amdgcn_make_buffer_rsrc((void*)myws, 0, WS_USED, RSRC_W3);

    if (copier) {
        const unsigned q = (unsigned)(rank ^ link) & (NR - 1);
        const int v = (idx & 3) * 4 + wave;
        const bool same = (link & 1) == 0;
        const int nu = same ? 56 : 58;
        const int sub = v & 3, a = v >> 2;
        const int gb1 = v < 4 ? (2 * v + 1 - rj) * 4 + ri : 32 + rj * 16 + v - 4;
        const int sp = 2 * a + 1 - rj, cq = 12 + (q >> 1);
        const int ux = same ? 48 : 34;
        auto ycol = [&](int n) { return n < 12 ? n : 12 + ((n + (q >> 1)) & 3); };
        unsigned poff = OFF_NONE, gate = 0;
        bool used = true;
        if (lane < 32) { poff = OFF_EF + (q * 64 + gb1) * 256 + lane * 4; gate = lane; }
        else if (lane == 32) { poff = OFF_EF + (q * 64 + 32 + rj * 16 + 12 + (same ? a : ri)) * 256 + (32 + sub) * 4; gate = 32; }
        else if (lane < 37 && same) { poff = OFF_RF + (q * 8 + sp) * 128 + (sub + 4 * (lane - 33)) * 4; gate = 40; }
        else if (lane == 37) { poff = OFF_EF + (q * 64 + rank * 4 + a) * 256 + (32 + sub) * 4; gate = ux; }
        else if (lane >= 40 && lane < 56 && !same) {
            poff = OFF_YF + ((q * 32 + (v >> 3) * 16 + ycol(lane - 40)) * 4 + ((v >> 1) & 3)) * 128; gate = 42 + lane - 40;
        } else used = false;
        gu32* const pa = (gu32*)(myws + poff);
        char* const rfa = p.ws[(rank ^ (2 * (lane & 3))) & (NR - 1)] + OFF_RF + (rank * 8 + q) * 128 + v * 4;
        const unsigned lo = lane * 16, yo = (lane >> 5) * NN * 4 + (lane & 31) * 16;

        GET_CALL();
        const unsigned fb = call * 64, thr = used ? fb + 32 : 0;
        const unsigned par = OFF_DATA + (call & 1) * P_BYTES;
        const char* const qp = p.ws[q] + par;
        char* const mp = myws + par;
        auto desc = [&](int u) {
            Desc d;
            d.fmask = 1; d.isy = false;
            unsigned so, slot, src = q, k, fs = 0;
            if (u < 32) { so = P_EXP + gb1 * BLOCK; slot = v < 4 ? v : v + 4; k = u; }
            else if (u >= ux && u < ux + 8) { so = P_EXP + (rank * 4 + a) * BLOCK; slot = 4 + a; k = 4 * (u - ux) + sub; }
            else if (!same) { so = P_EXP + (32 + rj * 16 + 12 + ri) * BLOCK; slot = 20 + ri; k = v + 16 * (u - 32); fs = a; }
            else if (u < 40) { so = P_EXP + (32 + rj * 16 + 12 + a) * BLOCK; slot = 20 + a; k = 4 * (u - 32) + sub; }
            else { src = sp; slot = 8 + cq; so = P_STG + (sp * NSLOT + slot) * BLOCK; k = 4 * (u - 40) + sub; }
            d.src = qp + so + k * UNIT;
            d.dst = mp + P_STG + (src * NSLOT + slot) * BLOCK + k * UNIT;
            d.foff = OFF_SF + ((src * NSLOT + slot) * 4 + (k & 3)) * 128 + fs * 4;
            d.fval = fb + k + 1;
            if (!same && u >= 32) {
                if (u == 32) d.foff = OFF_DUMMY + (link * 16 + v) * 128;
                if (u == 33) { d.fval = fb + 32; d.fmask = 15; }
                if (u >= 42) {
                    const int n = ycol(u - 42);
                    d.src = qp + P_Y + ((ri * 2 + (v >> 3)) * 16 + n) * 32768 + (v & 7) * UNIT;
                    d.dst = p.out + (((q >> 1) * 128 + v * 8) * NN + (q & 1) * 2048 + n * 128) * 4;
                    d.foff = OFF_DUMMY + (link * 16 + v) * 128;
                    d.isy = true;
                }
            }
            return d;
        };
        auto cleared = [&](unsigned seen) {
            const u64 m = __ballot(seen < thr);
            return m ? (int)__builtin_amdgcn_readlane(gate, __builtin_ctzll(m)) : nu;
        };
        u32x4 q0, q1, q2, q3;
        unsigned pend = 0, seen = 0;
        int lim = 0, u = 0;
        Desc cur = desc(0);
        unsigned pfo = OFF_DUMMY + (link * 16 + v) * 128, pfv = 0, pfm = 1;
        while (u < nu) {
            if (lim <= u) {
                const u64 t0 = __builtin_amdgcn_s_memrealtime();
                bool dead = false;
                for (;;) {
                    seen = poll(pa);
                    lim = cleared(seen);
                    if (lim > u) break;
                    if (__builtin_amdgcn_s_memrealtime() - t0 > DEADLINE_TICKS) {
                        dead = true;
                        break;
                    }
                    __builtin_amdgcn_s_sleep(2);
                }
                if (dead) {
                    if (lane == 0 && !same) *(float*)desc(57).dst = __builtin_nanf("");
                    return;
                }
                STAMP_AT(u == 0 ? 1 : 12 + (u >> 3));
            }
            pend = seen;
            asm volatile("s_nop 4\n\t"
                         "global_load_dwordx4 %0, %4, %5 sc0 sc1\n\t"
                         "global_load_dwordx4 %1, %4, %5 offset:1024 sc0 sc1\n\t"
                         "global_load_dwordx4 %2, %4, %5 offset:2048 sc0 sc1\n\t"
                         "global_load_dwordx4 %3, %4, %5 offset:3072 sc0 sc1\n\t"
                         "s_waitcnt vmcnt(0)"
                         : "=&v"(q0), "=&v"(q1), "=&v"(q2), "=&v"(q3) : "v"(lo), "s"(cur.src) : "memory");
            while (u + 1 < lim) {
                const Desc nxt = desc(u + 1);
                const unsigned o0 = cur.isy ? yo : lo, st = cur.isy ? 2 * NN * 4 : 1024;
                char* const fa = lane == 0 ? myws + pfo : rfa;
                asm volatile("s_nop 4\n\t"
                             "s_waitcnt vmcnt(8)\n\t"
                             "global_store_dwordx4 %6, %0, %11 sc0 sc1\n\t"
                             "global_load_dwordx4 %0, %10, %12 sc0 sc1\n\t"
                             "s_waitcnt vmcnt(8)\n\t"
                             "global_store_dwordx4 %7, %1, %11 sc0 sc1\n\t"
                             "global_load_dwordx4 %1, %10, %12 offset:1024 sc0 sc1\n\t"
                             "s_waitcnt vmcnt(8)\n\t"
                             "global_store_dwordx4 %8, %2, %11 sc0 sc1\n\t"
                             "global_load_dwordx4 %2, %10, %12 offset:2048 sc0 sc1\n\t"
                             "s_waitcnt vmcnt(6)\n\t"
                             "v_mov_b32 %5, %4\n\t"
                             "global_store_dwordx4 %9, %3, %11 sc0 sc1\n\t"
                             "s_mov_b64 exec, %15\n\t"
                             "global_store_dword %13, %14, off sc0 sc1\n\t"
                             "s_mov_b64 exec, -1\n\t"
                             "global_load_dword %4, %16, off sc0 sc1\n\t"
                             "global_load_dwordx4 %3, %10, %12 offset:3072 sc0 sc1"
                             : "+v"(q0), "+v"(q1), "+v"(q2), "+v"(q3), "+v"(pend), "=&v"(seen)
                             : "v"(o0), "v"(o0 + st), "v"(o0 + 2 * st), "v"(o0 + 3 * st), "v"(lo),
                               "s"(cur.dst), "s"(nxt.src), "v"(fa), "v"(pfv), "s"((u64)pfm), "v"(pa)
                             : "memory");
                pfo = cur.foff; pfv = cur.fval; pfm = cur.fmask;
                cur = nxt;
                ++u;
                if (lim < nu) lim = cleared(seen);
                if ((u & 7) == 0) STAMP_AT(2 + (u >> 3));
            }
            {
                const unsigned o0 = cur.isy ? yo : lo, st = cur.isy ? 2 * NN * 4 : 1024;
                char* const fa = lane == 0 ? myws + pfo : rfa;
                char* const fc = lane == 0 ? myws + cur.foff : rfa;
                asm volatile("s_nop 4\n\t"
                             "s_waitcnt vmcnt(0)\n\t"
                             "global_store_dwordx4 %5, %0, %9 sc0 sc1\n\t"
                             "global_store_dwordx4 %6, %1, %9 sc0 sc1\n\t"
                             "global_store_dwordx4 %7, %2, %9 sc0 sc1\n\t"
                             "global_store_dwordx4 %8, %3, %9 sc0 sc1\n\t"
                             "s_mov_b64 exec, %12\n\t"
                             "global_store_dword %10, %11, off sc0 sc1\n\t"
                             "s_mov_b64 exec, %15\n\t"
                             "s_waitcnt vmcnt(0)\n\t"
                             "global_store_dword %13, %14, off sc0 sc1\n\t"
                             "s_mov_b64 exec, -1"
                             : "+v"(q0), "+v"(q1), "+v"(q2), "+v"(q3), "+v"(pend)
                             : "v"(o0), "v"(o0 + st), "v"(o0 + 2 * st), "v"(o0 + 3 * st), "s"(cur.dst),
                               "v"(fa), "v"(pfv), "s"((u64)pfm), "v"(fc), "v"(cur.fval), "s"((u64)cur.fmask)
                             : "memory");
                pfo = cur.foff; pfv = cur.fval; pfm = 1;
                ++u;
                if (u < nu) cur = desc(u);
            }
        }
        STAMP_AT(11);
        return;
    }

    for (int share = 0; share < (idx >= 24 ? 2 : 1); ++share) {
        const int gb = share ? 44 + ((b >> 2) & 3) + ((b >> 4) & 1) * 16 : (b >> 2) + (b >= 176 ? 4 : 0), sub = b & 3;
        const bool isx = gb < 32;
        const __amdgpu_buffer_rsrc_t src =
            __builtin_amdgcn_make_buffer_rsrc((void*)(isx ? p.x : p.w), 0, IN_BYTES, RSRC_W3);
        const int r = lane & 31, h = lane >> 5;
        const unsigned voff = isx ? ((gb * 128 + r) * KS + sub * 128 + h * 8) * 4
                                  : ((sub * 128 + h * 8) * NN + (gb - 32) * 128 + r * 4) * 4;
        const unsigned dl = isx ? ((lane & 1) ? (gb & 3) * 2 + 1 - ((gb >> 2) & 1) : gb >> 2)
                                : (lane & 3) * 2 + ((gb >> 4) & 1);
        gu32* const ef = (gu32*)(p.ws[dl & (NR - 1)] + OFF_EF + (rank * 64 + gb) * 256 + sub * 4);
        const bool efl = lane < (isx ? 2 : 4);
        f32x4 v0[8];
        if (share == 0)
            for (int i = 0; i < wave; ++i) __builtin_amdgcn_s_sleep(DELAY1);
        for (int m = wave; m < 8; m += 4) {
            conv_load(v0, src, isx, voff, m);
            if (share == 0 && m < 4) GET_CALL();
            conv_store(v0, buf, isx, OFF_DATA + (call & 1) * P_BYTES + P_EXP + gb * BLOCK + (4 * m + sub) * UNIT + lane * 16);
            asm volatile("s_waitcnt vmcnt(0)" ::: "memory");
            if (efl && share == 0 && dl != rank && !(isx && lane == 0)) post(ef + 4 * m, call * 64 + 32);
            if (m < 4) { STAMP_AT(1); __builtin_amdgcn_s_sleep(DELAY2); }
        }
        unsigned n = 0;
        if (lane == 0) n = __hip_atomic_fetch_add(cnt + 2, 1u, __ATOMIC_RELAXED, __HIP_MEMORY_SCOPE_WORKGROUP);
        if ((__builtin_amdgcn_readfirstlane(n) & 3) == 3 && efl) post(ef + 32, call * 64 + 32);
        if (share == 0 && idx >= 24) __syncthreads();
        STAMP_AT(12);
    }

    call = __builtin_amdgcn_readfirstlane(call);
    const unsigned fb = call * 64;
    const unsigned par = OFF_DATA + (call & 1) * P_BYTES;
    const int rh = (xcd & 1) * 4 + (idx & 3);
    const int njob = idx < 4 ? 2 : 1;
    const u32x4 bufw = {(unsigned)(size_t)myws, (unsigned)((size_t)myws >> 32) & 0xffff, WS_USED, RSRC_W3};
    unsigned dead = 0;
    for (int job = 0; job < njob; ++job) {
        const bool own = job == 1 || (idx >= 12 && idx < 24);
        const int cb = idx < 24 && job == 0 ? (xcd >> 1) * 3 + (idx >> 2) % 3 : 12 + (xcd >> 1);
        const int a = rh >> 1;
        const int s = lane >> 3, jj = lane & 7;
        const int gx = own ? rank * 4 + a : (2 * a + 1 - rj) * 4 + ri, gw = 32 + rj * 16 + cb;
        const int sx = own ? 4 + a : a;
        const bool isw = jj >= 1 && jj <= 4;
        unsigned poff;
        if (s == rank) poff = OFF_EF + (rank * 64 + (isw ? gw : gx)) * 256 + (32 + wave) * 4;
        else if (!isw) poff = OFF_SF + ((s * NSLOT + sx) * 4 + wave) * 128;
        else poff = OFF_SF + ((s * NSLOT + 8 + cb) * 4 + wave) * 128 + (((s ^ rank) & 1) && cb == 12 + ri ? jj - 1 : 0) * 4;
        unsigned xoff = par + (s == rank ? P_EXP + gx * BLOCK : P_STG + (s * NSLOT + sx) * BLOCK) + wave * UNIT + (rh & 1) * 2048;
        const unsigned wd = par + (s == rank ? P_EXP + gw * BLOCK : P_STG + (s * NSLOT + 8 + cb) * BLOCK) + wave * UNIT - xoff;
        const unsigned step = jj == 0 ? 4 * UNIT : 0, xb0 = jj == 0 ? xoff : 0;
        const unsigned lds = (unsigned)(size_t)(__attribute__((address_space(3))) f32x4*)red + wave * 32768 + lane * 16;
        if (job == 1) __builtin_amdgcn_s_barrier();
        asm volatile(R"(
.macro g_x r, o
    buffer_load_dwordx4 v[64+24*\r+4*\o:67+24*\r+4*\o], %[vo], %[buf], s40 offen offset:1024*\o sc0 sc1
.endm
.macro g_w r, o
    buffer_load_dwordx4 v[72+24*\r+4*\o:75+24*\r+4*\o], %[vo], %[buf], s41 offen offset:1024*\o sc0 sc1
.endm
.macro g_m c, m, e
    v_mfma_f32_32x32x16_f16 a[64*\m+16*\e:64*\m+16*\e+15], v[64+24*\c+4*\m:67+24*\c+4*\m], v[72+24*\c+4*\e:75+24*\c+4*\e], a[64*\m+16*\e:64*\m+16*\e+15]
.endm
.macro g_poll r
  .if (\r & 3) == 0
    buffer_load_dword v[60+\r/4], %[poff], %[buf], 0 offen sc0 sc1
  .endif
.endm
.macro g_take p
    v_max_u32 v62, v62, v[60+\p]
    s_nop 1
    v_min_u32_dpp v63, v62, v62 quad_perm:[1,0,3,2] row_mask:0xf bank_mask:0xf
    v_cmp_gt_u32 vcc, %[fin], v62
    s_nop 0
    v_min_u32_dpp v63, v63, v63 quad_perm:[2,3,0,1] row_mask:0xf bank_mask:0xf
    v_cndmask_b32 %[poff], v58, %[poff], vcc
    s_nop 0
    v_min_u32_dpp v63, v63, v63 row_half_mirror row_mask:0xf bank_mask:0xf
    v_sub_u32 v63, v63, %[base] clamp
    v_lshrrev_b32 v63, 2, v63
    v_mad_u32_u24 v59, v63, %[step], %[xb0]
.endm
.macro g_sel
    v_cmp_gt_u32 vcc, v59, %[xoff]
    s_ff1_i32_b64 s42, vcc
    v_readlane_b32 s45, %[xoff], s42
    v_readlane_b32 s46, %[wd], s42
    v_cmp_eq_u32 vcc, s42, %[lane]
    s_mov_b32 s40, s45
    s_add_u32 s41, s45, s46
    v_cndmask_b32 v63, 0, %[step], vcc
    v_add_u32 %[xoff], %[xoff], v63
.endm
.macro g_wait r, c, np
    s_and_b32 s45, s43, 0xff-(1<<\c)
    s_bcnt1_i32_b32 s45, s45
  .irp n, 6, 5, 4, 3, 2, 1
    s_waitcnt vmcnt(6*\n+\np)
    s_cmp_eq_u32 s45, \n
    s_cbranch_scc1 Lw\r\()_%=
  .endr
    s_waitcnt vmcnt(\np)
Lw\r\()_%=:
.endm
.macro g_stamp r, k
)"
R"(
.endm
.macro g_fast r, c, np
Lf\r\()_%=:
    s_waitcnt vmcnt(36+\np)
    g_m \c, 0, 0
    g_poll \r
    g_x \r, 0
  .if (\c & 3) == 0
    v_max_u32 v62, v62, v[60+\c/4]
  .endif
    s_bitset1_b32 s43, \r
    g_m \c, 0, 1
    g_x \r, 1
  .if (\c & 3) == 0
    v_min_u32_dpp v63, v62, v62 quad_perm:[1,0,3,2] row_mask:0xf bank_mask:0xf
    v_cmp_gt_u32 vcc, %[fin], v62
  .endif
    s_bitset0_b32 s43, \c
    g_m \c, 0, 2
    g_w \r, 0
  .if (\c & 3) == 0
    v_min_u32_dpp v63, v63, v63 quad_perm:[2,3,0,1] row_mask:0xf bank_mask:0xf
    v_cndmask_b32 %[poff], v58, %[poff], vcc
  .endif
    s_sub_u32 s44, s44, 1
    g_m \c, 0, 3
    g_w \r, 1
  .if (\c & 3) == 0
    v_min_u32_dpp v63, v63, v63 row_half_mirror row_mask:0xf bank_mask:0xf
    v_sub_u32 v63, v63, %[base] clamp
    v_lshrrev_b32 v63, 2, v63
    v_mad_u32_u24 v59, v63, %[step], %[xb0]
  .endif
    g_m \c, 1, 0
    g_w \r, 2
    v_cmp_gt_u32 vcc, v59, %[xoff]
    s_ff1_i32_b64 s42, vcc
    v_readlane_b32 s45, %[xoff], s42
    v_readlane_b32 s46, %[wd], s42
    g_m \c, 1, 1
    g_w \r, 3
    v_cmp_eq_u32 vcc, s42, %[lane]
    s_mov_b32 s40, s45
    s_add_u32 s41, s45, s46
    g_m \c, 1, 2
    v_cndmask_b32 v63, 0, %[step], vcc
    v_add_u32 %[xoff], %[xoff], v63
    g_stamp \r, f
    s_ashr_i32 s45, s42, 31
    s_or_b32 s45, s45, s43
    s_cmp_eq_u32 s45, 0xff-(1<<\c)
    g_m \c, 1, 3
    s_cbranch_scc0 Lg\c\()_%=
.endm
.macro g_slow r, c, np
Lg\r\()_%=:
  .if (\c & 3) == 0
    g_wait \r, \c, \np
    g_take \c/4
    s_cmp_ge_i32 s42, 0
    s_cbranch_scc1 Lh\r\()_%=
    g_sel
Lh\r\()_%=:
    s_bitcmp1_b32 s43, \c
    s_cbranch_scc0 Ln\r\()_%=
  .else
    s_bitcmp1_b32 s43, \c
    s_cbranch_scc0 Ln\r\()_%=
    g_wait \r, \c, \np
  .endif
    s_cmp_ge_i32 s42, 0
    s_cbranch_scc1 Lf\r\()_%=
    g_poll \r
  .irp m, 0, 1
  .irp e, 0, 1, 2, 3
    g_m \c, \m, \e
  .endr
  .endr
    s_bitset0_b32 s43, \c
    s_branch Le\r\()_%=
Ln\r\()_%=:
    g_poll \r
    s_cmp_ge_i32 s42, 0
    s_cbranch_scc0 Le\r\()_%=
    g_x \r, 0
    g_x \r, 1
    g_w \r, 0
    g_w \r, 1
    g_w \r, 2
    g_w \r, 3
    s_bitset1_b32 s43, \r
    s_sub_u32 s44, s44, 1
    g_stamp \r, g
    g_sel
Le\r\()_%=:
    s_cmp_eq_u32 s43, 0
    s_cbranch_scc1 Lb_%=
.endm
    s_nop 4
    g_poll 0
    v_mov_b32 v61, 0
    v_mov_b32 v62, 0
    v_mov_b32 v58, %[oob]
    s_mov_b32 s43, 0
    s_mov_b32 s44, 64
  .set g_i, 0
  .rept 128
    v_accvgpr_write_b32 a[g_i], 0
  .set g_i, g_i+1
  .endr
Lb_%=:
    s_memrealtime s[48:49]
    s_waitcnt vmcnt(0)
    s_cmp_eq_u32 s44, 0
    s_cbranch_scc1 Ld_%=
    v_max_u32 v60, v60, v61
Lp_%=:
    v_max_u32 v60, %[dead], v60
    g_take 0
    g_sel
    s_cmp_ge_i32 s42, 0
    s_cbranch_scc1 Lg0_%=
    s_sleep %[slp]
    g_poll 0
    s_memrealtime s[50:51]
    s_waitcnt vmcnt(0) lgkmcnt(0)
    s_sub_u32 s45, s50, s48
    s_cmp_gt_u32 s45, %[ticks]
    s_cselect_b32 %[dead], %[last], %[dead]
    s_branch Lp_%=
    g_fast 0, 1, 1
    g_fast 1, 2, 2
    g_fast 2, 3, 2
    g_fast 3, 4, 1
    g_fast 4, 5, 1
    g_fast 5, 6, 2
    g_fast 6, 7, 2
    g_fast 7, 0, 1
    s_branch Lf0_%=
    g_slow 0, 1, 1
    g_slow 1, 2, 2
    g_slow 2, 3, 2
    g_slow 3, 4, 1
    g_slow 4, 5, 1
    g_slow 5, 6, 2
    g_slow 6, 7, 2
    g_slow 7, 0, 1
    s_branch Lg0_%=
Ld_%=:
    s_nop 7
  .set g_i, 0
  .rept 32
    ds_write_b128 %[lds], a[64*(g_i/16)+16*(g_i&3)+(g_i&12):64*(g_i/16)+16*(g_i&3)+(g_i&12)+3] offset:1024*g_i
  .set g_i, g_i+1
  .endr
    s_waitcnt lgkmcnt(0)
.purgem g_x
.purgem g_w
.purgem g_m
.purgem g_poll
.purgem g_take
.purgem g_sel
.purgem g_wait
.purgem g_stamp
.purgem g_fast
.purgem g_slow
)"
            : [poff] "+v"(poff), [xoff] "+v"(xoff), [dead] "+s"(dead)
            : [vo] "v"(lane * 16), [wd] "v"(wd), [step] "v"(step), [xb0] "v"(xb0), [lane] "v"(lane), [lds] "v"(lds),
              [buf] "s"(bufw), [fin] "s"(fb + wave + 29), [base] "s"(fb + wave - 3), [last] "s"(fb + 32),
              [oob] "n"(WS_USED), [slp] "n"(POLL_SLEEP), [ticks] "n"(DEADLINE_TICKS)
            : "memory", "vcc", "scc", "s40", "s41", "s42", "s43", "s44", "s45", "s46", "s48", "s49", "s50", "s51",
              CLOBBER_V, CLOBBER_A);
        if (dead) cnt[3] = 1;
        STAMP_AT(13 + job * 12);
        asm volatile("s_waitcnt lgkmcnt(0)\n\ts_barrier" ::: "memory");
        STAMP_AT(10 + job * 12);
        const bool bad = cnt[3] != 0;
        const __amdgpu_buffer_rsrc_t od = own ? __builtin_amdgcn_make_buffer_rsrc((void*)p.out, 0, IN_BYTES, RSRC_W3) : buf;
        const unsigned pitch = own ? NN * 4 : 512;
        const unsigned obase = own ? (rh * 64 * NN + rj * 2048 + cb * 128) * 4 : par + P_Y + (rh * 16 + cb) * 32768;
        const unsigned ooff = obase + (wave * 16 + (lane >> 5) * 4) * pitch + (lane & 31) * 16;
#pragma unroll
        for (int h = 0; h < 2; ++h) {
            f32x4 y[4];
#pragma unroll
            for (int e = 0; e < 4; ++e) {
                const int i = ((wave * 2 + h) * 4 + e) * 64 + lane;
                y[e] = (red[i] + red[i + 32 * 64]) + (red[i + 2 * 32 * 64] + red[i + 3 * 32 * 64]);
            }
#pragma unroll
            for (int t = 0; t < 4; ++t) {
                f32x4 o;
#pragma unroll
                for (int e = 0; e < 4; ++e) {
                    const float z = fminf(fmaxf(y[e][t], -60.f), 60.f);
                    o[e] = bad ? __builtin_nanf("") : y[e][t] * __builtin_amdgcn_rcpf(1.f + __expf(-z));
                }
                __builtin_amdgcn_raw_buffer_store_b128(__builtin_bit_cast(u32x4, o), od, ooff + (t + h * 8) * pitch, 0, AUX_SYS);
            }
        }
        if (!own) {
            asm volatile("s_waitcnt vmcnt(0)" ::: "memory");
            if (lane == 0)
                post((gu32*)(p.ws[(2 * a + 1 - rj) & (NR - 1)] + OFF_YF + ((rank * 32 + (rh & 1) * 16 + cb) * 4 + wave) * 128), fb + 32);
        }
        STAMP_AT(11 + job * 12);
    }
}

extern "C" void kernel_launch(int rank, int nranks, ncclComm_t comm,
                              hipStream_t stream,
                              void* const* d_in, const int* in_sizes, int n_in,
                              void* d_out, int out_size,
                              const int* in_dtypes, int out_dtype,
                              void* const* d_out_all,
                              void* const* d_ws_all, size_t ws_bytes) {
    if (nranks != NR || (unsigned)rank >= (unsigned)NR || ws_bytes < WS_USED || n_in != 2 ||
        in_sizes[0] != 4096 * 512 || in_sizes[1] != 4096 * 512 || out_size != 4096 * 512) {
        fprintf(stderr, "unexpected problem shape\n");
        return;
    }
    Params p;
    p.x = (const float*)d_in[0];
    p.w = (const float*)d_in[1];
    for (int r = 0; r < NR; ++r) p.ws[r] = (char*)d_ws_all[r];
    p.out = (char*)d_out;
    p.rank = rank;
    p.pad = 0;
    fused_gemm_silu<<<NWG, 256, 0, stream>>>(p);
    HIP_CHECK(hipGetLastError());
}
```

```cpp
#include <hip/hip_runtime.h>
#include <rccl/rccl.h>
#include <cstdio>

#define HIP_CHECK(expr) do { hipError_t _e = (expr); if (_e != hipSuccess) { \
    fprintf(stderr, "HIP error %s at %s:%d\n", hipGetErrorString(_e), __FILE__, __LINE__); \
    return; } } while (0)

typedef unsigned u32x4 __attribute__((ext_vector_type(4)));
typedef unsigned u32x2 __attribute__((ext_vector_type(2)));
typedef unsigned short u16x2 __attribute__((ext_vector_type(2)));
typedef float f32x4 __attribute__((ext_vector_type(4)));
typedef float f32x16 __attribute__((ext_vector_type(16)));
typedef _Float16 f16x8 __attribute__((ext_vector_type(8)));
typedef __attribute__((address_space(1))) unsigned gu32;
typedef volatile __attribute__((address_space(3))) unsigned short lds16;
typedef unsigned long long u64;

#define POLL_SLEEP 4
#define EARLY 2
#define DELAY3 4
#define DELAY1 40
#define DELAY2 40

constexpr int NR = 8;
constexpr int NN = 4096;
constexpr int KS = 512;
constexpr unsigned UNIT = 4096;
constexpr unsigned BLOCK = 32 * UNIT;
constexpr int NSLOT = 24;
constexpr unsigned IN_BYTES = 4096u * 512u * 4u;
constexpr int NWG = 256;
constexpr unsigned OFF_SEQ = 0;
constexpr unsigned OFF_EF = 64u << 10;
constexpr unsigned OFF_SF = 384u << 10;
constexpr unsigned OFF_RF = 224u << 10;
constexpr unsigned OFF_NONE = 240u << 10;
constexpr unsigned OFF_DUMMY = 192u << 10;
constexpr unsigned OFF_YF = 256u << 10;
constexpr unsigned OFF_DBG = 512u << 10;
constexpr unsigned OFF_DATA = 1u << 20;
constexpr unsigned PUNIT = 128 + UNIT;
constexpr unsigned PBLOCK = 32 * PUNIT;
constexpr unsigned P_EXP = 0;
constexpr unsigned P_RLY = 64 * PBLOCK;
constexpr unsigned P_STG = P_RLY + NR * PBLOCK;
constexpr unsigned P_Y = P_STG + NR * NSLOT * BLOCK;
constexpr unsigned P_BYTES = P_Y + 128 * 32768;
constexpr unsigned WS_USED = OFF_DATA + 2 * P_BYTES;
constexpr u64 DEADLINE_TICKS = 50000000ull;
constexpr int AUX_SYS = 17;
constexpr unsigned GROUP = 0xb2;
constexpr int RSRC_W3 = 0x00020000;

struct Params {
    const float* x;
    const float* w;
    char* ws[NR];
    char* out;
    int rank;
    unsigned links;
};

#define STAMP_AT(idx) do {} while (0)

__device__ __forceinline__ u32x4 pack8(float a0, float a1, float a2, float a3,
                                       float a4, float a5, float a6, float a7) {
    f16x8 h = {(_Float16)a0, (_Float16)a1, (_Float16)a2, (_Float16)a3,
               (_Float16)a4, (_Float16)a5, (_Float16)a6, (_Float16)a7};
    return __builtin_bit_cast(u32x4, h);
}
__device__ __forceinline__ unsigned poll(gu32* a) {
    return __hip_atomic_load(a, __ATOMIC_RELAXED, __HIP_MEMORY_SCOPE_SYSTEM);
}
__device__ __forceinline__ void post(gu32* a, unsigned v) {
    __hip_atomic_store(a, v, __ATOMIC_RELAXED, __HIP_MEMORY_SCOPE_SYSTEM);
}

__device__ __forceinline__ void conv_load(f32x4 (&v)[8], __amdgpu_buffer_rsrc_t src, bool isx, unsigned voff, int m) {
    if (isx) {
#pragma unroll
        for (int t = 0; t < 8; ++t)
            v[t] = __builtin_bit_cast(f32x4, __builtin_amdgcn_raw_buffer_load_b128(
                src, voff + (t & 1) * 16, (t >> 1) * 32 * KS * 4 + m * 64, 0));
    } else {
#pragma unroll
        for (int t = 0; t < 8; ++t)
            v[t] = __builtin_bit_cast(f32x4, __builtin_amdgcn_raw_buffer_load_b128(
                src, voff, (m * 16 + t) * NN * 4, 0));
    }
}
__device__ __forceinline__ void conv_half(const f32x4 (&v)[8], bool isx, u32x4 (&f)[4]) {
    if (isx) {
#pragma unroll
        for (int e = 0; e < 4; ++e) {
            const f32x4 a = v[2 * e], b = v[2 * e + 1];
            f[e] = pack8(a.x, a.y, a.z, a.w, b.x, b.y, b.z, b.w);
        }
    } else {
#pragma unroll
        for (int e = 0; e < 4; ++e)
            f[e] = pack8(v[0][e], v[1][e], v[2][e], v[3][e], v[4][e], v[5][e], v[6][e], v[7][e]);
    }
}
__device__ __forceinline__ unsigned bfi(unsigned m, unsigned a, unsigned b) { return (a & m) | (b & ~m); }

constexpr unsigned HI2 = 0x60006000u;
__device__ __forceinline__ void pack_store(const f32x4 (&v)[8], const u32x4 (&f)[4], __amdgpu_buffer_rsrc_t dst,
                                           unsigned off, lds16* lst, int lane) {
    float mx = 0.f;
#pragma unroll
    for (int t = 0; t < 8; ++t) {
        asm("v_max3_f32 %0, abs(%1), abs(%2), %0\n\tv_max3_f32 %0, abs(%3), abs(%4), %0"
            : "+v"(mx) : "v"(v[t].x), "v"(v[t].y), "v"(v[t].z), "v"(v[t].w));
    }
    lst[lane] = 0x800 | lane;
    unsigned c = __builtin_bit_cast(unsigned short, (_Float16)mx);
    c = max(c, (unsigned)__builtin_amdgcn_update_dpp(0, c, 0xb1, 0xf, 0xf, true));
    c = max(c, (unsigned)__builtin_amdgcn_update_dpp(0, c, 0x4e, 0xf, 0xf, true));
    c = max(c, (unsigned)__builtin_amdgcn_update_dpp(0, c, 0x141, 0xf, 0xf, true));
    c = max(c, (unsigned)__builtin_amdgcn_update_dpp(0, c, 0x140, 0xf, 0xf, true));
    const unsigned top = max(max((unsigned)__builtin_amdgcn_readlane(c, 0), (unsigned)__builtin_amdgcn_readlane(c, 16)),
                             max((unsigned)__builtin_amdgcn_readlane(c, 32), (unsigned)__builtin_amdgcn_readlane(c, 48)));
    const unsigned B = max(top & 0x7fff, 8191u) - 8191u;
    const u16x2 B2 = {(unsigned short)B, (unsigned short)B};
    unsigned d[16], e[2];
#pragma unroll
    for (int i = 0; i < 16; ++i) {
        const unsigned t = f[i >> 2][i & 3];
        const u16x2 r = __builtin_bit_cast(u16x2, t) - B2;
        d[i] = __builtin_bit_cast(unsigned, r);
    }
#pragma unroll
    for (int a = 0; a < 2; ++a) {
        unsigned x = d[7 * a] & HI2, g = bfi(0x40004000u, d[14 + a] >> 1, d[14 + a] << 1);
#pragma unroll
        for (int i = 1; i < 7; ++i) x = bfi(HI2, d[7 * a + i], x >> 2);
        e[a] = bfi(0xc000c000u, d[14 + a] << 1, x >> 1);
#pragma unroll
        for (int i = 6; i >= 0; --i) { d[7 * a + i] = bfi(HI2, g, d[7 * a + i]); g <<= 2; }
    }
    unsigned n = 0;
    for (;;) {
        const unsigned x = e[0] ? e[0] : e[1];
        const u64 m = __ballot(x != 0);
        const unsigned slot = n + __builtin_amdgcn_mbcnt_hi(m >> 32, __builtin_amdgcn_mbcnt_lo((unsigned)m, 0));
        n += __builtin_popcountll(m);
        if (m == 0 || n > 63) break;
        if (x) {
            const unsigned bit = __builtin_ctz(x) & 30, y = x & ~(3u << bit);
            lst[slot] = ((x >> bit) & 3) << 13 | (e[0] ? 0 : 1024) | (bit & 8) << 6 | lane << 3 | (bit & 6) | bit >> 4;
            if (e[0]) e[0] = y; else e[1] = y;
        }
    }
    unsigned short ent = lst[lane];
    if (lane == 63) ent = n > 63 ? 0x8000 : B;
    if (n > 63) {
#pragma unroll
        for (int i = 0; i < 14; ++i) d[i] = f[i >> 2][i & 3];
        __builtin_amdgcn_raw_buffer_store_b64(u32x2{f[3][2], f[3][3]}, dst, off + 128 + 3584 + lane * 8, 0, AUX_SYS);
    }
#pragma unroll
    for (int i = 0; i < 3; ++i)
        __builtin_amdgcn_raw_buffer_store_b128(u32x4{d[4 * i], d[4 * i + 1], d[4 * i + 2], d[4 * i + 3]}, dst,
                                               off + 128 + i * 1024 + lane * 16, 0, AUX_SYS);
    __builtin_amdgcn_raw_buffer_store_b64(u32x2{d[12], d[13]}, dst, off + 128 + 3072 + lane * 8, 0, AUX_SYS);
    __builtin_amdgcn_raw_buffer_store_b16(ent, dst, off + lane * 2, 0, AUX_SYS);
}

#define CLOBBER_V \
    "v58", "v59", "v60", "v61", "v62", "v63", "v64", "v65", "v66", "v67", "v68", "v69", "v70", "v71", "v72", \
    "v73", "v74", "v75", "v76", "v77", "v78", "v79", "v80", "v81", "v82", "v83", "v84", "v85", "v86", "v87", \
    "v88", "v89", "v90", "v91", "v92", "v93", "v94", "v95", "v96", "v97", "v98", "v99", "v100", "v101", "v102", \
    "v103", "v104", "v105", "v106", "v107", "v108", "v109", "v110", "v111", "v112", "v113", "v114", "v115", \
    "v116", "v117", "v118", "v119", "v120", "v121", "v122", "v123", "v124", "v125", "v126", "v127", "v128", \
    "v129", "v130", "v131", "v132", "v133", "v134", "v135", "v136", "v137", "v138", "v139", "v140", "v141", \
    "v142", "v143", "v144", "v145", "v146", "v147", "v148", "v149", "v150", "v151", "v152", "v153", "v154", \
    "v155", "v156", "v157", "v158", "v159", "v160", "v161", "v162", "v163", "v164", "v165", "v166", "v167", \
    "v168", "v169", "v170", "v171", "v172", "v173", "v174", "v175", "v176", "v177", "v178", "v179", "v180", \
    "v181", "v182", "v183", "v184", "v185", "v186", "v187", "v188", "v189", "v190", "v191", "v192", "v193", \
    "v194", "v195", "v196", "v197", "v198", "v199", "v200", "v201", "v202", "v203", "v204", "v205", "v206", \
    "v207", "v208", "v209", "v210", "v211", "v212", "v213", "v214", "v215", "v216", "v217", "v218", "v219", \
    "v220", "v221", "v222", "v223", "v224", "v225", "v226", "v227", "v228", "v229", "v230", "v231", "v232", \
    "v233", "v234", "v235", "v236", "v237", "v238", "v239", "v240", "v241", "v242", "v243", "v244", "v245", \
    "v246", "v247", "v248", "v249", "v250", "v251", "v252", "v253", "v254", "v255"
#define CLOBBER_A \
    "a0", "a1", "a2", "a3", "a4", "a5", "a6", "a7", "a8", "a9", "a10", "a11", "a12", "a13", "a14", "a15", "a16", \
    "a17", "a18", "a19", "a20", "a21", "a22", "a23", "a24", "a25", "a26", "a27", "a28", "a29", "a30", "a31", \
    "a32", "a33", "a34", "a35", "a36", "a37", "a38", "a39", "a40", "a41", "a42", "a43", "a44", "a45", "a46", \
    "a47", "a48", "a49", "a50", "a51", "a52", "a53", "a54", "a55", "a56", "a57", "a58", "a59", "a60", "a61", \
    "a62", "a63", "a64", "a65", "a66", "a67", "a68", "a69", "a70", "a71", "a72", "a73", "a74", "a75", "a76", \
    "a77", "a78", "a79", "a80", "a81", "a82", "a83", "a84", "a85", "a86", "a87", "a88", "a89", "a90", "a91", \
    "a92", "a93", "a94", "a95", "a96", "a97", "a98", "a99", "a100", "a101", "a102", "a103", "a104", "a105", \
    "a106", "a107", "a108", "a109", "a110", "a111", "a112", "a113", "a114", "a115", "a116", "a117", "a118", \
    "a119", "a120", "a121", "a122", "a123", "a124", "a125", "a126", "a127"

__global__ void __launch_bounds__(256, 1) fused_gemm_silu(Params p) {
    __shared__ f32x4 red[4 * 32 * 64 + 1];
    const int lane = threadIdx.x & 63;
    const int wave = __builtin_amdgcn_readfirstlane(threadIdx.x >> 6);
    const int b = blockIdx.x;
    const int rank = p.rank, ri = rank >> 1, rj = rank & 1;
    char* const myws = p.ws[rank];
    unsigned* const cnt = (unsigned*)&red[4 * 32 * 64];
    const int xcd = b & 7, idx = b >> 3;
    STAMP_AT(0);

    if (threadIdx.x == 0) red[4 * 32 * 64] = f32x4{0.f, 0.f, 0.f, 0.f};
    __syncthreads();

    const int link = (p.links >> (4 * xcd)) & 7;
    const bool copier = idx >= 28;
    if (copier && link == 0) return;

    gu32* const seq = (gu32*)(myws + OFF_SEQ + (b * 4 + wave) * 64);
    unsigned call = poll(seq);
#define GET_CALL() do { call = __builtin_amdgcn_readfirstlane(call) + 1u; if (lane == 0) post(seq, call); } while (0)
    const __amdgpu_buffer_rsrc_t buf = __builtin_amdgcn_make_buffer_rsrc((void*)myws, 0, WS_USED, RSRC_W3);

    if (copier) {
        const unsigned q = (unsigned)(rank ^ link) & (NR - 1);
        const int v = (idx & 3) * 4 + wave;
        const bool same = (link & 1) == 0;
        const int nu = same ? 56 : 58;
        const int sub = v & 3, a = v >> 2;
        auto slot1 = [&](int u) { const int c = v & 7; return u < 16 ? (c < 4 ? c : 3 * c - 4) : 9 + c + (c >> 1); };
        auto gb1 = [&](int u) { const int t = slot1(u); return t < 4 ? (2 * t + 1 - rj) * 4 + ri : 32 + rj * 16 + t - 8; };
        auto k1 = [&](int u) { return 2 * (u & 15) + (v >> 3); };
        const int sp = 2 * a + 1 - rj, cq = 12 + (q >> 1);
        const int ux = same ? 48 : 34;
        auto ycol = [&](int n) { return n < 4 ? 3 * n : n < 12 ? n - 3 + ((n - 4) >> 1) : 12 + ((n + (q >> 1)) & 3); };
        unsigned poff = OFF_NONE, gate = 0;
        bool used = true;
        if (lane < 32) { poff = OFF_EF + (q * 64 + gb1(lane)) * 256 + k1(lane) * 4; gate = lane; }
        else if (lane == 32) { poff = OFF_EF + (q * 64 + 32 + rj * 16 + 12 + (same ? a : ri)) * 256 + (32 + sub) * 4; gate = 32; }
        else if (lane < 37 && same) { poff = OFF_RF + (q * 8 + sp) * 128 + (sub + 4 * (lane - 33)) * 4; gate = 40; }
        else if (lane == 37) { poff = OFF_EF + (q * 64 + rank * 4 + a) * 256 + (32 + sub) * 4; gate = ux; }
        else if (lane >= 40 && lane < 56 && !same) {
            poff = OFF_YF + ((q * 32 + (v >> 3) * 16 + ycol(lane - 40)) * 4 + ((v >> 1) & 3)) * 128; gate = 42 + lane - 40;
        } else used = false;
        if (lane == 63) gate = nu;
        char* const rfa = p.ws[(rank ^ (2 * (lane & 3))) & (NR - 1)] + OFF_RF + (rank * 8 + q) * 128 + v * 4;
        const unsigned yo = (lane >> 5) * 4 * NN * 4 + (lane & 31) * 16;
        const unsigned tab = (unsigned)(size_t)(__attribute__((address_space(3))) f32x4*)red + wave * 32768;
#pragma unroll
        for (int i = 0; i < 5; ++i) red[wave * 2048 + i * 64 + lane] = f32x4{0.f, 0.f, 0.f, 0.f};

        GET_CALL();
        const unsigned fb = call * 64, thr = lane == 63 ? ~0u : used ? fb + 32 : 0;
        const unsigned par = OFF_DATA + (call & 1) * P_BYTES;
        unsigned so, dst, fo, fv;
        {
            const int u = lane;
            unsigned slot, src = q, k, fs = 0;
            if (u < 32) { so = P_EXP + gb1(u) * PBLOCK; slot = slot1(u); k = k1(u); }
            else if (u >= ux && u < ux + 8) { so = P_EXP + (rank * 4 + a) * PBLOCK; slot = 4 + a; k = 4 * (u - ux) + sub; }
            else if (!same) { so = P_EXP + (32 + rj * 16 + 12 + ri) * PBLOCK; slot = 20 + ri; k = v + 16 * (u & 1); fs = a; }
            else if (u < 40) { so = P_EXP + (32 + rj * 16 + 12 + a) * PBLOCK; slot = 20 + a; k = 4 * (u - 32) + sub; }
            else { src = sp; slot = 8 + cq; so = P_RLY + sp * PBLOCK; k = 4 * (u & 7) + sub; }
            so += par + k * PUNIT + 128;
            dst = par + P_STG + (src * NSLOT + slot) * BLOCK + k * UNIT;
            fo = OFF_SF + ((src * NSLOT + slot) * 4 + (k & 3)) * 128 + fs * 4;
            fv = fb + k + 1;
            if (!same && u == 33) fv = fb + 32;
            if (!same && u >= 42) {
                const int n = ycol((u - 42) & 15);
                so = par + P_Y + ((ri * 2 + (v >> 3)) * 16 + n) * 32768 + (v & 7) * UNIT;
                dst = (((q >> 1) * 128 + v * 8) * NN + (q & 1) * 2048 + n * 128) * 4;
            }
            if (lane >= nu || (!same && (u == 32 || u >= 42))) fo = OFF_DUMMY + (link * 16 + v) * 128;
        }
        unsigned dead = 0;
        asm volatile(R"(
.macro c_load r, i
  .if \i < 3
    global_load_dwordx4 v[\r+4*\i:\r+4*\i+3], %[l16], s[44:45] offset:1024*\i sc0 sc1
  .elseif \i == 3
    global_load_dwordx2 v[\r+12:\r+13], %[l8], s[44:45] offset:3072 sc0 sc1
  .else
    s_cmp_gt_u32 s41, %[uy]
    s_cbranch_scc1 Lca\@
    global_load_ushort v[\r+14], %[l2], s[44:45] offset:-128 sc0 sc1
    s_branch Lcb\@
Lca\@:
    global_load_dwordx2 v[\r+14:\r+15], %[l8], s[44:45] offset:3584 sc0 sc1
Lcb\@:
  .endif
.endm
.macro c_early r
    v_readlane_b32 s48, %[so], s41
    s_add_u32 s41, s41, 1
    s_add_u32 s44, s58, s48
    s_addc_u32 s45, s59, 0
  .irp i, 0, 1, 2, 3, 4
   .if \i < %[early]
    c_load \r, \i
   .endif
  .endr
.endm
.macro c_late r
  .irp i, 0, 1, 2, 3, 4
   .if \i >= %[early]
    c_load \r, \i
   .endif
  .endr
.endm
.macro c_flag
    s_sub_u32 s48, s40, 1
    s_lshl_b64 exec, 1, s48
    s_sub_u32 s48, s40, 2
    global_store_dword %[fo], %[fv], s[60:61] sc0 sc1
    s_cmp_eq_u32 s48, %[ur]
    s_cbranch_scc0 Lcc\@
    s_mov_b64 exec, 14
    global_store_dword %[rfa], %[thr], off sc0 sc1
Lcc\@:
    s_mov_b64 exec, -1
.endm
.macro c_take
    v_cmp_lt_u32 vcc, v112, %[thr]
    s_ff1_i32_b64 s48, vcc
    v_readlane_b32 s42, %[gate], s48
.endm
.macro c_poll
    global_load_dword v112, %[poff], s[60:61] sc0 sc1
.endm
.macro c_stamp n
)"
R"(
.endm
.macro c_step r, o
    s_sub_u32 s48, s41, s40
    s_cmp_eq_u32 s48, 2
    s_cbranch_scc0 Lcd\r\()_%=
    s_waitcnt vmcnt(%[early])
    c_late \o
    c_flag
    c_take
Lce\r\()_%=:
    v_readlane_b32 s48, %[dst], s40
    s_cmp_ge_u32 s40, %[uy]
    s_cbranch_scc1 Lcy\r\()_%=
    v_readlane_b32 s56, v[\r+14], 63
    s_add_u32 s46, s60, s48
    s_addc_u32 s47, s61, 0
    s_sub_u32 s49, s40, %[ur]
    s_cmp_lt_u32 s49, 2
    s_cbranch_scc1 Lcr\r\()_%=
Lcf\r\()_%=:
    s_bitcmp1_b32 s56, 15
    s_cbranch_scc1 Lcw\r\()_%=
    v_writelane_b32 v[\r+14], s57, 63
    s_pack_ll_b32_b16 s56, s56, s56
    v_and_b32 v113, 0xfff, v[\r+14]
    v_lshl_add_u32 v113, v113, 1, %[tab]
    v_and_b32 v114, s54, v[\r]
    ds_write_b16 v113, v[\r+14]
    ds_read_b128 v[96:99], %[tl]
    ds_read_b128 v[100:103], %[tl] offset:1024
    ds_read_b128 v[104:107], %[tl] offset:2048
    ds_read_b128 v[108:111], %[tl] offset:3072
    ds_write_b16 v113, v116
    v_and_b32 v115, s54, v[\r+7]
  .irp i, 1, 2, 3, 4, 5, 6
    v_lshrrev_b32 v114, 2, v114
    v_lshrrev_b32 v115, 2, v115
    v_bfi_b32 v114, s54, v[\r+\i], v114
    v_bfi_b32 v115, s54, v[\r+7+\i], v115
  .endr
    v_lshlrev_b32 v[\r+14], 1, v114
    v_lshlrev_b32 v[\r+15], 1, v115
    v_lshrrev_b32 v114, 1, v114
    v_lshrrev_b32 v115, 1, v115
    v_bfi_b32 v[\r+14], s55, v[\r+14], v114
    v_bfi_b32 v[\r+15], s55, v[\r+15], v115
    s_waitcnt lgkmcnt(0)
  .irp i, 0, 1, 2, 3, 4, 5, 6
    v_bfi_b32 v[\r+\i], s54, v[96+\i], v[\r+\i]
    v_bfi_b32 v[\r+7+\i], s54, v[104+\i], v[\r+7+\i]
  .endr
    v_bfi_b32 v[\r+14], s54, v103, v[\r+14]
    v_bfi_b32 v[\r+15], s54, v111, v[\r+15]
  .irp i, 0, 1, 2, 3, 4, 5, 6, 7, 8, 9, 10, 11, 12, 13, 14, 15
    v_pk_add_u16 v[\r+\i], v[\r+\i], s56
  .endr
Lcs\r\()_%=:
    global_store_dwordx4 %[l16], v[\r:\r+3], s[46:47] sc0 sc1
    global_store_dwordx4 %[l16], v[\r+4:\r+7], s[46:47] offset:1024 sc0 sc1
    global_store_dwordx4 %[l16], v[\r+8:\r+11], s[46:47] offset:2048 sc0 sc1
    global_store_dwordx4 %[l16], v[\r+12:\r+15], s[46:47] offset:3072 sc0 sc1
Lcn\r\()_%=:
    s_cmp_lt_u32 s42, %[nu]
    s_cbranch_scc0 Lcp\r\()_%=
    c_poll
Lcp\r\()_%=:
    s_add_u32 s40, s40, 1
    s_cmp_gt_u32 s41, s40
    s_cselect_b32 s48, s42, 0
    s_cmp_gt_u32 s48, s41
    s_cbranch_scc0 Lcq\r\()_%=
    c_early \r
Lcq\r\()_%=:
    c_stamp s40
.endm
.macro c_rare r, o
Lcd\r\()_%=:
    s_waitcnt vmcnt(0)
    c_flag
    c_take
    s_cmp_gt_u32 s41, s40
    s_cbranch_scc0 Lcg\r\()_%=
    s_cmp_gt_u32 s42, s41
    s_cbranch_scc0 Lce\r\()_%=
    c_early \o
    c_late \o
    s_branch Lce\r\()_%=
Lcg\r\()_%=:
    s_cmp_eq_u32 s40, %[nu]
    s_cbranch_scc1 Lcz_%=
    s_memrealtime s[50:51]
Lch\r\()_%=:
    s_cmp_gt_u32 s42, s40
    s_cbranch_scc1 Lci\r\()_%=
    s_sleep 2
    c_poll
    s_memrealtime s[52:53]
    s_waitcnt vmcnt(0) lgkmcnt(0)
    c_take
    s_sub_u32 s48, s52, s50
    s_cmp_gt_u32 s48, %[ticks]
    s_cbranch_scc0 Lch\r\()_%=
    s_mov_b32 %[dead], 1
    s_branch Lcz_%=
Lci\r\()_%=:
    c_early \r
    c_late \r
    s_cmp_gt_u32 s42, s41
    s_cbranch_scc1 Lcj\r\()_%=
    s_waitcnt vmcnt(0) lgkmcnt(0)
    s_branch Lce\r\()_%=
Lcj\r\()_%=:
    c_early \o
    s_waitcnt vmcnt(%[early]) lgkmcnt(0)
    c_late \o
    s_branch Lce\r\()_%=
Lcy\r\()_%=:
    s_add_u32 s46, s62, s48
    s_addc_u32 s47, s63, 0
  .irp i, 0, 4, 8, 12
    global_store_dwordx4 %[yo], v[\r+\i:\r+\i+3], s[46:47] sc0 sc1
    s_add_u32 s46, s46, 16384
    s_addc_u32 s47, s47, 0
  .endr
    s_branch Lcn\r\()_%=
Lcr\r\()_%=:
    s_mul_i32 s48, s49, %[rstep]
    s_add_u32 s48, s48, %[rly]
    s_add_u32 s64, s60, s48
    s_addc_u32 s65, s61, 0
    global_store_dwordx4 %[l16], v[\r:\r+3], s[64:65] sc0 sc1
    global_store_dwordx4 %[l16], v[\r+4:\r+7], s[64:65] offset:1024 sc0 sc1
    global_store_dwordx4 %[l16], v[\r+8:\r+11], s[64:65] offset:2048 sc0 sc1
    global_store_dwordx2 %[l8], v[\r+12:\r+13], s[64:65] offset:3072 sc0 sc1
    global_store_short %[l2], v[\r+14], s[64:65] offset:-128 sc0 sc1
    s_branch Lcf\r\()_%=
Lcw\r\()_%=:
    v_readlane_b32 s48, %[so], s40
    s_add_u32 s44, s58, s48
    s_addc_u32 s45, s59, 0
    global_load_dwordx2 v[\r+14:\r+15], %[l8], s[44:45] offset:3584 sc0 sc1
    s_cmp_lt_u32 s49, 2
    s_waitcnt vmcnt(0)
    s_cbranch_scc0 Lcs\r\()_%=
    global_store_dwordx2 %[l8], v[\r+14:\r+15], s[64:65] offset:3584 sc0 sc1
    s_branch Lcs\r\()_%=
.endm
    s_mov_b64 s[58:59], %[qb]
    s_mov_b64 s[60:61], %[mb]
    s_mov_b64 s[62:63], %[ob]
    s_mov_b32 s54, 0x60006000
    s_mov_b32 s55, 0x80008000
    s_movk_i32 s57, 0x83f
    s_mov_b32 s40, 0
    s_mov_b32 s41, 0
    v_mov_b32 v116, 0
    c_poll
Lc0_%=:
    c_step 64, 80
    c_step 80, 64
    s_branch Lc0_%=
    c_rare 64, 80
    c_rare 80, 64
Lcz_%=:
    s_waitcnt vmcnt(0) lgkmcnt(0)
.purgem c_load
.purgem c_early
.purgem c_late
.purgem c_flag
.purgem c_take
.purgem c_poll
.purgem c_stamp
.purgem c_step
.purgem c_rare
)"
            : [dead] "+s"(dead)
            : [so] "v"(so), [dst] "v"(dst), [fo] "v"(fo), [fv] "v"(fv), [poff] "v"(poff), [gate] "v"(gate), [thr] "v"(thr),
              [l16] "v"(lane * 16), [l8] "v"(lane * 8), [l2] "v"(lane * 2), [yo] "v"(yo), [rfa] "v"(rfa),
              [tl] "v"(tab + lane * 16), [tab] "s"(tab), [qb] "s"(p.ws[q]), [mb] "s"(myws), [ob] "s"(p.out),
              [nu] "s"(nu), [uy] "s"(same ? 64 : 42), [ur] "s"(same ? 64 : 32),
              [rly] "s"(par + P_RLY + q * PBLOCK + v * PUNIT + 128), [rstep] "n"(16 * PUNIT), [early] "n"(EARLY), [ticks] "n"(DEADLINE_TICKS)
            : "memory", "vcc", "scc", "s40", "s41", "s42", "s44", "s45", "s46", "s47", "s48", "s49", "s50", "s51", "s52",
              "s53", "s54", "s55", "s56", "s57", "s58", "s59", "s60", "s61", "s62", "s63", "s64", "s65", CLOBBER_V);
        if (dead && !same && lane == nu - 1) *(float*)(p.out + dst) = __builtin_nanf("");
        STAMP_AT(11);
        return;
    }

    for (int share = 0; share < (idx >= 24 ? 2 : 1); ++share) {
        const int gb = share ? 44 + ((b >> 2) & 3) + ((b >> 4) & 1) * 16 : (b >> 2) + (b >= 176 ? 4 : 0), sub = b & 3;
        const bool isx = gb < 32;
        const __amdgpu_buffer_rsrc_t src =
            __builtin_amdgcn_make_buffer_rsrc((void*)(isx ? p.x : p.w), 0, IN_BYTES, RSRC_W3);
        const int r = lane & 31, h = lane >> 5;
        const unsigned voff = isx ? ((gb * 128 + r) * KS + sub * 128 + h * 8) * 4
                                  : ((sub * 128 + h * 8) * NN + (gb - 32) * 128 + r * 4) * 4;
        const unsigned dl = isx ? ((lane & 1) ? (gb & 3) * 2 + 1 - ((gb >> 2) & 1) : gb >> 2)
                                : (lane & 3) * 2 + ((gb >> 4) & 1);
        gu32* const ef = (gu32*)(p.ws[dl & (NR - 1)] + OFF_EF + (rank * 64 + gb) * 256 + sub * 4);
        const bool efl = lane < (isx ? 2 : 4);
        const int mine = !isx ? (((gb >> 4) & 1) == rj ? 8 + (gb & 15) : -1)
                              : gb >> 2 == rank ? 4 + (gb & 3) : (gb & 3) == ri && ((gb >> 2) & 1) != rj ? gb >> 3 : -1;
        f32x4 v0[8];
        u32x4 f[4];
        if (share == 0)
            for (int i = !isx && (gb & 15) % 3 ? -DELAY3 : 0; i < wave; ++i) __builtin_amdgcn_s_sleep(DELAY1);
        for (int m = wave; m < 8; m += 4) {
            conv_load(v0, src, isx, voff, m);
            if (share == 0 && m < 4) GET_CALL();
            const unsigned par = OFF_DATA + (call & 1) * P_BYTES, k = 4 * m + sub;
            conv_half(v0, isx, f);
            pack_store(v0, f, buf, par + P_EXP + gb * PBLOCK + k * PUNIT, (lds16*)red + wave * 16384, lane);
            if (mine >= 0) {
#pragma unroll
                for (int e = 0; e < 4; ++e)
                    __builtin_amdgcn_raw_buffer_store_b128(f[e], buf, par + P_STG + (rank * NSLOT + mine) * BLOCK + k * UNIT +
                                                           e * 1024 + lane * 16, 0, AUX_SYS);
            }
            asm volatile("s_waitcnt vmcnt(0)" ::: "memory");
            if (efl && share == 0 && dl != rank && !(isx && lane == 0)) post(ef + 4 * m, call * 64 + 32);
            if (m < 4) { STAMP_AT(1); __builtin_amdgcn_s_sleep(DELAY2); }
        }
        unsigned n = 0;
        if (lane == 0) n = __hip_atomic_fetch_add(cnt + 2, 1u, __ATOMIC_RELAXED, __HIP_MEMORY_SCOPE_WORKGROUP);
        if ((__builtin_amdgcn_readfirstlane(n) & 3) == 3 && efl) post(ef + 32, call * 64 + 32);
        if (share == 0 && idx >= 24) __syncthreads();
        STAMP_AT(12);
    }

    call = __builtin_amdgcn_readfirstlane(call);
    const unsigned fb = call * 64;
    const unsigned par = OFF_DATA + (call & 1) * P_BYTES;
    const int rh = (xcd & 1) * 4 + (idx & 3);
    const int njob = idx < 4 ? 2 : 1;
    const u32x4 bufw = {(unsigned)(size_t)myws, (unsigned)((size_t)myws >> 32) & 0xffff, WS_USED, RSRC_W3};
    unsigned dead = 0;
    for (int job = 0; job < njob; ++job) {
        const bool own = job == 1 || (idx >= 12 && idx < 24);
        const int cb = idx < 24 && job == 0 ? (xcd >> 1) * 3 + (idx >> 2) % 3 : 12 + (xcd >> 1);
        const int a = rh >> 1;
        const int s = lane >> 3, jj = lane & 7;
        const int gx = own ? rank * 4 + a : (2 * a + 1 - rj) * 4 + ri, gw = 32 + rj * 16 + cb;
        const int sx = own ? 4 + a : a;
        const bool isw = jj >= 1 && jj <= 4;
        unsigned poff;
        if (s == rank) poff = OFF_EF + (rank * 64 + (isw ? gw : gx)) * 256 + (32 + wave) * 4;
        else if (!isw) poff = OFF_SF + ((s * NSLOT + sx) * 4 + wave) * 128;
        else poff = OFF_SF + ((s * NSLOT + 8 + cb) * 4 + wave) * 128 + (((s ^ rank) & 1) && cb == 12 + ri ? jj - 1 : 0) * 4;
        unsigned xoff = par + P_STG + (s * NSLOT + sx) * BLOCK + wave * UNIT + (rh & 1) * 2048;
        const unsigned wd = par + P_STG + (s * NSLOT + 8 + cb) * BLOCK + wave * UNIT - xoff;
        const unsigned step = jj == 0 ? 4 * UNIT : 0, xb0 = jj == 0 ? xoff : 0;
        const unsigned lds = (unsigned)(size_t)(__attribute__((address_space(3))) f32x4*)red + wave * 32768 + lane * 16;
        if (job == 1) __builtin_amdgcn_s_barrier();
        asm volatile(R"(
.macro g_x r, o
    buffer_load_dwordx4 v[64+24*\r+4*\o:67+24*\r+4*\o], %[vo], %[buf], s40 offen offset:1024*\o sc0 sc1
.endm
.macro g_w r, o
    buffer_load_dwordx4 v[72+24*\r+4*\o:75+24*\r+4*\o], %[vo], %[buf], s41 offen offset:1024*\o sc0 sc1
.endm
.macro g_m c, m, e
    v_mfma_f32_32x32x16_f16 a[64*\m+16*\e:64*\m+16*\e+15], v[64+24*\c+4*\m:67+24*\c+4*\m], v[72+24*\c+4*\e:75+24*\c+4*\e], a[64*\m+16*\e:64*\m+16*\e+15]
.endm
.macro g_poll r
  .if (\r & 3) == 0
    buffer_load_dword v[60+\r/4], %[poff], %[buf], 0 offen sc0 sc1
  .endif
.endm
.macro g_take p
    v_max_u32 v62, v62, v[60+\p]
    s_nop 1
    v_min_u32_dpp v63, v62, v62 quad_perm:[1,0,3,2] row_mask:0xf bank_mask:0xf
    v_cmp_gt_u32 vcc, %[fin], v62
    s_nop 0
    v_min_u32_dpp v63, v63, v63 quad_perm:[2,3,0,1] row_mask:0xf bank_mask:0xf
    v_cndmask_b32 %[poff], v58, %[poff], vcc
    s_nop 0
    v_min_u32_dpp v63, v63, v63 row_half_mirror row_mask:0xf bank_mask:0xf
    v_sub_u32 v63, v63, %[base] clamp
    v_lshrrev_b32 v63, 2, v63
    v_mad_u32_u24 v59, v63, %[step], %[xb0]
.endm
.macro g_sel
    v_cmp_gt_u32 vcc, v59, %[xoff]
    s_ff1_i32_b64 s42, vcc
    v_readlane_b32 s45, %[xoff], s42
    v_readlane_b32 s46, %[wd], s42
    v_cmp_eq_u32 vcc, s42, %[lane]
    s_mov_b32 s40, s45
    s_add_u32 s41, s45, s46
    v_cndmask_b32 v63, 0, %[step], vcc
    v_add_u32 %[xoff], %[xoff], v63
.endm
.macro g_wait r, c, np
    s_and_b32 s45, s43, 0xff-(1<<\c)
    s_bcnt1_i32_b32 s45, s45
  .irp n, 6, 5, 4, 3, 2, 1
    s_waitcnt vmcnt(6*\n+\np)
    s_cmp_eq_u32 s45, \n
    s_cbranch_scc1 Lw\r\()_%=
  .endr
    s_waitcnt vmcnt(\np)
Lw\r\()_%=:
.endm
.macro g_stamp r, k
)"
R"(
.endm
.macro g_fast r, c, np
Lf\r\()_%=:
    s_waitcnt vmcnt(36+\np)
    g_m \c, 0, 0
    g_poll \r
    g_x \r, 0
  .if (\c & 3) == 0
    v_max_u32 v62, v62, v[60+\c/4]
  .endif
    s_bitset1_b32 s43, \r
    g_m \c, 0, 1
    g_x \r, 1
  .if (\c & 3) == 0
    v_min_u32_dpp v63, v62, v62 quad_perm:[1,0,3,2] row_mask:0xf bank_mask:0xf
    v_cmp_gt_u32 vcc, %[fin], v62
  .endif
    s_bitset0_b32 s43, \c
    g_m \c, 0, 2
    g_w \r, 0
  .if (\c & 3) == 0
    v_min_u32_dpp v63, v63, v63 quad_perm:[2,3,0,1] row_mask:0xf bank_mask:0xf
    v_cndmask_b32 %[poff], v58, %[poff], vcc
  .endif
    s_sub_u32 s44, s44, 1
    g_m \c, 0, 3
    g_w \r, 1
  .if (\c & 3) == 0
    v_min_u32_dpp v63, v63, v63 row_half_mirror row_mask:0xf bank_mask:0xf
    v_sub_u32 v63, v63, %[base] clamp
    v_lshrrev_b32 v63, 2, v63
    v_mad_u32_u24 v59, v63, %[step], %[xb0]
  .endif
    g_m \c, 1, 0
    g_w \r, 2
    v_cmp_gt_u32 vcc, v59, %[xoff]
    s_ff1_i32_b64 s42, vcc
    v_readlane_b32 s45, %[xoff], s42
    v_readlane_b32 s46, %[wd], s42
    g_m \c, 1, 1
    g_w \r, 3
    v_cmp_eq_u32 vcc, s42, %[lane]
    s_mov_b32 s40, s45
    s_add_u32 s41, s45, s46
    g_m \c, 1, 2
    v_cndmask_b32 v63, 0, %[step], vcc
    v_add_u32 %[xoff], %[xoff], v63
    g_stamp \r, f
    s_ashr_i32 s45, s42, 31
    s_or_b32 s45, s45, s43
    s_cmp_eq_u32 s45, 0xff-(1<<\c)
    g_m \c, 1, 3
    s_cbranch_scc0 Lg\c\()_%=
.endm
.macro g_slow r, c, np
Lg\r\()_%=:
  .if (\c & 3) == 0
    g_wait \r, \c, \np
    g_take \c/4
    s_cmp_ge_i32 s42, 0
    s_cbranch_scc1 Lh\r\()_%=
    g_sel
Lh\r\()_%=:
    s_bitcmp1_b32 s43, \c
    s_cbranch_scc0 Ln\r\()_%=
  .else
    s_bitcmp1_b32 s43, \c
    s_cbranch_scc0 Ln\r\()_%=
    g_wait \r, \c, \np
  .endif
    s_cmp_ge_i32 s42, 0
    s_cbranch_scc1 Lf\r\()_%=
    g_poll \r
  .irp m, 0, 1
  .irp e, 0, 1, 2, 3
    g_m \c, \m, \e
  .endr
  .endr
    s_bitset0_b32 s43, \c
    s_branch Le\r\()_%=
Ln\r\()_%=:
    g_poll \r
    s_cmp_ge_i32 s42, 0
    s_cbranch_scc0 Le\r\()_%=
    g_x \r, 0
    g_x \r, 1
    g_w \r, 0
    g_w \r, 1
    g_w \r, 2
    g_w \r, 3
    s_bitset1_b32 s43, \r
    s_sub_u32 s44, s44, 1
    g_stamp \r, g
    g_sel
Le\r\()_%=:
    s_cmp_eq_u32 s43, 0
    s_cbranch_scc1 Lb_%=
.endm
    s_nop 4
    g_poll 0
    v_mov_b32 v61, 0
    v_mov_b32 v62, 0
    v_mov_b32 v58, %[oob]
    s_mov_b32 s43, 0
    s_mov_b32 s44, 64
  .set g_i, 0
  .rept 128
    v_accvgpr_write_b32 a[g_i], 0
  .set g_i, g_i+1
  .endr
Lb_%=:
    s_memrealtime s[48:49]
    s_waitcnt vmcnt(0)
    s_cmp_eq_u32 s44, 0
    s_cbranch_scc1 Ld_%=
    v_max_u32 v60, v60, v61
Lp_%=:
    v_max_u32 v60, %[dead], v60
    g_take 0
    g_sel
    s_cmp_ge_i32 s42, 0
    s_cbranch_scc1 Lg0_%=
    s_sleep %[slp]
    g_poll 0
    s_memrealtime s[50:51]
    s_waitcnt vmcnt(0) lgkmcnt(0)
    s_sub_u32 s45, s50, s48
    s_cmp_gt_u32 s45, %[ticks]
    s_cselect_b32 %[dead], %[last], %[dead]
    s_branch Lp_%=
    g_fast 0, 1, 1
    g_fast 1, 2, 2
    g_fast 2, 3, 2
    g_fast 3, 4, 1
    g_fast 4, 5, 1
    g_fast 5, 6, 2
    g_fast 6, 7, 2
    g_fast 7, 0, 1
    s_branch Lf0_%=
    g_slow 0, 1, 1
    g_slow 1, 2, 2
    g_slow 2, 3, 2
    g_slow 3, 4, 1
    g_slow 4, 5, 1
    g_slow 5, 6, 2
    g_slow 6, 7, 2
    g_slow 7, 0, 1
    s_branch Lg0_%=
Ld_%=:
    s_nop 7
  .set g_i, 0
  .rept 32
    ds_write_b128 %[lds], a[64*(g_i/16)+16*(g_i&3)+(g_i&12):64*(g_i/16)+16*(g_i&3)+(g_i&12)+3] offset:1024*g_i
  .set g_i, g_i+1
  .endr
    s_waitcnt lgkmcnt(0)
.purgem g_x
.purgem g_w
.purgem g_m
.purgem g_poll
.purgem g_take
.purgem g_sel
.purgem g_wait
.purgem g_stamp
.purgem g_fast
.purgem g_slow
)"
            : [poff] "+v"(poff), [xoff] "+v"(xoff), [dead] "+s"(dead)
            : [vo] "v"(lane * 16), [wd] "v"(wd), [step] "v"(step), [xb0] "v"(xb0), [lane] "v"(lane), [lds] "v"(lds),
              [buf] "s"(bufw), [fin] "s"(fb + wave + 29), [base] "s"(fb + wave - 3), [last] "s"(fb + 32),
              [oob] "n"(WS_USED), [slp] "n"(POLL_SLEEP), [ticks] "n"(DEADLINE_TICKS)
            : "memory", "vcc", "scc", "s40", "s41", "s42", "s43", "s44", "s45", "s46", "s48", "s49", "s50", "s51",
              CLOBBER_V, CLOBBER_A);
        if (dead) cnt[3] = 1;
        STAMP_AT(13 + job * 12);
        asm volatile("s_waitcnt lgkmcnt(0)\n\ts_barrier" ::: "memory");
        STAMP_AT(10 + job * 12);
        const bool bad = cnt[3] != 0;
        const __amdgpu_buffer_rsrc_t od = own ? __builtin_amdgcn_make_buffer_rsrc((void*)p.out, 0, IN_BYTES, RSRC_W3) : buf;
        const unsigned pitch = own ? NN * 4 : 1024, hstep = own ? 8 * NN * 4 : UNIT;
        const unsigned ybase = par + P_Y + (rh * 16 + cb) * 32768 + wave * 2 * UNIT;
        const unsigned ooff = own ? ((rh * 64 + wave * 16 + (lane >> 5) * 4) * NN + rj * 2048 + cb * 128) * 4 + (lane & 31) * 16
                                  : ybase + lane * 16;
#pragma unroll
        for (int h = 0; h < 2; ++h) {
            f32x4 y[4];
#pragma unroll
            for (int e = 0; e < 4; ++e) {
                const int i = ((wave * 2 + h) * 4 + e) * 64 + lane;
                y[e] = (red[i] + red[i + 32 * 64]) + (red[i + 2 * 32 * 64] + red[i + 3 * 32 * 64]);
            }
#pragma unroll
            for (int t = 0; t < 4; ++t) {
                f32x4 o;
#pragma unroll
                for (int e = 0; e < 4; ++e) {
                    const float z = fminf(fmaxf(y[e][t], -60.f), 60.f);
                    o[e] = bad ? __builtin_nanf("") : y[e][t] * __builtin_amdgcn_rcpf(1.f + __expf(-z));
                }
                const u32x4 ou = __builtin_bit_cast(u32x4, o);
                if (t < 3 || own) __builtin_amdgcn_raw_buffer_store_b128(ou, od, ooff + t * pitch + h * hstep, 0, AUX_SYS);
                else {
                    __builtin_amdgcn_raw_buffer_store_b64(u32x2{ou.x, ou.y}, od, ybase + h * UNIT + 3072 + lane * 8, 0, AUX_SYS);
                    __builtin_amdgcn_raw_buffer_store_b64(u32x2{ou.z, ou.w}, od, ybase + h * UNIT + 3584 + lane * 8, 0, AUX_SYS);
                }
            }
        }
        if (!own) {
            asm volatile("s_waitcnt vmcnt(0)" ::: "memory");
            if (lane == 0)
                post((gu32*)(p.ws[(2 * a + 1 - rj) & (NR - 1)] + OFF_YF + ((rank * 32 + (rh & 1) * 16 + cb) * 4 + wave) * 128), fb + 32);
        }
        STAMP_AT(11 + job * 12);
    }
}

extern "C" void kernel_launch(int rank, int nranks, ncclComm_t comm,
                              hipStream_t stream,
                              void* const* d_in, const int* in_sizes, int n_in,
                              void* d_out, int out_size,
                              const int* in_dtypes, int out_dtype,
                              void* const* d_out_all,
                              void* const* d_ws_all, size_t ws_bytes) {
    if (nranks != NR || (unsigned)rank >= (unsigned)NR || ws_bytes < WS_USED || n_in != 2 ||
        in_sizes[0] != 4096 * 512 || in_sizes[1] != 4096 * 512 || out_size != 4096 * 512) {
        fprintf(stderr, "unexpected problem shape\n");
        return;
    }
    Params p;
    p.x = (const float*)d_in[0];
    p.w = (const float*)d_in[1];
    for (int r = 0; r < NR; ++r) p.ws[r] = (char*)d_ws_all[r];
    p.out = (char*)d_out;
    p.rank = rank;
    p.links = 0;
    for (int q = 0, lo = 1, hi = 4; q < NR; ++q)
        if (q != rank) p.links |= (unsigned)(rank ^ q) << 4 * (((GROUP >> q) ^ (GROUP >> rank)) & 1 ? hi++ : lo++);
    fused_gemm_silu<<<NWG, 256, 0, stream>>>(p);
    HIP_CHECK(hipGetLastError());
}
```

```cpp
#include <hip/hip_runtime.h>
#include <rccl/rccl.h>
#include <cstddef>
#include <cstdio>

#define HIP_CHECK(expr) do { hipError_t _e = (expr); if (_e != hipSuccess) { \
    fprintf(stderr, "HIP error %s at %s:%d\n", hipGetErrorString(_e), __FILE__, __LINE__); \
    return; } } while (0)

typedef unsigned u32x4 __attribute__((ext_vector_type(4)));
typedef unsigned u32x2 __attribute__((ext_vector_type(2)));
typedef unsigned short u16x2 __attribute__((ext_vector_type(2)));
typedef float f32x4 __attribute__((ext_vector_type(4)));
typedef float f32x16 __attribute__((ext_vector_type(16)));
typedef _Float16 f16x8 __attribute__((ext_vector_type(8)));
typedef __attribute__((address_space(1))) unsigned gu32;
typedef volatile __attribute__((address_space(3))) unsigned short lds16;
typedef unsigned long long u64;

#define POLL_SLEEP 4
#define EARLY 2
#define POLLWIN 3
#define DIRECT 2
#define MSTAG 80
#define MBASE 2100
#define BOOST 0
#define G1 14
#define ROT 0
#define DELAY0 0
#define DELAYS 40
#define DELAYT 0
#define DELAY3 4
#define DELAY1 120
#define DELAY2 40

static_assert((G1 == 15 || G1 == 14) && DIRECT % 2 == 0 && EARLY < 5, "pairs of units");
constexpr int NR = 8;
constexpr int NN = 4096;
constexpr int KS = 512;
constexpr unsigned UNIT = 4096;
constexpr unsigned BLOCK = 32 * UNIT;
constexpr int NSLOT = 24;
constexpr unsigned IN_BYTES = 4096u * 512u * 4u;
constexpr int NWG = 256;
constexpr unsigned OFF_SEQ = 0;
constexpr unsigned OFF_EF = 64u << 10;
constexpr unsigned OFF_SF = 384u << 10;
constexpr unsigned OFF_RF = 224u << 10;
constexpr unsigned OFF_MC = 236u << 10;
constexpr unsigned OFF_NONE = 240u << 10;
constexpr unsigned OFF_PACE = 244u << 10;
constexpr unsigned OFF_ROLE = 208u << 10;
constexpr unsigned OFF_YF = 256u << 10;
constexpr unsigned OFF_DBG = 512u << 10;
constexpr unsigned OFF_DATA = 1u << 20;
constexpr unsigned PUNIT = 128 + UNIT;
constexpr unsigned PBLOCK = 32 * PUNIT;
constexpr unsigned P_EXP = 0;
constexpr unsigned P_RLY = 64 * PBLOCK;
constexpr unsigned P_STG = P_RLY + NR * PBLOCK;
constexpr unsigned P_OW = P_STG + NR * NSLOT * BLOCK;
constexpr unsigned P_Y = P_OW + 16 * BLOCK;
constexpr unsigned P_M = P_Y + 128 * 8 * PUNIT;
constexpr unsigned P_BYTES = P_M + 128 * 8 * UNIT;
constexpr unsigned WS_USED = OFF_DATA + 2 * P_BYTES;
constexpr u64 DEADLINE_TICKS = 50000000ull;
constexpr int AUX_SYS = 17;
#define GROUP_ 0xb2
constexpr unsigned GROUP = GROUP_;
constexpr int RSRC_W3 = 0x00020000;

struct Params {
    const float* x;
    const float* w;
    char* mine;
    u64 rank_links;
    char* out;
    char* ws[NR];
};

#define STAMP_AT(idx) do {} while (0)
#define STAMP_T0() do {} while (0)

__device__ __forceinline__ u32x4 pack8(float a0, float a1, float a2, float a3,
                                       float a4, float a5, float a6, float a7) {
    f16x8 h = {(_Float16)a0, (_Float16)a1, (_Float16)a2, (_Float16)a3,
               (_Float16)a4, (_Float16)a5, (_Float16)a6, (_Float16)a7};
    return __builtin_bit_cast(u32x4, h);
}
__device__ __forceinline__ unsigned poll(gu32* a) {
    return __hip_atomic_load(a, __ATOMIC_RELAXED, __HIP_MEMORY_SCOPE_SYSTEM);
}
__device__ __forceinline__ void post(gu32* a, unsigned v) {
    __hip_atomic_store(a, v, __ATOMIC_RELAXED, __HIP_MEMORY_SCOPE_SYSTEM);
}

__device__ __forceinline__ char* ws_of_lane() {
    unsigned lo = 0, hi = 0;
    asm volatile(R"(
    s_load_dwordx16 s[40:55], %2, %3
    s_waitcnt lgkmcnt(0)
  .irp r, 0, 1, 2, 3, 4, 5, 6, 7
    v_writelane_b32 %0, s[40+2*\r], \r
    v_writelane_b32 %1, s[41+2*\r], \r
  .endr
)"
        : "+v"(lo), "+v"(hi) : "s"(__builtin_amdgcn_kernarg_segment_ptr()), "n"(offsetof(Params, ws))
        : "s40", "s41", "s42", "s43", "s44", "s45", "s46", "s47", "s48", "s49", "s50", "s51", "s52", "s53", "s54", "s55");
    return (char*)((u64)hi << 32 | lo);
}
__device__ __forceinline__ char* ws_of(char* wsl, unsigned r) {
    return (char*)((u64)(unsigned)__builtin_amdgcn_readlane((u64)wsl >> 32, r & (NR - 1)) << 32 |
                   (unsigned)__builtin_amdgcn_readlane((unsigned)(u64)wsl, r & (NR - 1)));
}

__device__ __forceinline__ void conv_load(f32x4 (&v)[8], u64 x, u64 w, int gb, int sub, int m, int lane) {
    const unsigned isw = gb >> 5;
    const __amdgpu_buffer_rsrc_t src = __builtin_amdgcn_make_buffer_rsrc((void*)(isw ? w : x), 0, IN_BYTES, RSRC_W3);
    const unsigned voff = ((lane & 31) << (11 - 7 * isw)) + ((lane >> 5) << (5 + 12 * isw));
    const unsigned base = ((gb & 31) << (18 - 9 * isw)) + ((sub * 8 + m) << (6 + 12 * isw));
    const unsigned odd = 16u << 10 * isw, two = 65536u >> isw;
#pragma unroll
    for (int t = 0; t < 8; ++t)
        v[t] = __builtin_bit_cast(f32x4, __builtin_amdgcn_raw_buffer_load_b128(src, voff, base + (t >> 1) * two + (t & 1) * odd, 0));
}
__device__ __forceinline__ int slot_of(int v, int q, int t) {
    return ((q ^ t) & 1) && (v & 7) == (q >> 1) ? 20 + (t >> 1) : (v & 7) < 4 ? v & 7 : 3 * (v & 7) - 4;
}
__device__ __forceinline__ int block_of(int slot, int t) {
    return slot < 4 ? (2 * slot + 1 - (t & 1)) * 4 + (t >> 1) : 24 + (t & 1) * 16 + slot;
}
__device__ __forceinline__ void conv_half(const f32x4 (&v)[8], bool isx, u32x4 (&f)[4]) {
    if (isx) {
#pragma unroll
        for (int e = 0; e < 4; ++e) {
            const f32x4 a = v[2 * e], b = v[2 * e + 1];
            f[e] = pack8(a.x, a.y, a.z, a.w, b.x, b.y, b.z, b.w);
        }
    } else {
#pragma unroll
        for (int e = 0; e < 4; ++e)
            f[e] = pack8(v[0][e], v[1][e], v[2][e], v[3][e], v[4][e], v[5][e], v[6][e], v[7][e]);
    }
}
__device__ __forceinline__ unsigned bfi(unsigned m, unsigned a, unsigned b) { return (a & m) | (b & ~m); }

template <bool Y, int N>
__device__ __forceinline__ u64 pack_store(const f32x4 (&v)[N], const u32x4 (&f)[4], __amdgpu_buffer_rsrc_t dst,
                                          unsigned off, unsigned loff, lds16* lst, int lane) {
    constexpr unsigned HI = Y ? 0x78000000u : 0x60006000u, W = Y ? 4 : 2, CAP = Y ? 2 : 41;
    float mx = 0.f;
#pragma unroll
    for (int t = 0; t < N; ++t) {
        asm("v_max3_f32 %0, abs(%1), abs(%2), %0\n\tv_max3_f32 %0, abs(%3), abs(%4), %0"
            : "+v"(mx) : "v"(v[t].x), "v"(v[t].y), "v"(v[t].z), "v"(v[t].w));
    }
    lst[lane] = Y ? 0x400 : 0;
    unsigned c = Y ? __builtin_bit_cast(unsigned, mx) >> 16 : __builtin_bit_cast(unsigned short, (_Float16)mx);
    c = max(c, (unsigned)__builtin_amdgcn_update_dpp(0, c, 0xb1, 0xf, 0xf, true));
    c = max(c, (unsigned)__builtin_amdgcn_update_dpp(0, c, 0x4e, 0xf, 0xf, true));
    c = max(c, (unsigned)__builtin_amdgcn_update_dpp(0, c, 0x141, 0xf, 0xf, true));
    c = max(c, (unsigned)__builtin_amdgcn_update_dpp(0, c, 0x140, 0xf, 0xf, true));
    const unsigned top = max(max((unsigned)__builtin_amdgcn_readlane(c, 0), (unsigned)__builtin_amdgcn_readlane(c, 16)),
                             max((unsigned)__builtin_amdgcn_readlane(c, 32), (unsigned)__builtin_amdgcn_readlane(c, 48)));
    const unsigned B = max(top & 0x7fff, Y ? 2047u : 8191u) - (Y ? 2047u : 8191u);
    const u16x2 B2 = {(unsigned short)(Y ? 0 : B), (unsigned short)B};
    unsigned d[16], e[2];
#pragma unroll
    for (int i = 0; i < 16; ++i) {
        const unsigned t = f[i >> 2][i & 3];
        const u16x2 r = __builtin_bit_cast(u16x2, t) - B2;
        d[i] = __builtin_bit_cast(unsigned, r);
    }
#pragma unroll
    for (int a = 0; a < 2; ++a) {
        unsigned x = d[7 * a] & HI, g = bfi(Y ? 0x40000000u : 0x40004000u, d[14 + a] >> 1, d[14 + a] << (W - 1));
#pragma unroll
        for (int i = 1; i < 7; ++i) x = bfi(HI, d[7 * a + i], x >> W);
        e[a] = bfi(HI << 1, d[14 + a] << 1, x >> (W - 1));
#pragma unroll
        for (int i = 6; i >= 0; --i) { d[7 * a + i] = bfi(HI, g, d[7 * a + i]); g <<= W; }
    }
    unsigned n = Y || __ballot((((e[0] & ~(e[0] >> 1)) | (e[1] & ~(e[1] >> 1))) & 0x55555555u) != 0) == 0 ? 0 : 64;
    for (;;) {
        const unsigned x = e[0] ? e[0] : e[1];
        const u64 m = __ballot(x != 0);
        const unsigned slot = n + __builtin_amdgcn_mbcnt_hi(m >> 32, __builtin_amdgcn_mbcnt_lo((unsigned)m, 0));
        n += __builtin_popcountll(m);
        if (m == 0 || n > CAP) break;
        if (x) {
            const unsigned bit = __builtin_ctz(x) & (32 - W), y = x & ~(((1u << W) - 1) << bit);
            lst[slot] = Y ? ((x >> bit) & 15) << 11 | (e[0] ? 0 : 512) | (bit & 16) << 4 | lane << 2 | ((bit >> 2) & 3)
                          : ((x >> bit) & 1) << 11 | (e[0] ? 0 : 1024) | (bit & 8) << 6 | lane << 3 | (bit & 6) | bit >> 4;
            if (e[0]) e[0] = y; else e[1] = y;
        }
    }
    const unsigned i2 = Y ? 0 : 2 * (lane & 31), e0 = lst[i2 < n ? i2 : 0], e1 = lst[i2 + 1 < n ? i2 + 1 : 0];
    const unsigned head = n > CAP ? 0x8000 : B;
    if (n > CAP) {
#pragma unroll
        for (int i = 0; i < 14; ++i) d[i] = f[i >> 2][i & 3];
        __builtin_amdgcn_raw_buffer_store_b64(u32x2{f[3][2], f[3][3]}, dst, off + 128 + 3584 + lane * 8, 0, AUX_SYS);
    }
#pragma unroll
    for (int i = 0; i < 3; ++i)
        __builtin_amdgcn_raw_buffer_store_b128(u32x4{d[4 * i], d[4 * i + 1], d[4 * i + 2], d[4 * i + 3]}, dst,
                                               off + 128 + i * 1024 + lane * 16, 0, AUX_SYS);
    __builtin_amdgcn_raw_buffer_store_b64(u32x2{d[12], d[13]}, dst, off + 128 + 3072 + lane * 8, 0, AUX_SYS);
    if (!Y) {
        const unsigned two = e0 | (lane == 20 ? head << 16 | (n ? 0x4000 : 0) : e1 << 12), r8 = (lane & 3) * 8;
        const unsigned next = __builtin_amdgcn_update_dpp(0, two, 0x101, 0xf, 0xf, true);
        if (lane == 19 || (lane < 19 && r8 < 24))
            __builtin_amdgcn_raw_buffer_store_b32(two >> r8 | next << (24 - r8), dst, loff + (lane - (lane >> 2)) * 4, 0, AUX_SYS);
    }
    return head | e0 << 16 | (u64)e1 << 32;
}

#define CLOBBER_V \
    "v58", "v59", "v60", "v61", "v62", "v63", "v64", "v65", "v66", "v67", "v68", "v69", "v70", "v71", "v72", \
    "v73", "v74", "v75", "v76", "v77", "v78", "v79", "v80", "v81", "v82", "v83", "v84", "v85", "v86", "v87", \
    "v88", "v89", "v90", "v91", "v92", "v93", "v94", "v95", "v96", "v97", "v98", "v99", "v100", "v101", "v102", \
    "v103", "v104", "v105", "v106", "v107", "v108", "v109", "v110", "v111", "v112", "v113", "v114", "v115", \
    "v116", "v117", "v118", "v119", "v120", "v121", "v122", "v123", "v124", "v125", "v126", "v127", "v128", \
    "v129", "v130", "v131", "v132", "v133", "v134", "v135", "v136", "v137", "v138", "v139", "v140", "v141", \
    "v142", "v143", "v144", "v145", "v146", "v147", "v148", "v149", "v150", "v151", "v152", "v153", "v154", \
    "v155", "v156", "v157", "v158", "v159", "v160", "v161", "v162", "v163", "v164", "v165", "v166", "v167", \
    "v168", "v169", "v170", "v171", "v172", "v173", "v174", "v175", "v176", "v177", "v178", "v179", "v180", \
    "v181", "v182", "v183", "v184", "v185", "v186", "v187", "v188", "v189", "v190", "v191", "v192", "v193", \
    "v194", "v195", "v196", "v197", "v198", "v199", "v200", "v201", "v202", "v203", "v204", "v205", "v206", \
    "v207", "v208", "v209", "v210", "v211", "v212", "v213", "v214", "v215", "v216", "v217", "v218", "v219", \
    "v220", "v221", "v222", "v223", "v224", "v225", "v226", "v227", "v228", "v229", "v230", "v231", "v232", \
    "v233", "v234", "v235", "v236", "v237", "v238", "v239", "v240", "v241", "v242", "v243", "v244", "v245", \
    "v246", "v247", "v248", "v249", "v250", "v251", "v252", "v253", "v254", "v255"
#define CLOBBER_A \
    "a0", "a1", "a2", "a3", "a4", "a5", "a6", "a7", "a8", "a9", "a10", "a11", "a12", "a13", "a14", "a15", "a16", \
    "a17", "a18", "a19", "a20", "a21", "a22", "a23", "a24", "a25", "a26", "a27", "a28", "a29", "a30", "a31", \
    "a32", "a33", "a34", "a35", "a36", "a37", "a38", "a39", "a40", "a41", "a42", "a43", "a44", "a45", "a46", \
    "a47", "a48", "a49", "a50", "a51", "a52", "a53", "a54", "a55", "a56", "a57", "a58", "a59", "a60", "a61", \
    "a62", "a63", "a64", "a65", "a66", "a67", "a68", "a69", "a70", "a71", "a72", "a73", "a74", "a75", "a76", \
    "a77", "a78", "a79", "a80", "a81", "a82", "a83", "a84", "a85", "a86", "a87", "a88", "a89", "a90", "a91", \
    "a92", "a93", "a94", "a95", "a96", "a97", "a98", "a99", "a100", "a101", "a102", "a103", "a104", "a105", \
    "a106", "a107", "a108", "a109", "a110", "a111", "a112", "a113", "a114", "a115", "a116", "a117", "a118", \
    "a119", "a120", "a121", "a122", "a123", "a124", "a125", "a126", "a127"

__global__ void __launch_bounds__(256, 1) fused_gemm_silu(Params p) {
    __shared__ f32x4 red[4 * 32 * 64 + 33];
    const int lane = threadIdx.x & 63;
    const int wave = __builtin_amdgcn_readfirstlane(threadIdx.x >> 6);
    const int b = blockIdx.x;
    const unsigned t0 = __builtin_amdgcn_s_memrealtime();
    if (threadIdx.x == 0) red[4 * 32 * 64] = f32x4{0.f, 0.f, 0.f, 0.f};
    u64 px = (u64)p.x, pw = (u64)p.w, pm = (u64)p.mine, rl = p.rank_links;
    asm volatile("s_waitcnt lgkmcnt(0)\n\ts_barrier" : "+s"(px), "+s"(pw), "+s"(pm), "+s"(rl) :: "memory");
    const int rank = rl & (NR - 1), ri = rank >> 1, rj = rank & 1;
    const unsigned links = rl >> 32;
    char* const myws = (char*)pm;
    unsigned* const cnt = (unsigned*)&red[4 * 32 * 64];
    const int xcd = b & 7, idx = b >> 3;

    const bool copier = idx >= 28;

    gu32* const seq = (gu32*)(myws + OFF_SEQ + (b * 4 + wave) * 64);
    unsigned last = 0;
    asm("" : "+v"(last));
    last = poll(seq + last);

#define GET_CALL() const unsigned call = __builtin_amdgcn_readfirstlane(last) + 1u
    const __amdgpu_buffer_rsrc_t buf = __builtin_amdgcn_make_buffer_rsrc((void*)myws, 0, WS_USED, RSRC_W3);

    if (copier) {
        unsigned role = ((links >> 4 * (__builtin_amdgcn_s_getreg(20 | 3 << 11) & 7)) & 7) * 16 + (idx & 3) * 4 + wave;
        char* const wsl = ws_of_lane();
        STAMP_T0();
        if (role < 16) __builtin_amdgcn_s_sleep(127);
        for (unsigned dead = __builtin_amdgcn_readfirstlane(role < 16);;) {
        asm volatile("" : "+s"(dead));
        if (dead) {
            GET_CALL();
            if (lane == 0) post(seq, call);
            gu32* const taken = (gu32*)(myws + OFF_ROLE);
            const u64 m0 = __ballot(lane >= 16 && poll(taken + lane * 32) < call);
            const u64 m1 = __ballot(poll(taken + (64 + lane) * 32) < call);
            if ((m0 | m1) == 0) return;
            role = m0 ? __builtin_ctzll(m0) : 64 + __builtin_ctzll(m1);
        }
        const int link = role >> 4, v = role & 15;
        const unsigned q = (unsigned)(rank ^ link) & (NR - 1);
        const bool same = (link & 1) == 0;
        const int sub = v & 3, a = v >> 2;
        auto k1 = [&](int u) { return 8 * (u >> 2) + 4 * (u & 1) + (u & 2) + (v >> 3); };
        const int slot1 = slot_of(v, q, rank), gb1 = block_of(slot1, rank);
        const int sp = 2 * a + 1 - rj, cq = 12 + (q >> 1);
        const int uw = G1 + (same ? 24 - DIRECT : a == ri ? 8 : 8 + DIRECT), ux = uw + 8, uy = uw + 16, nu = same ? uy : uy + 16;
        auto ycol = [&](int n) { return n < 8 ? 3 * (n & 3) + (n >> 2) : n < 12 ? 12 + ((n + (q >> 1)) & 3) : 3 * n - 34; };
        unsigned poff = OFF_NONE, gate = 0;
        bool used = true;
        const unsigned efw = OFF_EF + (q * 64 + 32 + rj * 16) * 256 + (32 + sub) * 4;
        const unsigned yf = OFF_YF + (q * 8 + (v >> 1)) * 256;
        if (lane < 16) {
            const int u = G1 == 15 ? lane - (lane > 12) : lane - 2 * (lane > 11);
            gate = u & ~1;
            poff = OFF_EF + (q * 64 + gb1) * 256 + (G1 == 15 ? (lane == 12 ? 38 + (v >> 3) : k1(u))
                                                             : (lane >> 1 == 5 ? 16 + 2 * lane + (v >> 3) : k1(u))) * 4;
        } else if (lane == 16) { poff = efw + (3 * a + 1) * 256; gate = G1; }
        else if (lane == 17) { poff = efw + (12 + a) * 256; gate = G1 + 8; }
        else if (lane == 18 && same) { poff = OFF_RF + (q * 8 + sp) * 128 + (sub & 1) * 4; gate = G1 + 16; }
        else if (lane == 19) { poff = efw + (3 * a + 2) * 256; gate = uw; }
        else if (lane == 20) { poff = OFF_EF + (q * 64 + rank * 4 + a) * 256 + (32 + sub) * 4; gate = ux; }
        else if (lane == 21 && !same) { poff = OFF_MC; gate = uy; }
        else if (lane >= 32 && lane < 48 && !same) {
            poff = yf + ycol(lane - 32) * 16; gate = uy + lane - 32;
        } else used = false;
        if (lane == 63) gate = nu;
        char* const rfa = wsl + OFF_RF + (rank * 8 + q) * 128 + (v >> 3) * 4;
        const unsigned yo = (lane >> 5) * 4 * NN * 4 + (lane & 31) * 16;
        const unsigned tab = (unsigned)(size_t)(__attribute__((address_space(3))) f32x4*)red + wave * 32768;
#pragma unroll
        for (int i = 0; i < 5; ++i) red[wave * 2048 + i * 64 + lane] = f32x4{0.f, 0.f, 0.f, 0.f};

        unsigned so, dst, fo, fv;
        {
            const int u = lane;
            unsigned slot, src = q, k = 4 * ((u - (u < (same ? uw : G1 + 8) ? G1 : uw)) & 7) + sub;
            if (u < G1) { so = P_EXP + gb1 * PBLOCK; slot = slot1; k = k1(u); }
            else if (u >= ux) { so = P_EXP + (rank * 4 + a) * PBLOCK; slot = 4 + a; }
            else if (u >= G1 + 16 && u < uw) { src = sp; slot = 8 + cq; so = P_RLY + sp * PBLOCK; }
            else {
                slot = u < G1 + 8 ? 9 + 3 * a : u >= uw ? 10 + 3 * a : 20 + a;
                so = P_EXP + (24 + rj * 16 + slot) * PBLOCK;
            }
            so += k * PUNIT + 128 + ((k >> 2) & 1);
            dst = P_STG + (src * NSLOT + slot) * BLOCK + k * UNIT;
            fo = OFF_SF + ((src * NSLOT + slot) * 4 + (k & 3)) * 128;
            fv = u == 13 || u == (G1 == 15 ? 14 : 11) || u == uw - 1 ? 32 : k + 1;
            if (!same && u >= G1 + 8 && u < uw) { fo += 4; fv = u == uw - 1 ? 32 : 0; }
            if (u >= uy) {
                const int n = ycol((u - uy) & 15);
                so = P_Y + (((ri * 2 + (v >> 3)) * 16 + n) * 8 + (v & 7)) * PUNIT + 129;
                dst = (((q >> 1) * 128 + v * 8) * NN + (q & 1) * 2048 + n * 128) * 4;
                fo = u < nu ? yf + n * 16 + 4 + (v & 1) * 6 : OFF_ROLE + role * 128;
                fv = P_M + ((((q >> 1) * 2 + (v >> 3)) * 16 + n) * 8 + (v & 7)) * UNIT;
            }
        }
        const int li = lane & 31;
        const int skip = !same && a == ri ? DIRECT : 0, done = lane + 1 + (lane >= G1 + 7 ? skip : 0);
        const unsigned inc = lane >= uy ? 0 : lane == uy - 1 ? 65 - done : lane == G1 + 7 ? 1 + skip : 1;
        unsigned lim = 16 * done + BOOST - 1024, unit = !used ? 0 : lane == 21 ? 512 : 64;
        unsigned lk = ((lane & 32) * 2 + (li < 20 ? 3 * li & ~3 : li == 31 ? 60 : 0)) | (li < 20 ? (24 * li & 31) << 8 : 0) |
                            (lane >= 32 && lane < 63 ? 12 << 13 : 0) | (lane == 63 ? 0 : 2 + 2 * (lane & 1)) << 18 | inc << 24;
        asm volatile("" : "+v"(so), "+v"(dst), "+v"(fo), "+v"(fv), "+v"(poff), "+v"(gate), "+v"(lk), "+v"(lim), "+v"(unit));
        GET_CALL();
        if (lane == 0) post(seq, call);
        const unsigned par = OFF_DATA + (call & 1) * P_BYTES;
        const unsigned thr = lane == 63 ? ~0u : call * unit + (unit & 64) / 2;
        so += par;
        dst += lane < uy ? par : 0;
        fv = lane == 63 ? call : fv + (lane < uy ? call * 64 : par);
        lim += call * 1024;
        asm volatile(R"(
.macro c_load r, i
  .if \i < 3
    global_load_dwordx4 v[\r+4*\i:\r+4*\i+3], %[l16], s[44:45] offset:1024*\i sc0 sc1
  .elseif \i == 3
    global_load_dwordx2 v[\r+12:\r+13], v58, s[44:45] offset:3072 sc0 sc1
  .else
    s_bitcmp1_b32 s43, 0
    s_cbranch_scc1 Lcc\@
    global_load_dword v[\r+14], v59, s[44:45] offset:-128 sc0 sc1
Lcc\@:
  .endif
.endm
.macro c_early r
    v_readlane_b32 s43, %[so], s41
    s_add_u32 s41, s41, 1
    s_andn2_b32 s48, s43, 1
    s_add_u32 s44, s58, s48
    s_addc_u32 s45, s59, 0
  .irp i, 0, 1, 2, 3, 4
   .if \i < %[early]
    c_load \r, \i
   .endif
  .endr
.endm
.macro c_late r
  .irp i, 0, 1, 2, 3, 4
   .if \i >= %[early]
    c_load \r, \i
   .endif
  .endr
.endm
.macro c_sum r, skip
    s_cmp_gt_u32 s41, %[uy]
    s_cbranch_scc0 \skip
    s_sub_u32 s48, s41, 1
    v_readlane_b32 s49, %[fo], s48
    v_readlane_b32 s48, %[fv], s48
    s_add_u32 s46, s60, s48
    s_addc_u32 s47, s61, 0
  .irp i, 0, 1, 2, 3
    global_load_dwordx4 v[56+\r+4*\i:59+\r+4*\i], %[l16], s[46:47] offset:1024*\i sc0 sc1
  .endr
    s_add_u32 s46, s60, s49
    s_addc_u32 s47, s61, 0
    global_load_ushort v[\r+14], v63, s[46:47] sc0 sc1
.endm
.macro c_flag n
    s_mov_b32 s68, 0
Lc6\@:
    s_cmp_lt_u32 s74, \n
    s_cbranch_scc0 Lc7\@
    s_lshl_b64 exec, 1, s74
    global_store_dword %[fo], %[fv], s[60:61] sc0 sc1
    global_atomic_add v153, %[pace], v152, s[60:61] sc0 sc1
    s_add_u32 s74, s74, 1
    s_add_u32 s68, s68, 2
    s_cmp_eq_u32 s74, %[ur]
    s_cbranch_scc0 Lc6\@
    s_mov_b32 exec_lo, %[twins]
    global_store_dword %[rfa], %[thr], off sc0 sc1
    s_branch Lc6\@
Lc7\@:
    s_mov_b64 exec, -1
.endm
.macro c_take
    v_cmp_lt_u32 vcc, v112, %[thr]
    s_ff1_i32_b64 s48, vcc
    v_readlane_b32 s42, %[gate], s48
.endm
.macro c_poll
    global_load_dword v112, %[poff], s[60:61] sc0 sc1
.endm
.macro c_stamp n
)"
R"(
.endm
.macro c_near skip
    s_add_u32 s48, s41, %[win]
    s_min_u32 s48, s48, %[nu]
    s_cmp_lt_u32 s42, s48
    s_cbranch_scc0 \skip
    c_poll
    s_add_u32 s68, s68, 1
.endm
.macro c_step r, o
    s_sub_u32 s48, s41, s40
    s_cmp_eq_u32 s48, 2
    s_cbranch_scc0 Lcd\r\()_%=
    s_cmp_gt_u32 s41, %[uy]
    s_cbranch_scc1 Lck\r\()_%=
    s_cmp_lg_u32 s73, 0
    s_cbranch_scc1 Lc3\r\()_%=
    s_waitcnt vmcnt(%[early])
    c_late \o
Lca\r\()_%=:
    s_sub_u32 s48, s40, s73
    c_flag s48
Lcl\r\()_%=:
    c_take
Lce\r\()_%=:
    v_readlane_b32 s49, %[so], s40
    v_mov_b32_dpp v113, v[\r+14] quad_perm:[1,2,3,3] row_mask:0xf bank_mask:0xf
    s_cmp_lt_i32 s40, %[ur]
    s_cbranch_scc1 Lcr\r\()_%=
    s_bitcmp1_b32 s49, 0
    s_cbranch_scc1 Lcv\r\()_%=
Lcf\r\()_%=:
    v_alignbit_b32 v62, v113, v[\r+14], v60
    v_alignbit_b32 v[\r+14], v113, v[\r+14], v60
    s_nop 1
    v_permlane32_swap_b32 v[\r+14], v62
    s_nop 0
Lcx\r\()_%=:
    v_readlane_b32 s56, v[\r+14], 31
    v_lshrrev_b32 v[\r+14], v61, v[\r+14]
    s_and_b32 s48, s56, 0x4000
    s_lshr_b32 s56, s56, 16
    v_and_b32 v113, 0x7ff, v[\r+14]
    v_lshl_or_b32 v[\r+14], v[\r+14], 2, s48
Lcx2\r\()_%=:
    s_bitcmp1_b32 s56, 15
    s_cbranch_scc1 Lcw\r\()_%=
    s_mul_i32 s56, s56, s69
    v_mad_u32_u24 v113, v113, s66, v117
    v_and_b32 v114, s54, v[\r]
    ds_write_b16 v113, v[\r+14]
    ds_read_b128 v[96:99], %[tl]
    ds_read_b128 v[100:103], %[tl] offset:1024
    ds_read_b128 v[104:107], %[tl] offset:2048
    ds_read_b128 v[108:111], %[tl] offset:3072
    ds_write_b16 v113, v116
    v_and_b32 v115, s54, v[\r+7]
  .irp i, 1, 2, 3, 4, 5, 6
    v_lshrrev_b32 v114, s66, v114
    v_lshrrev_b32 v115, s66, v115
    v_bfi_b32 v114, s54, v[\r+\i], v114
    v_bfi_b32 v115, s54, v[\r+7+\i], v115
  .endr
    v_lshlrev_b32 v[\r+14], 1, v114
    v_lshlrev_b32 v[\r+15], 1, v115
    v_lshrrev_b32 v114, s67, v114
    v_lshrrev_b32 v115, s67, v115
    v_bfi_b32 v[\r+14], s55, v[\r+14], v114
    v_bfi_b32 v[\r+15], s55, v[\r+15], v115
    s_waitcnt lgkmcnt(0)
  .irp i, 0, 1, 2, 3, 4, 5, 6
    v_bfi_b32 v[\r+\i], s54, v[96+\i], v[\r+\i]
    v_bfi_b32 v[\r+7+\i], s54, v[104+\i], v[\r+7+\i]
  .endr
    v_bfi_b32 v[\r+14], s54, v103, v[\r+14]
    v_bfi_b32 v[\r+15], s54, v111, v[\r+15]
  .irp i, 0, 1, 2, 3, 4, 5, 6, 7, 8, 9, 10, 11, 12, 13, 14, 15
    v_pk_add_u16 v[\r+\i], v[\r+\i], s56
  .endr
Lcs\r\()_%=:
    v_readlane_b32 s48, %[dst], s40
    s_cmp_ge_u32 s40, %[uy]
    s_cbranch_scc1 Lcy\r\()_%=
    s_add_u32 s46, s60, s48
    s_addc_u32 s47, s61, 0
    global_store_dwordx4 %[l16], v[\r:\r+3], s[46:47] sc0 sc1
    global_store_dwordx4 %[l16], v[\r+4:\r+7], s[46:47] offset:1024 sc0 sc1
    global_store_dwordx4 %[l16], v[\r+8:\r+11], s[46:47] offset:2048 sc0 sc1
    global_store_dwordx4 %[l16], v[\r+12:\r+15], s[46:47] offset:3072 sc0 sc1
    c_near Lcp\r\()_%=
Lcp\r\()_%=:
    s_add_u32 s40, s40, 1
    s_cmp_eq_u32 s40, %[uy]
    s_cbranch_scc1 Lcm\r\()_%=
Lco\r\()_%=:
    s_cmp_gt_u32 s41, s40
    s_cselect_b32 s48, s42, 0
    s_cmp_gt_u32 s48, s41
    s_cbranch_scc0 Lcq\r\()_%=
    c_early \r
    v_cmp_ne_u32 vcc, 0, v153
    s_flbit_i32_b64 s48, vcc
    v_cmp_ge_u32 vcc, v153, %[lim]
    s_not_b32 s48, s48
    s_bitcmp1_b64 vcc, s48
    s_cselect_b32 s73, 1, 0
    c_sum \r, Lc1\r\()_%=
Lc2\r\()_%=:
    c_late \r
Lcq\r\()_%=:
    c_stamp s40
.endm
.macro c_rare r, o
Lc1\r\()_%=:
    s_cmp_lg_u32 s73, 0
    s_cbranch_scc0 Lcq\r\()_%=
    s_bitcmp0_b32 s43, 0
    s_addc_u32 s68, s68, 8
    s_branch Lc2\r\()_%=
Lc3\r\()_%=:
  .irp n, 14, 13, 12, 11, 10, 9
    s_waitcnt vmcnt(\n)
    s_cmp_ge_u32 s68, \n
    s_cbranch_scc1 Lca\r\()_%=
  .endr
    s_waitcnt vmcnt(8)
    s_branch Lca\r\()_%=
Lck\r\()_%=:
    s_waitcnt vmcnt(13)
    s_cmp_gt_u32 s40, %[uy]
    s_cbranch_scc1 Lcl\r\()_%=
    s_waitcnt vmcnt(9)
    s_mov_b32 s73, 0
    s_branch Lca\r\()_%=
Lcd\r\()_%=:
    s_waitcnt vmcnt(4)
    s_cmp_gt_u32 s40, %[uy]
    s_cbranch_scc1 Lcb\r\()_%=
    s_waitcnt vmcnt(0)
    c_flag s40
Lcb\r\()_%=:
    c_take
    s_mov_b32 s68, 0
    s_cmp_gt_u32 s41, s40
    s_cbranch_scc0 Lcg\r\()_%=
    s_cmp_gt_u32 s42, s41
    s_cbranch_scc0 Lce\r\()_%=
    c_early \o
    c_late \o
    c_sum \o, Lce\r\()_%=
    s_branch Lce\r\()_%=
Lcg\r\()_%=:
    s_cmp_eq_u32 s40, %[nu]
    s_cbranch_scc1 Lcz_%=
    s_memrealtime s[50:51]
Lch\r\()_%=:
    s_cmp_gt_u32 s42, s40
    s_cbranch_scc1 Lci\r\()_%=
    s_sleep 2
    c_poll
    s_memrealtime s[52:53]
    s_waitcnt vmcnt(0) lgkmcnt(0)
    c_take
    s_sub_u32 s48, s52, s50
    s_cmp_gt_u32 s48, %[ticks]
    s_cbranch_scc0 Lch\r\()_%=
    s_mov_b32 %[dead], 1
    s_branch Lcz_%=
Lci\r\()_%=:
    c_early \r
    c_late \r
    c_sum \r, Lct\r\()_%=
Lct\r\()_%=:
    s_cmp_gt_u32 s42, s41
    s_cbranch_scc1 Lcj\r\()_%=
    s_waitcnt vmcnt(0) lgkmcnt(0)
    s_branch Lce\r\()_%=
Lcj\r\()_%=:
    c_early \o
    c_sum \o, Lcu\r\()_%=
    c_late \o
    s_waitcnt vmcnt(9) lgkmcnt(0)
    s_branch Lce\r\()_%=
Lcu\r\()_%=:
    s_waitcnt vmcnt(%[early]) lgkmcnt(0)
    c_late \o
    s_branch Lce\r\()_%=
Lcy\r\()_%=:
    s_add_u32 s46, s62, s48
    s_addc_u32 s47, s63, 0
    c_near Lcn\r\()_%=
Lcn\r\()_%=:
  .irp i, 0, 2, 4, 6, 8, 10, 12, 14
    v_pk_add_f32 v[\r+\i:\r+\i+1], v[\r+\i:\r+\i+1], v[56+\r+\i:57+\r+\i]
  .endr
  .irp i, 0, 2, 4, 6, 8, 10, 12, 14
    v_pk_mul_f32 v[96+\i:97+\i], v[\r+\i:\r+\i+1], s[70:71]
  .endr
  .irp i, 0, 1, 2, 3, 4, 5, 6, 7, 8, 9, 10, 11, 12, 13, 14, 15
    v_med3_f32 v[96+\i], v[96+\i], -s72, s72
  .endr
  .irp i, 0, 1, 2, 3, 4, 5, 6, 7, 8, 9, 10, 11, 12, 13, 14, 15
    v_exp_f32 v[96+\i], v[96+\i]
  .endr
  .irp i, 0, 2, 4, 6, 8, 10, 12, 14
    v_pk_add_f32 v[96+\i:97+\i], v[96+\i:97+\i], v[118:119]
  .endr
  .irp i, 0, 4, 8, 12
    v_rcp_f32 v[96+\i], v[96+\i]
    v_rcp_f32 v[97+\i], v[97+\i]
    v_rcp_f32 v[98+\i], v[98+\i]
    v_rcp_f32 v[99+\i], v[99+\i]
    v_pk_mul_f32 v[\r+\i:\r+\i+1], v[\r+\i:\r+\i+1], v[96+\i:97+\i]
    v_pk_mul_f32 v[\r+\i+2:\r+\i+3], v[\r+\i+2:\r+\i+3], v[98+\i:99+\i]
    global_store_dwordx4 %[yo], v[\r+\i:\r+\i+3], s[46:47] sc0 sc1
    s_add_u32 s46, s46, 16384
    s_addc_u32 s47, s47, 0
  .endr
    s_branch Lcp\r\()_%=
Lcm\r\()_%=:
    s_mov_b32 s54, 0x78000000
    s_brev_b32 s55, 1
    s_mov_b32 s66, 4
    s_mov_b32 s67, 3
    s_mov_b32 s69, 0x10000
    v_add_u32 v117, 2, %[tab]
    s_branch Lco\r\()_%=
Lcr\r\()_%=:
    s_andn2_b32 s48, s49, 1
    s_add_u32 s48, s48, %[rly]
    s_add_u32 s64, s60, s48
    s_addc_u32 s65, s61, 0
    global_store_dwordx4 %[l16], v[\r:\r+3], s[64:65] sc0 sc1
    global_store_dwordx4 %[l16], v[\r+4:\r+7], s[64:65] offset:1024 sc0 sc1
    global_store_dwordx4 %[l16], v[\r+8:\r+11], s[64:65] offset:2048 sc0 sc1
    global_store_dwordx2 v58, v[\r+12:\r+13], s[64:65] offset:3072 sc0 sc1
    s_bitcmp1_b32 s49, 0
    s_cbranch_scc1 Lcv\r\()_%=
    global_store_dword v59, v[\r+14], s[64:65] offset:-128 sc0 sc1
    s_branch Lcf\r\()_%=
Lcv\r\()_%=:
    s_cmp_ge_u32 s40, %[uy]
    s_cbranch_scc1 Lcv2\r\()_%=
    v_mov_b32 v[\r+14], v62
    s_branch Lcx\r\()_%=
Lcv2\r\()_%=:
    v_readlane_b32 s56, v[\r+14], 63
    v_writelane_b32 v[\r+14], s57, 63
    v_and_b32 v113, 0x7ff, v[\r+14]
    s_branch Lcx2\r\()_%=
Lcw\r\()_%=:
    s_andn2_b32 s48, s49, 1
    s_add_u32 s44, s58, s48
    s_addc_u32 s45, s59, 0
    global_load_dwordx2 v[\r+14:\r+15], v58, s[44:45] offset:3584 sc0 sc1
    s_cmp_lt_i32 s40, %[ur]
    s_waitcnt vmcnt(0)
    s_cbranch_scc0 Lcs\r\()_%=
    global_store_dwordx2 v58, v[\r+14:\r+15], s[64:65] offset:3584 sc0 sc1
    s_branch Lcs\r\()_%=
.endm
    s_mov_b64 s[58:59], %[qb]
    s_mov_b64 s[60:61], %[mb]
    s_lshl_b64 exec, 1, 63
    global_atomic_umax v154, %[fo], %[fv], s[60:61] sc0 sc1
    s_mov_b64 exec, -1
    c_poll
    s_mov_b64 s[62:63], %[ob]
    s_mov_b32 s54, 0x60006000
    s_mov_b32 s55, 0x80008000
    s_movk_i32 s57, 0x400
    s_mov_b32 s66, 2
    s_mov_b32 s67, 1
    v_lshrrev_b32 v58, 1, %[l16]
    v_and_b32 v59, 0xff, %[lk]
    v_bfe_u32 v60, %[lk], 8, 5
    v_bfe_u32 v61, %[lk], 13, 5
    v_bfe_u32 v63, %[lk], 18, 6
    v_lshrrev_b32 v152, 24, %[lk]
    v_mov_b32 v153, 0
    s_mov_b32 s69, 0x10001
    s_mov_b32 s70, 0xbfb8aa3b
    s_mov_b32 s71, s70
    s_mov_b32 s72, 0x42ad1f97
    v_mov_b32 v117, %[tab]
    v_mov_b32 v118, 1.0
    v_mov_b32 v119, 1.0
    s_mov_b32 s40, 0
    s_mov_b32 s74, 0
    s_mov_b32 s41, 0
    v_mov_b32 v116, 0
    s_waitcnt vmcnt(0)
    v_cmp_ge_u32 vcc, v154, %[fv]
    s_lshr_b32 %[dead], vcc_hi, 30
    s_bitcmp1_b32 vcc_hi, 31
    s_cbranch_scc1 Lcz_%=
    s_mov_b32 %[dead], 0
Lc0_%=:
    c_step 64, 80
    c_step 80, 64
    s_branch Lc0_%=
    c_rare 64, 80
    c_rare 80, 64
Lcz_%=:
    s_waitcnt vmcnt(0) lgkmcnt(0)
.purgem c_load
.purgem c_early
.purgem c_late
.purgem c_flag
.purgem c_take
.purgem c_poll
.purgem c_stamp
.purgem c_near
.purgem c_sum
.purgem c_step
.purgem c_rare
)"
            : [dead] "=&s"(dead)
            : [so] "v"(so), [dst] "v"(dst), [fo] "v"(fo), [fv] "v"(fv), [poff] "v"(poff), [gate] "v"(gate), [thr] "v"(thr),
              [l16] "v"(lane * 16), [lk] "v"(lk), [lim] "v"(lim), [pace] "v"(OFF_PACE + link * 128), [twins] "s"((0x55u << rj) & ~(1u << rank)), [yo] "v"(yo), [rfa] "v"(rfa),
              [tl] "v"(tab + lane * 16), [tab] "s"(tab), [qb] "s"(ws_of(wsl, q)), [mb] "s"(myws), [ob] "s"(p.out),
              [nu] "s"(nu), [uy] "s"(same ? 64 : uy), [ur] "s"(slot1 >= 20 ? 16 - 2 * DIRECT : -1),
              [rly] "s"(P_RLY + q * PBLOCK - P_EXP - gb1 * PBLOCK), [early] "n"(EARLY), [win] "n"(POLLWIN), [ticks] "n"(DEADLINE_TICKS)
            : "memory", "vcc", "scc", "s40", "s41", "s42", "s43", "s44", "s45", "s46", "s47", "s48", "s49", "s50", "s51", "s52",
              "s53", "s54", "s55", "s56", "s57", "s58", "s59", "s60", "s61", "s62", "s63", "s64", "s65", "s66", "s67", "s68",
              "s69", "s70", "s71", "s72", "s73", "s74", CLOBBER_V);
        if (dead >= 2) continue;
        if (dead && !same && lane == nu - 1) *(float*)(p.out + dst) = __builtin_nanf("");
        STAMP_AT(11);
        return;
        }
    }

    lds16* const lst = (lds16*)&red[4 * 32 * 64 + 1] + wave * 64;
    const int gb0 = b >> 4 == rank ? 60 - rj * 16 + ((b >> 2) & 3) : (b >> 2) + ((b + 80) >> 8) * 4;
    const bool first = (0xf249f249ffffffffull >> gb0) & 1;
    unsigned turn = (wave & 2) + ((b >> 1) & 1), nap = turn * DELAY1, to = rank;
    int g = gb0, gs = b & 3, gm = wave, sslot = 0;
    if (!first) {
        turn = wave + DELAY3;
        nap = turn * DELAY1;
        if (wave >= 2 * G1 - 28) to = (rank ^ (links >> 4 * ((xcd + ROT) & 7))) & (NR - 1);
        if (to != rank) {
            sslot = slot_of((gb0 & 16) / 4 + (gb0 & 15) / 3, rank, to);
            g = block_of(sslot, to);
            gs = wave | 2;
            gm = 6 + (wave >> 1);
            nap = wave < 2 ? DELAYS + (idx & 7) * DELAYT : 0;
        }
    }
    if (nap)
        while ((unsigned)__builtin_amdgcn_s_memrealtime() - t0 < nap) __builtin_amdgcn_s_sleep(2);
    f32x4 v0[8];
    conv_load(v0, px, pw, g, gs, gm, lane);
    __builtin_amdgcn_sched_barrier(0);
    char* const wsl = ws_of_lane();
    STAMP_AT(28);
    STAMP_T0();
    for (int job = to != rank ? -1 : 0;;) {
        const int sub = b & 3, part = (b >> 2) & 3, k = 16 * (job & 1) + 4 * wave + sub;
        const bool isx = g < 32, late = job >= 2;
        const int other = (g & 3) * 2 + 1 - ((g >> 2) & 1);
        const bool efw = lane < NR && (lane & 1) == ((g >> 4) & 1);
        const bool efl = isx ? lane == g >> 2 || (lane == other && !late) : efw || lane == rank;
        const bool efu = !late && first && lane != rank && (isx ? lane == other : efw && ((g & 15) < 12 || lane >> 1 == part));
        const unsigned efo = OFF_EF + (rank * 64 + g) * 256;
        const int mine = !isx ? (((g >> 4) & 1) == rj ? rank * NSLOT + 8 : NR * NSLOT) + (g & 15)
                              : late ? rank * NSLOT + 4 + part : other == rank ? rank * NSLOT + (g >> 3) : -1;
        u32x4 f[4];
        conv_half(v0, isx, f);
        GET_CALL();
        const unsigned par = OFF_DATA + (call & 1) * P_BYTES;
        if (job < 0) {
            const __amdgpu_buffer_rsrc_t dst = __builtin_amdgcn_make_buffer_rsrc(ws_of(wsl, to), 0, WS_USED, RSRC_W3);
#pragma unroll
            for (int e = 0; e < 4; ++e)
                __builtin_amdgcn_raw_buffer_store_b128(f[e], dst, par + P_STG + (rank * NSLOT + sslot) * BLOCK +
                                                       (26 + wave + (wave & 2)) * UNIT + e * 1024 + lane * 16, 0, AUX_SYS);
            STAMP_AT(29);
            asm volatile("s_waitcnt vmcnt(0)" ::: "memory");
            STAMP_AT(30);
            if (lane == to) post((gu32*)(wsl + efo + (36 + wave) * 4), call * 64 + 32);
            while ((unsigned)__builtin_amdgcn_s_memrealtime() - t0 < turn * DELAY1) __builtin_amdgcn_s_sleep(2);
        } else {
            if (!(isx && late))
                pack_store<false>(v0, f, buf, par + P_EXP + g * PBLOCK + k * PUNIT,
                                  par + P_EXP + g * PBLOCK + (k & ~4) * PUNIT + (k & 4) * 16, lst, lane);
            if (mine >= 0) {
#pragma unroll
                for (int e = 0; e < 4; ++e)
                    __builtin_amdgcn_raw_buffer_store_b128(f[e], buf, par + P_STG + mine * BLOCK + k * UNIT +
                                                           e * 1024 + lane * 16, 0, AUX_SYS);
            }
            asm volatile("s_waitcnt vmcnt(0)" ::: "memory");
            if (efu) post((gu32*)(wsl + efo + k * 4), call * 64 + 32);
            if (job & 1) {
                unsigned n = 0;
                if (lane == 0) n = __hip_atomic_fetch_add(cnt + 2, 1u, __ATOMIC_RELAXED, __HIP_MEMORY_SCOPE_WORKGROUP);
                if ((__builtin_amdgcn_readfirstlane(n) & 3) == 3 && efl) post((gu32*)(wsl + efo + (32 + sub) * 4), call * 64 + 32);
                STAMP_AT(12);
                if (late || idx < 24) break;
                __syncthreads();
            } else { STAMP_AT(1); __builtin_amdgcn_s_sleep(DELAY2); }
        }
        ++job;
        g = job >= 2 ? (b & 16 ? rank * 4 : 44 + rj * 16) + part : gb0;
        conv_load(v0, px, pw, g, sub, wave + 4 * (job & 1), lane);
    }

    GET_CALL();
    if (lane == 0) post(seq, call);
    const unsigned fb = call * 64;
    const unsigned par = OFF_DATA + (call & 1) * P_BYTES;
    const int rh = (xcd & 1) * 4 + (idx & 3);
    const int njob = idx < 4 || idx >= 12 ? 2 : 1;
    const u32x4 bufw = {(unsigned)(size_t)myws, (unsigned)((size_t)myws >> 32) & 0xffff, WS_USED, RSRC_W3};
    unsigned dead = 0;
    for (int job = 0; job < njob; ++job) {
        const bool own = job == 1 && idx < 24, slab = job == 0 && idx >= 12;
        const int cb = idx >= 24 || (idx < 4 && job == 1) ? 12 + (xcd >> 1) : (xcd >> 1) * 3 + (idx >> 2) % 3;
        const int a = rh >> 1, peer = 2 * a + 1 - rj;
        const int s = lane >> 3, jj = lane & 7;
        const int gx = own || slab ? rank * 4 + a : peer * 4 + ri, gw = 32 + (rj ^ slab) * 16 + cb;
        const int sx = own || slab ? 4 + a : a;
        const bool isw = jj == 1 || jj == 2, on = own || (slab ? s == rank : s != peer);
        unsigned poff;
        if (!on) poff = WS_USED;
        else if (s == rank) poff = OFF_EF + (rank * 64 + (isw ? gw : gx)) * 256 + (32 + wave) * 4;
        else if (!isw) poff = OFF_SF + ((s * NSLOT + sx) * 4 + wave) * 128;
        else poff = OFF_SF + ((s * NSLOT + 8 + cb) * 4 + wave) * 128 + (((s ^ rank) & 1) && cb >= 12 && cb != 12 + ri ? jj - 1 : 0) * 4;
        unsigned xoff = par + P_STG + (s * NSLOT + sx) * BLOCK + wave * UNIT + (rh & 1) * 2048;
        const unsigned wd = par + P_STG + ((slab ? NR * NSLOT : s * NSLOT + 8) + cb) * BLOCK + wave * UNIT - xoff;
        const unsigned step = jj == 0 && on ? 4 * UNIT : 0, xb0 = jj == 0 ? xoff : 0;
        const unsigned lds = (unsigned)(size_t)(__attribute__((address_space(3))) f32x4*)red + wave * 32768 + lane * 16;
        if (slab && idx < 24)
            while ((unsigned)__builtin_amdgcn_s_memrealtime() - t0 < MBASE + (idx - 12) * MSTAG) __builtin_amdgcn_s_sleep(8);
        if (job == 1) __builtin_amdgcn_s_barrier();
        asm volatile(R"(
.macro g_x r, o
    buffer_load_dwordx4 v[64+24*\r+4*\o:67+24*\r+4*\o], %[vo], %[buf], s40 offen offset:1024*\o sc0 sc1
.endm
.macro g_w r, o
    buffer_load_dwordx4 v[72+24*\r+4*\o:75+24*\r+4*\o], %[vo], %[buf], s41 offen offset:1024*\o sc0 sc1
.endm
.macro g_m c, m, e
    v_mfma_f32_32x32x16_f16 a[64*\m+16*\e:64*\m+16*\e+15], v[64+24*\c+4*\m:67+24*\c+4*\m], v[72+24*\c+4*\e:75+24*\c+4*\e], a[64*\m+16*\e:64*\m+16*\e+15]
.endm
.macro g_poll r
  .if (\r & 3) == 0
    buffer_load_dword v[60+\r/4], %[poff], %[buf], 0 offen sc0 sc1
  .endif
.endm
.macro g_take p
    v_max_u32 v62, v62, v[60+\p]
    s_nop 1
    v_min_u32_dpp v63, v62, v62 quad_perm:[1,0,3,2] row_mask:0xf bank_mask:0xf
    v_cmp_gt_u32 vcc, %[fin], v62
    s_nop 0
    v_min_u32_dpp v63, v63, v63 quad_perm:[2,3,0,1] row_mask:0xf bank_mask:0xf
    v_cndmask_b32 %[poff], v58, %[poff], vcc
    s_nop 0
    v_min_u32_dpp v63, v63, v63 row_half_mirror row_mask:0xf bank_mask:0xf
    v_sub_u32 v63, v63, %[base] clamp
    v_lshrrev_b32 v63, 2, v63
    v_mad_u32_u24 v59, v63, %[step], %[xb0]
.endm
.macro g_sel
    v_cmp_gt_u32 vcc, v59, %[xoff]
    s_ff1_i32_b64 s42, vcc
    v_readlane_b32 s45, %[xoff], s42
    v_readlane_b32 s46, %[wd], s42
    v_cmp_eq_u32 vcc, s42, %[lane]
    s_mov_b32 s40, s45
    s_add_u32 s41, s45, s46
    v_cndmask_b32 v63, 0, %[step], vcc
    v_add_u32 %[xoff], %[xoff], v63
.endm
.macro g_wait r, c, np
    s_and_b32 s45, s43, 0xff-(1<<\c)
    s_bcnt1_i32_b32 s45, s45
  .irp n, 6, 5, 4, 3, 2, 1
    s_waitcnt vmcnt(6*\n+\np)
    s_cmp_eq_u32 s45, \n
    s_cbranch_scc1 Lw\r\()_%=
  .endr
    s_waitcnt vmcnt(\np)
Lw\r\()_%=:
.endm
.macro g_stamp r, k
)"
R"(
.endm
.macro g_fast r, c, np
Lf\r\()_%=:
    s_waitcnt vmcnt(36+\np)
    g_m \c, 0, 0
    g_poll \r
    g_x \r, 0
  .if (\c & 3) == 0
    v_max_u32 v62, v62, v[60+\c/4]
  .endif
    s_bitset1_b32 s43, \r
    g_m \c, 0, 1
    g_x \r, 1
  .if (\c & 3) == 0
    v_min_u32_dpp v63, v62, v62 quad_perm:[1,0,3,2] row_mask:0xf bank_mask:0xf
    v_cmp_gt_u32 vcc, %[fin], v62
  .endif
    s_bitset0_b32 s43, \c
    g_m \c, 0, 2
    g_w \r, 0
  .if (\c & 3) == 0
    v_min_u32_dpp v63, v63, v63 quad_perm:[2,3,0,1] row_mask:0xf bank_mask:0xf
    v_cndmask_b32 %[poff], v58, %[poff], vcc
  .endif
    s_sub_u32 s44, s44, 1
    g_m \c, 0, 3
    g_w \r, 1
  .if (\c & 3) == 0
    v_min_u32_dpp v63, v63, v63 row_half_mirror row_mask:0xf bank_mask:0xf
    v_sub_u32 v63, v63, %[base] clamp
    v_lshrrev_b32 v63, 2, v63
    v_mad_u32_u24 v59, v63, %[step], %[xb0]
  .endif
    g_m \c, 1, 0
    g_w \r, 2
    v_cmp_gt_u32 vcc, v59, %[xoff]
    s_ff1_i32_b64 s42, vcc
    v_readlane_b32 s45, %[xoff], s42
    v_readlane_b32 s46, %[wd], s42
    g_m \c, 1, 1
    g_w \r, 3
    v_cmp_eq_u32 vcc, s42, %[lane]
    s_mov_b32 s40, s45
    s_add_u32 s41, s45, s46
    g_m \c, 1, 2
    v_cndmask_b32 v63, 0, %[step], vcc
    v_add_u32 %[xoff], %[xoff], v63
    g_stamp \r, f
    s_ashr_i32 s45, s42, 31
    s_or_b32 s45, s45, s43
    s_cmp_eq_u32 s45, 0xff-(1<<\c)
    g_m \c, 1, 3
    s_cbranch_scc0 Lg\c\()_%=
.endm
.macro g_slow r, c, np
Lg\r\()_%=:
  .if (\c & 3) == 0
    g_wait \r, \c, \np
    g_take \c/4
    s_cmp_ge_i32 s42, 0
    s_cbranch_scc1 Lh\r\()_%=
    g_sel
Lh\r\()_%=:
    s_bitcmp1_b32 s43, \c
    s_cbranch_scc0 Ln\r\()_%=
  .else
    s_bitcmp1_b32 s43, \c
    s_cbranch_scc0 Ln\r\()_%=
    g_wait \r, \c, \np
  .endif
    s_cmp_ge_i32 s42, 0
    s_cbranch_scc1 Lf\r\()_%=
    g_poll \r
  .irp m, 0, 1
  .irp e, 0, 1, 2, 3
    g_m \c, \m, \e
  .endr
  .endr
    s_bitset0_b32 s43, \c
    s_branch Le\r\()_%=
Ln\r\()_%=:
    g_poll \r
    s_cmp_ge_i32 s42, 0
    s_cbranch_scc0 Le\r\()_%=
    g_x \r, 0
    g_x \r, 1
    g_w \r, 0
    g_w \r, 1
    g_w \r, 2
    g_w \r, 3
    s_bitset1_b32 s43, \r
    s_sub_u32 s44, s44, 1
    g_stamp \r, g
    g_sel
Le\r\()_%=:
    s_cmp_eq_u32 s43, 0
    s_cbranch_scc1 Lb_%=
.endm
    s_nop 4
    g_poll 0
    v_mov_b32 v61, 0
    v_mov_b32 v62, 0
    v_mov_b32 v58, %[oob]
    s_mov_b32 s43, 0
    s_mov_b32 s44, %[nst]
  .set g_i, 0
  .rept 128
    v_accvgpr_write_b32 a[g_i], 0
  .set g_i, g_i+1
  .endr
Lb_%=:
    s_memrealtime s[48:49]
    s_waitcnt vmcnt(0)
    s_cmp_eq_u32 s44, 0
    s_cbranch_scc1 Ld_%=
    v_max_u32 v60, v60, v61
Lp_%=:
    v_max_u32 v60, %[dead], v60
    g_take 0
    g_sel
    s_cmp_ge_i32 s42, 0
    s_cbranch_scc1 Lg0_%=
    s_sleep %[slp]
    g_poll 0
    s_memrealtime s[50:51]
    s_waitcnt vmcnt(0) lgkmcnt(0)
    s_sub_u32 s45, s50, s48
    s_cmp_gt_u32 s45, %[ticks]
    s_cselect_b32 %[dead], %[last], %[dead]
    s_branch Lp_%=
    g_fast 0, 1, 1
    g_fast 1, 2, 2
    g_fast 2, 3, 2
    g_fast 3, 4, 1
    g_fast 4, 5, 1
    g_fast 5, 6, 2
    g_fast 6, 7, 2
    g_fast 7, 0, 1
    s_branch Lf0_%=
    g_slow 0, 1, 1
    g_slow 1, 2, 2
    g_slow 2, 3, 2
    g_slow 3, 4, 1
    g_slow 4, 5, 1
    g_slow 5, 6, 2
    g_slow 6, 7, 2
    g_slow 7, 0, 1
    s_branch Lg0_%=
Ld_%=:
    s_nop 7
  .set g_i, 0
  .rept 32
    ds_write_b128 %[lds], a[64*(g_i/16)+16*(g_i&3)+(g_i&12):64*(g_i/16)+16*(g_i&3)+(g_i&12)+3] offset:1024*g_i
  .set g_i, g_i+1
  .endr
    s_waitcnt lgkmcnt(0)
.purgem g_x
.purgem g_w
.purgem g_m
.purgem g_poll
.purgem g_take
.purgem g_sel
.purgem g_wait
.purgem g_stamp
.purgem g_fast
.purgem g_slow
)"
            : [poff] "+v"(poff), [xoff] "+v"(xoff), [dead] "+s"(dead)
            : [vo] "v"(lane * 16), [wd] "v"(wd), [step] "v"(step), [xb0] "v"(xb0), [lane] "v"(lane), [lds] "v"(lds),
              [buf] "s"(bufw), [fin] "s"(fb + wave + 29), [base] "s"(fb + wave - 3), [last] "s"(fb + 32), [nst] "s"(own ? 64 : slab ? 8 : 56),
              [oob] "n"(WS_USED), [slp] "n"(POLL_SLEEP), [ticks] "n"(DEADLINE_TICKS)
            : "memory", "vcc", "scc", "s40", "s41", "s42", "s43", "s44", "s45", "s46", "s48", "s49", "s50", "s51",
              CLOBBER_V, CLOBBER_A);
        if (dead) cnt[3] = 1;
        STAMP_AT(13 + job * 12);
        asm volatile("s_waitcnt lgkmcnt(0)\n\ts_barrier" ::: "memory");
        STAMP_AT(10 + job * 12);
        const bool bad = cnt[3] != 0;
        const __amdgpu_buffer_rsrc_t od = own ? __builtin_amdgcn_make_buffer_rsrc((void*)p.out, 0, IN_BYTES, RSRC_W3) : buf;
        const unsigned pitch = own ? NN * 4 : 1024, hstep = own ? 8 * NN * 4 : UNIT, yu = (rh * 16 + cb) * 8 + wave * 2;
        const unsigned ooff = own ? ((rh * 64 + wave * 16 + (lane >> 5) * 4) * NN + rj * 2048 + cb * 128) * 4 + (lane & 31) * 16
                                  : par + P_M + yu * UNIT + lane * 16;
        u64 yl[2] = {0, 0};
#pragma unroll
        for (int h = 0; h < 2; ++h) {
            f32x4 y[4];
            u32x4 o[4];
#pragma unroll
            for (int e = 0; e < 4; ++e) {
                const int i = ((wave * 2 + h) * 4 + e) * 64 + lane;
                y[e] = (red[i] + red[i + 32 * 64]) + (red[i + 2 * 32 * 64] + red[i + 3 * 32 * 64]);
            }
            if (bad) y[0].x = __builtin_nanf("");
            if (own) {
#pragma unroll
                for (int e = 0; e < 4; ++e) {
                    f32x4 z = y[e] * -1.44269504f;
#pragma unroll
                    for (int t = 0; t < 4; ++t) z[t] = __builtin_amdgcn_exp2f(__builtin_amdgcn_fmed3f(z[t], -86.5617025f, 86.5617025f));
                    z += 1.f;
#pragma unroll
                    for (int t = 0; t < 4; ++t) z[t] = __builtin_amdgcn_rcpf(z[t]);
                    y[e] *= z;
                }
            }
#pragma unroll
            for (int t = 0; t < 4; ++t) o[t] = __builtin_bit_cast(u32x4, f32x4{y[0][t], y[1][t], y[2][t], y[3][t]});
            if (own || slab) {
#pragma unroll
                for (int t = 0; t < 4; ++t) __builtin_amdgcn_raw_buffer_store_b128(o[t], od, ooff + t * pitch + h * hstep, 0, AUX_SYS);
            } else yl[h] = pack_store<true>(y, o, buf, par + P_Y + (yu + h) * PUNIT, 0, lst, lane);
        }
        if (!own) {
            asm volatile("s_waitcnt vmcnt(0)" ::: "memory");
            if (lane == 0) {
                if (slab) __hip_atomic_fetch_add((gu32*)(myws + OFF_MC), 1u, __ATOMIC_RELAXED, __HIP_MEMORY_SCOPE_SYSTEM);
                else __builtin_amdgcn_raw_buffer_store_b128(
                    u32x4{fb + 32, (unsigned)yl[0], (unsigned)(yl[0] >> 32 | yl[1] << 16), (unsigned)(yl[1] >> 16)},
                    __builtin_amdgcn_make_buffer_rsrc(ws_of(wsl, peer), 0, WS_USED, RSRC_W3),
                    OFF_YF + ((rank * 2 + (rh & 1)) * 4 + wave) * 256 + cb * 16, 0, AUX_SYS);
            }
        }
        STAMP_AT(11 + job * 12);
    }
}

extern "C" void kernel_launch(int rank, int nranks, ncclComm_t comm,
                              hipStream_t stream,
                              void* const* d_in, const int* in_sizes, int n_in,
                              void* d_out, int out_size,
                              const int* in_dtypes, int out_dtype,
                              void* const* d_out_all,
                              void* const* d_ws_all, size_t ws_bytes) {
    if (nranks != NR || (unsigned)rank >= (unsigned)NR || ws_bytes < WS_USED || n_in != 2 ||
        in_sizes[0] != 4096 * 512 || in_sizes[1] != 4096 * 512 || out_size != 4096 * 512) {
        fprintf(stderr, "unexpected problem shape\n");
        return;
    }
    Params p;
    p.x = (const float*)d_in[0];
    p.w = (const float*)d_in[1];
    for (int r = 0; r < NR; ++r) p.ws[r] = (char*)d_ws_all[r];
    p.mine = p.ws[rank];
    p.out = (char*)d_out;

    p.rank_links = rank;
    for (int q = 0, lo = 1, hi = 4; q < NR; ++q)
        if (q != rank) p.rank_links |= (u64)(rank ^ q) << (32 + 4 * (((GROUP >> q) ^ (GROUP >> rank)) & 1 ? hi++ : lo++));
    fused_gemm_silu<<<NWG, 256, 0, stream>>>(p);
    HIP_CHECK(hipGetLastError());
}
```

```cpp
#include <hip/hip_runtime.h>
#include <rccl/rccl.h>
#include <cstddef>
#include <cstdio>

#define HIP_CHECK(expr) do { hipError_t _e = (expr); if (_e != hipSuccess) { \
    fprintf(stderr, "HIP error %s at %s:%d\n", hipGetErrorString(_e), __FILE__, __LINE__); \
    return; } } while (0)

#define STR_(x) #x
#define STR(x) STR_(x)
typedef unsigned u32x4 __attribute__((ext_vector_type(4)));
typedef unsigned u32x2 __attribute__((ext_vector_type(2)));
typedef unsigned short u16x2 __attribute__((ext_vector_type(2)));
typedef float f32x4 __attribute__((ext_vector_type(4)));
typedef float f32x16 __attribute__((ext_vector_type(16)));
typedef _Float16 f16x8 __attribute__((ext_vector_type(8)));
typedef __attribute__((address_space(1))) unsigned gu32;
typedef volatile __attribute__((address_space(3))) unsigned short lds16;
typedef unsigned long long u64;
typedef u64 u64x4 __attribute__((ext_vector_type(4)));

#define POLL_SLEEP 4
#define TAIL 8
#define PACE 20
#define IDLE 30
#define PSC "sc0 sc1"
#define DSC "sc0 sc1"
#define REP 4
#define FEW 1
#define EARLY 2
#define POLLWIN 3
#define DIRECT 2
#define MSTAG 80
#define MBASE 2100
#define BOOST 0
#define G1 14
#define DELAYS 60
#define DELAY3 4
#define DELAY1 120
#define DELAY2 40

static_assert((G1 == 15 || G1 == 14) && DIRECT % 2 == 0 && EARLY < 5, "pairs of units");
constexpr int NR = 8;
constexpr int NN = 4096;
constexpr int KS = 512;
constexpr unsigned UNIT = 4096;
constexpr unsigned BLOCK = 32 * UNIT;
constexpr int NSLOT = 24;
constexpr unsigned IN_BYTES = 4096u * 512u * 4u;
constexpr int NWG = 256;
constexpr unsigned OFF_SEQ = 0;
constexpr unsigned OFF_EF = 64u << 10;
constexpr unsigned OFF_SF = 1024u << 10;
constexpr unsigned OFF_RF = 224u << 10;
constexpr unsigned OFF_MC = 236u << 10;
constexpr unsigned OFF_NONE = 240u << 10;
constexpr unsigned OFF_PACE = 244u << 10;
constexpr unsigned OFF_ROLE = 208u << 10;
constexpr unsigned OFF_YF = 256u << 10;
constexpr unsigned OFF_DBG = 512u << 10;
constexpr unsigned OFF_DATA = 1536u << 10;
constexpr unsigned PUNIT = 128 + UNIT;
constexpr unsigned PBLOCK = 32 * PUNIT;
constexpr unsigned P_EXP = 0;
constexpr unsigned P_RLY = 64 * PBLOCK;
constexpr unsigned P_STG = P_RLY + NR * PBLOCK;
constexpr unsigned P_OW = P_STG + NR * NSLOT * BLOCK;
constexpr unsigned P_Y = P_OW + 16 * BLOCK;
constexpr unsigned P_M = P_Y + 128 * 8 * PUNIT;
constexpr unsigned P_BYTES = P_M + 128 * 8 * UNIT;
constexpr unsigned WS_USED = OFF_DATA + 2 * P_BYTES;
constexpr u64 DEADLINE_TICKS = 50000000ull;
constexpr int AUX_SYS = 17;
#define GROUP_ 0xb2
constexpr unsigned GROUP = GROUP_;
constexpr int RSRC_W3 = 0x00020000;

struct Params {
    const float* x;
    const float* w;
    char* mine;
    u64 rank_links;
    char* out;
    char* ws[NR];
    unsigned start[NWG];
};

#define STAMP_V(idx) do {} while (0)
#define STAMP_AT(idx) do {} while (0)
#define STAMP_T0() do {} while (0)

__device__ __forceinline__ u32x4 pack8(float a0, float a1, float a2, float a3,
                                       float a4, float a5, float a6, float a7) {
    f16x8 h = {(_Float16)a0, (_Float16)a1, (_Float16)a2, (_Float16)a3,
               (_Float16)a4, (_Float16)a5, (_Float16)a6, (_Float16)a7};
    return __builtin_bit_cast(u32x4, h);
}
__device__ __forceinline__ unsigned sf_of(unsigned src, unsigned slot, unsigned sub) {
    return OFF_SF + ((src * NSLOT + slot) * 4 + sub) * 128 * REP;
}
__device__ __forceinline__ unsigned poll(gu32* a) {
    return __hip_atomic_load(a, __ATOMIC_RELAXED, __HIP_MEMORY_SCOPE_SYSTEM);
}
__device__ __forceinline__ void post(gu32* a, unsigned v) {
    __hip_atomic_store(a, v, __ATOMIC_RELAXED, __HIP_MEMORY_SCOPE_SYSTEM);
}

__device__ __forceinline__ char* ws_of_lane() {
    unsigned lo = 0, hi = 0;
    asm volatile(R"(
    s_load_dwordx16 s[40:55], %2, %3
    s_waitcnt lgkmcnt(0)
  .irp r, 0, 1, 2, 3, 4, 5, 6, 7
    v_writelane_b32 %0, s[40+2*\r], \r
    v_writelane_b32 %1, s[41+2*\r], \r
  .endr
)"
        : "+v"(lo), "+v"(hi) : "s"(__builtin_amdgcn_kernarg_segment_ptr()), "n"(offsetof(Params, ws))
        : "s40", "s41", "s42", "s43", "s44", "s45", "s46", "s47", "s48", "s49", "s50", "s51", "s52", "s53", "s54", "s55");
    return (char*)((u64)hi << 32 | lo);
}
__device__ __forceinline__ char* ws_of(char* wsl, unsigned r) {
    return (char*)((u64)(unsigned)__builtin_amdgcn_readlane((u64)wsl >> 32, r & (NR - 1)) << 32 |
                   (unsigned)__builtin_amdgcn_readlane((unsigned)(u64)wsl, r & (NR - 1)));
}

__device__ __forceinline__ void conv_load(f32x4 (&v)[8], u64 x, u64 w, unsigned gb, unsigned ks, unsigned vx, unsigned vw) {
    const bool isw = gb & 32;
    const __amdgpu_buffer_rsrc_t src = __builtin_amdgcn_make_buffer_rsrc((void*)(isw ? w : x), 0, IN_BYTES, RSRC_W3);
    const unsigned base = (isw ? ks : gb) << 18 | (isw ? (gb & 31) * 8 : ks) << 6;
    const unsigned odd = isw ? 16384 : 16, two = isw ? 32768 : 65536;
#pragma unroll
    for (int t = 0; t < 8; ++t)
        v[t] = __builtin_bit_cast(f32x4, __builtin_amdgcn_raw_buffer_load_b128(src, isw ? vw : vx, base + (t >> 1) * two + (t & 1) * odd, 0));
}
__host__ __device__ __forceinline__ int slot_of(int v, int q, int t) {
    return ((q ^ t) & 1) && (v & 7) == (q >> 1) ? 20 + (t >> 1) : (v & 7) < 4 ? v & 7 : 3 * (v & 7) - 4;
}
__host__ __device__ __forceinline__ int block_of(int slot, int t) {
    return slot < 4 ? (2 * slot + 1 - (t & 1)) * 4 + (t >> 1) : 24 + (t & 1) * 16 + slot;
}
__device__ __forceinline__ void conv_half(const f32x4 (&v)[8], bool isx, u32x4 (&f)[4]) {
    if (isx) {
#pragma unroll
        for (int e = 0; e < 4; ++e) {
            const f32x4 a = v[2 * e], b = v[2 * e + 1];
            f[e] = pack8(a.x, a.y, a.z, a.w, b.x, b.y, b.z, b.w);
        }
    } else {
#pragma unroll
        for (int e = 0; e < 4; ++e)
            f[e] = pack8(v[0][e], v[1][e], v[2][e], v[3][e], v[4][e], v[5][e], v[6][e], v[7][e]);
    }
}
__device__ __forceinline__ unsigned bfi(unsigned m, unsigned a, unsigned b) { return (a & m) | (b & ~m); }

template <bool Y, int N>
__device__ __forceinline__ u64 pack_store(const f32x4 (&v)[N], const u32x4 (&f)[4], __amdgpu_buffer_rsrc_t dst,
                                          unsigned off, unsigned loff, lds16* lst, int lane) {
    constexpr unsigned HI = Y ? 0x78000000u : 0x60006000u, W = Y ? 4 : 2, CAP = Y ? 2 : 41;
    float mx = 0.f;
#pragma unroll
    for (int t = 0; t < N; t += 4) {
        asm("v_max3_f32 %0, abs(%1), abs(%2), %0\n\tv_max3_f32 %0, abs(%3), abs(%4), %0\n\t"
            "v_max3_f32 %0, abs(%5), abs(%6), %0\n\tv_max3_f32 %0, abs(%7), abs(%8), %0\n\t"
            "v_max3_f32 %0, abs(%9), abs(%10), %0\n\tv_max3_f32 %0, abs(%11), abs(%12), %0\n\t"
            "v_max3_f32 %0, abs(%13), abs(%14), %0\n\tv_max3_f32 %0, abs(%15), abs(%16), %0"
            : "+v"(mx) : "v"(v[t].x), "v"(v[t].y), "v"(v[t].z), "v"(v[t].w), "v"(v[t + 1].x), "v"(v[t + 1].y), "v"(v[t + 1].z),
              "v"(v[t + 1].w), "v"(v[t + 2].x), "v"(v[t + 2].y), "v"(v[t + 2].z), "v"(v[t + 2].w), "v"(v[t + 3].x),
              "v"(v[t + 3].y), "v"(v[t + 3].z), "v"(v[t + 3].w));
    }
    lst[lane] = Y ? 0x400 : 0;
    unsigned c = Y ? __builtin_bit_cast(unsigned, mx) >> 16 : __builtin_bit_cast(unsigned short, (_Float16)mx);
    c = max(c, (unsigned)__builtin_amdgcn_update_dpp(0, c, 0xb1, 0xf, 0xf, true));
    c = max(c, (unsigned)__builtin_amdgcn_update_dpp(0, c, 0x4e, 0xf, 0xf, true));
    c = max(c, (unsigned)__builtin_amdgcn_update_dpp(0, c, 0x141, 0xf, 0xf, true));
    c = max(c, (unsigned)__builtin_amdgcn_update_dpp(0, c, 0x140, 0xf, 0xf, true));
    const unsigned top = max(max((unsigned)__builtin_amdgcn_readlane(c, 0), (unsigned)__builtin_amdgcn_readlane(c, 16)),
                             max((unsigned)__builtin_amdgcn_readlane(c, 32), (unsigned)__builtin_amdgcn_readlane(c, 48)));
    const unsigned B = max(top & 0x7fff, Y ? 2047u : 8191u) - (Y ? 2047u : 8191u);
    const u16x2 B2 = {(unsigned short)(Y ? 0 : B), (unsigned short)B};
    unsigned d[16], e[2];
#pragma unroll
    for (int i = 0; i < 16; ++i) {
        const unsigned t = f[i >> 2][i & 3];
        const u16x2 r = __builtin_bit_cast(u16x2, t) - B2;
        d[i] = __builtin_bit_cast(unsigned, r);
    }
#pragma unroll
    for (int a = 0; a < 2; ++a) {
        unsigned x = d[7 * a] & HI, g = bfi(Y ? 0x40000000u : 0x40004000u, d[14 + a] >> 1, d[14 + a] << (W - 1));
#pragma unroll
        for (int i = 1; i < 7; ++i) x = bfi(HI, d[7 * a + i], x >> W);
        e[a] = bfi(HI << 1, d[14 + a] << 1, x >> (W - 1));
#pragma unroll
        for (int i = 6; i >= 0; --i) { d[7 * a + i] = bfi(HI, g, d[7 * a + i]); g <<= W; }
    }
    auto put = [&]() {
#pragma unroll
        for (int i = 0; i < 3; ++i)
            __builtin_amdgcn_raw_buffer_store_b128(u32x4{d[4 * i], d[4 * i + 1], d[4 * i + 2], d[4 * i + 3]}, dst,
                                                   off + 128 + i * 1024 + lane * 16, 0, AUX_SYS);
        __builtin_amdgcn_raw_buffer_store_b64(u32x2{d[12], d[13]}, dst, off + 128 + 3072 + lane * 8, 0, AUX_SYS);
    };
    put();
    unsigned n = Y || __ballot((((e[0] & ~(e[0] >> 1)) | (e[1] & ~(e[1] >> 1))) & 0x55555555u) != 0) == 0 ? 0 : 64;
    for (;;) {
        const bool two = e[0] == 0;
        const unsigned x = two ? e[1] : e[0];
        const u64 m = __ballot(x != 0);
        const unsigned slot = n + __builtin_amdgcn_mbcnt_hi(m >> 32, __builtin_amdgcn_mbcnt_lo((unsigned)m, 0));
        n += __builtin_popcountll(m);
        if (m == 0 || n > CAP) break;
        const unsigned bit = __builtin_ctz(x | 0x80000000u) & (32 - W), y = x & ~(((1u << W) - 1) << bit);
        lst[x ? slot : 63] = Y ? ((x >> bit) & 15) << 11 | (two ? 512 : 0) | (bit & 16) << 4 | lane << 2 | ((bit >> 2) & 3)
                               : ((x >> bit) & 1) << 11 | (two ? 1024 : 0) | (bit & 8) << 6 | lane << 3 | (bit & 6) | bit >> 4;
        e[1] = two ? y : e[1];
        e[0] = two ? 0 : y;
    }
    const unsigned i2 = Y ? 0 : 2 * (lane & 31), e0 = lst[i2 < n ? i2 : 0], e1 = lst[i2 + 1 < n ? i2 + 1 : 0];
    const unsigned head = n > CAP ? 0x8000 : B;
    if (n > CAP) {
        asm volatile("s_waitcnt vmcnt(0)" ::: "memory");
#pragma unroll
        for (int i = 0; i < 14; ++i) d[i] = f[i >> 2][i & 3];
        put();
        __builtin_amdgcn_raw_buffer_store_b64(u32x2{f[3][2], f[3][3]}, dst, off + 128 + 3584 + lane * 8, 0, AUX_SYS);
    }
    if (!Y) {
        const unsigned two = e0 | (lane == 20 ? head << 16 | (n ? 0x4000 : 0) : e1 << 12), r8 = (lane & 3) * 8;
        const unsigned next = __builtin_amdgcn_update_dpp(0, two, 0x101, 0xf, 0xf, true);
        if (lane == 19 || (lane < 19 && r8 < 24))
            __builtin_amdgcn_raw_buffer_store_b32(two >> r8 | next << (24 - r8), dst, loff + (lane - (lane >> 2)) * 4, 0, AUX_SYS);
    }
    return head | e0 << 16 | (u64)e1 << 32;
}

#define CLOBBER_V \
    "v58", "v59", "v60", "v61", "v62", "v63", "v64", "v65", "v66", "v67", "v68", "v69", "v70", "v71", "v72", \
    "v73", "v74", "v75", "v76", "v77", "v78", "v79", "v80", "v81", "v82", "v83", "v84", "v85", "v86", "v87", \
    "v88", "v89", "v90", "v91", "v92", "v93", "v94", "v95", "v96", "v97", "v98", "v99", "v100", "v101", "v102", \
    "v103", "v104", "v105", "v106", "v107", "v108", "v109", "v110", "v111", "v112", "v113", "v114", "v115", \
    "v116", "v117", "v118", "v119", "v120", "v121", "v122", "v123", "v124", "v125", "v126", "v127", "v128", \
    "v129", "v130", "v131", "v132", "v133", "v134", "v135", "v136", "v137", "v138", "v139", "v140", "v141", \
    "v142", "v143", "v144", "v145", "v146", "v147", "v148", "v149", "v150", "v151", "v152", "v153", "v154", \
    "v155", "v156", "v157", "v158", "v159", "v160", "v161", "v162", "v163", "v164", "v165", "v166", "v167", \
    "v168", "v169", "v170", "v171", "v172", "v173", "v174", "v175", "v176", "v177", "v178", "v179", "v180", \
    "v181", "v182", "v183", "v184", "v185", "v186", "v187", "v188", "v189", "v190", "v191", "v192", "v193", \
    "v194", "v195", "v196", "v197", "v198", "v199", "v200", "v201", "v202", "v203", "v204", "v205", "v206", \
    "v207", "v208", "v209", "v210", "v211", "v212", "v213", "v214", "v215", "v216", "v217", "v218", "v219", \
    "v220", "v221", "v222", "v223", "v224", "v225", "v226", "v227", "v228", "v229", "v230", "v231", "v232", \
    "v233", "v234", "v235", "v236", "v237", "v238", "v239", "v240", "v241", "v242", "v243", "v244", "v245", \
    "v246", "v247", "v248", "v249", "v250", "v251", "v252", "v253", "v254", "v255"
#define CLOBBER_A \
    "a0", "a1", "a2", "a3", "a4", "a5", "a6", "a7", "a8", "a9", "a10", "a11", "a12", "a13", "a14", "a15", "a16", \
    "a17", "a18", "a19", "a20", "a21", "a22", "a23", "a24", "a25", "a26", "a27", "a28", "a29", "a30", "a31", \
    "a32", "a33", "a34", "a35", "a36", "a37", "a38", "a39", "a40", "a41", "a42", "a43", "a44", "a45", "a46", \
    "a47", "a48", "a49", "a50", "a51", "a52", "a53", "a54", "a55", "a56", "a57", "a58", "a59", "a60", "a61", \
    "a62", "a63", "a64", "a65", "a66", "a67", "a68", "a69", "a70", "a71", "a72", "a73", "a74", "a75", "a76", \
    "a77", "a78", "a79", "a80", "a81", "a82", "a83", "a84", "a85", "a86", "a87", "a88", "a89", "a90", "a91", \
    "a92", "a93", "a94", "a95", "a96", "a97", "a98", "a99", "a100", "a101", "a102", "a103", "a104", "a105", \
    "a106", "a107", "a108", "a109", "a110", "a111", "a112", "a113", "a114", "a115", "a116", "a117", "a118", \
    "a119", "a120", "a121", "a122", "a123", "a124", "a125", "a126", "a127"

__global__ void __launch_bounds__(256, 1) fused_gemm_silu(Params p) {
    __shared__ f32x4 red[4 * 32 * 64 + 33];
    const int b = blockIdx.x;
    u64x4 arg;
    u64 now;
    unsigned start, more;
    asm volatile("s_load_dwordx8 %0, %4, 0\n\ts_load_dword %1, %4, %5 offset:%6\n\ts_load_dword %2, %4, 64\n\ts_memrealtime %3"
                 : "=&s"(arg), "=&s"(start), "=&s"(more), "=&s"(now)
                 : "s"(__builtin_amdgcn_kernarg_segment_ptr()), "s"(b * 4), "n"(offsetof(Params, start)));
    const int lane = threadIdx.x & 63;
    const int wave = __builtin_amdgcn_readfirstlane(threadIdx.x >> 6);
    const int xcd = b & 7, idx = b >> 3;
    if (threadIdx.x == 0) red[4 * 32 * 64] = f32x4{0.f, 0.f, 0.f, 0.f};
    const unsigned first = (0x249f249ffffffffull >> ((b >> 2) + ((b + 80) >> 8) * 4)) & 1;
    unsigned turn = (((wave & 2) + ((b >> 1) & 1)) * first + (wave + DELAY3) * (1 - first)) * DELAY1;
    unsigned kso = (b & 3) * 8 + wave, kss = (wave | 2) * 8 + 6 + (wave >> 1), sqo = OFF_SEQ + (b * 4 + wave) * 64;
    unsigned vx = (lane & 31) << 11 | (lane >> 5) << 5, vw = (lane & 31) << 4 | (lane >> 5) << 17;
    asm volatile("" : "+v"(sqo), "+v"(vx), "+v"(vw));
    asm volatile("s_waitcnt lgkmcnt(0)" : "+s"(arg), "+s"(start), "+s"(more), "+s"(now), "+s"(turn), "+s"(kso), "+s"(kss) :: "memory");
    const u64 px = arg[0], pw = arg[1], pm = arg[2], rl = arg[3];
    const unsigned t0 = now;
    const int rank = rl & (NR - 1), ri = rank >> 1, rj = rank & 1;
    const unsigned links = rl >> 32;
    char* const myws = (char*)pm;
    unsigned* const cnt = (unsigned*)&red[4 * 32 * 64];

    const bool copier = idx >= 28;

    gu32* const seq = (gu32*)(myws + sqo);
    const unsigned last = poll(seq);

#define GET_CALL() const unsigned call = __builtin_amdgcn_readfirstlane(last) + 1u
    const __amdgpu_buffer_rsrc_t buf = __builtin_amdgcn_make_buffer_rsrc((void*)myws, 0, WS_USED, RSRC_W3);

    if (copier) {
        unsigned role = ((links >> 4 * (__builtin_amdgcn_s_getreg(20 | 3 << 11) & 7)) & 7) * 16 + (idx & 3) * 4 + wave;
        char* const wsl = ws_of_lane();
        if (role < 16) __builtin_amdgcn_s_sleep(127);
        for (unsigned dead = __builtin_amdgcn_readfirstlane(role < 16);;) {
        asm volatile("" : "+s"(dead));
        if (dead) {
            GET_CALL();
            if (lane == 0) post(seq, call);
            gu32* const taken = (gu32*)(myws + OFF_ROLE);
            const u64 m0 = __ballot(lane >= 16 && poll(taken + lane * 32) < call);
            const u64 m1 = __ballot(poll(taken + (64 + lane) * 32) < call);
            if ((m0 | m1) == 0) return;
            role = m0 ? __builtin_ctzll(m0) : 64 + __builtin_ctzll(m1);
        }
        const int link = role >> 4, v = role & 15;
        const unsigned q = (unsigned)(rank ^ link) & (NR - 1);
        const bool same = (link & 1) == 0;
        const int sub = v & 3, a = v >> 2;
        auto k1 = [&](int u) { return 8 * (u >> 2) + 4 * (u & 1) + (u & 2) + (v >> 3); };
        const int slot1 = slot_of(v, q, rank), gb1 = block_of(slot1, rank);
        const int sp = 2 * a + 1 - rj, cq = 12 + (q >> 1);
        const int uw = G1 + (same ? 24 - DIRECT : a == ri ? 8 : 8 + DIRECT), ux = uw + 8, uy = uw + 16, nu = same ? uy : uy + 16;
        auto ycol = [&](int n) { return n < 8 ? 3 * (n & 3) + (n >> 2) : n < 12 ? 12 + ((n + (q >> 1)) & 3) : 3 * n - 34; };
        unsigned poff = OFF_NONE, gate = 0;
        bool used = true;
        const unsigned efw = OFF_EF + (q * 64 + 32 + rj * 16) * 256 + (32 + sub) * 4;
        const unsigned yf = OFF_YF + (q * 8 + (v >> 1)) * 256;
        if (lane < 16) {
            const int u = G1 == 15 ? lane - (lane > 12) : lane - 2 * (lane > 11);
            gate = u & ~1;
            poff = OFF_EF + (q * 64 + gb1) * 256 + (G1 == 15 ? (lane == 12 ? 38 + (v >> 3) : k1(u))
                                                             : (lane >> 1 == 5 ? 16 + 2 * lane + (v >> 3) : k1(u))) * 4;
        } else if (lane == 16) { poff = efw + (3 * a + 1) * 256; gate = G1; }
        else if (lane == 17) { poff = efw + (12 + a) * 256; gate = G1 + 8; }
        else if (lane == 18 && same) { poff = OFF_RF + (q * 8 + sp) * 128 + (sub & 1) * 4; gate = G1 + 16; }
        else if (lane == 19) { poff = efw + (3 * a + 2) * 256; gate = uw; }
        else if (lane == 20) { poff = OFF_EF + (q * 64 + rank * 4 + a) * 256 + (32 + sub) * 4; gate = ux; }
        else if (lane == 21 && !same) { poff = OFF_MC; gate = uy; }
        else if (lane >= 32 && lane < 48 && !same) {
            poff = yf + ycol(lane - 32) * 16; gate = uy + lane - 32;
        } else used = false;
        if (lane == 63) gate = nu;
        char* const rfa = wsl + OFF_RF + (rank * 8 + q) * 128 + (v >> 3) * 4;
        const unsigned yo = (lane >> 5) * 4 * NN * 4 + (lane & 31) * 16;
        const unsigned tab = (unsigned)(size_t)(__attribute__((address_space(3))) f32x4*)red + wave * 32768;
#pragma unroll
        for (int i = 0; i < 5; ++i) red[wave * 2048 + i * 64 + lane] = f32x4{0.f, 0.f, 0.f, 0.f};

        unsigned so, dst, fo, fv;
        {
            const int u = lane;
            unsigned slot, src = q, k = 4 * ((u - (u < (same ? uw : G1 + 8) ? G1 : uw)) & 7) + sub;
            if (u < G1) { so = P_EXP + gb1 * PBLOCK; slot = slot1; k = k1(u); }
            else if (u >= ux) { so = P_EXP + (rank * 4 + a) * PBLOCK; slot = 4 + a; }
            else if (u >= G1 + 16 && u < uw) { src = sp; slot = 8 + cq; so = P_RLY + sp * PBLOCK; }
            else {
                slot = u < G1 + 8 ? 9 + 3 * a : u >= uw ? 10 + 3 * a : 20 + a;
                so = P_EXP + (24 + rj * 16 + slot) * PBLOCK;
            }
            so += k * PUNIT + 128 + ((k >> 2) & 1);
            dst = P_STG + (src * NSLOT + slot) * BLOCK + k * UNIT;
            fo = sf_of(src, slot, k & 3);
            fv = u == 13 || u == (G1 == 15 ? 14 : 11) || u == uw - 1 ? 32 : k + 1;
            if (!same && u >= G1 + 8 && u < uw) { fo += 4; fv = u == uw - 1 ? 32 : 0; }
            if (u >= uy) {
                const int n = ycol((u - uy) & 15);
                so = P_Y + (((ri * 2 + (v >> 3)) * 16 + n) * 8 + (v & 7)) * PUNIT + 129;
                dst = (((q >> 1) * 128 + v * 8) * NN + (q & 1) * 2048 + n * 128) * 4;
                fo = u < nu ? yf + n * 16 + 4 + (v & 1) * 6 : OFF_ROLE + role * 128;
                fv = P_M + ((((q >> 1) * 2 + (v >> 3)) * 16 + n) * 8 + (v & 7)) * UNIT;
            }
        }
        const int li = lane & 31;
        const int skip = !same && a == ri ? DIRECT : 0, done = lane + 1 + (lane >= G1 + 7 ? skip : 0);
        const unsigned inc = lane >= uy ? 0 : lane == uy - 1 ? 65 - done : lane == G1 + 7 ? 1 + skip : 1;
        unsigned lim = 16 * done + BOOST - 1024, unit = !used ? 0 : lane == 21 ? 512 : 64;
        unsigned lk = ((lane & 32) * 2 + (li < 20 ? 3 * li & ~3 : li == 31 ? 60 : 0)) | (li < 20 ? (24 * li & 31) << 8 : 0) |
                            (lane >= 32 && lane < 63 ? 12 << 13 : 0) | (lane == 63 ? 0 : 2 + 2 * (lane & 1)) << 18 | inc << 24;
        asm volatile("" : "+v"(so), "+v"(dst), "+v"(fo), "+v"(fv), "+v"(poff), "+v"(gate), "+v"(lk), "+v"(lim), "+v"(unit));
        GET_CALL();
        if (lane == 0) post(seq, call);
        const unsigned par = OFF_DATA + (call & 1) * P_BYTES;
        const unsigned thr = lane == 63 ? ~0u : call * unit + (unit & 64) / 2;
        so += par;
        dst += lane < uy ? par : 0;
        fv = lane == 63 ? call : fv + (lane < uy ? call * 64 : par);
        lim += call * 1024;
        asm volatile(R"(
.macro c_load r, i
  .if \i < 3
    global_load_dwordx4 v[\r+4*\i:\r+4*\i+3], %[l16], s[44:45] offset:1024*\i sc0 sc1
  .elseif \i == 3
    global_load_dwordx2 v[\r+12:\r+13], v58, s[44:45] offset:3072 sc0 sc1
  .else
    s_bitcmp1_b32 s43, 0
    s_cbranch_scc1 Lcc\@
    global_load_dword v[\r+14], v59, s[44:45] offset:-128 sc0 sc1
Lcc\@:
  .endif
.endm
.macro c_early r
    v_readlane_b32 s43, %[so], s41
    s_add_u32 s41, s41, 1
    s_andn2_b32 s48, s43, 1
    s_add_u32 s44, s58, s48
    s_addc_u32 s45, s59, 0
  .irp i, 0, 1, 2, 3, 4
   .if \i < %[early]
    c_load \r, \i
   .endif
  .endr
.endm
.macro c_late r
  .irp i, 0, 1, 2, 3, 4
   .if \i >= %[early]
    c_load \r, \i
   .endif
  .endr
.endm
.macro c_sum r, skip
    s_cmp_gt_u32 s41, %[uy]
    s_cbranch_scc0 \skip
    s_sub_u32 s48, s41, 1
    v_readlane_b32 s49, %[fo], s48
    v_readlane_b32 s48, %[fv], s48
    s_add_u32 s46, s60, s48
    s_addc_u32 s47, s61, 0
  .irp i, 0, 1, 2, 3
    global_load_dwordx4 v[56+\r+4*\i:59+\r+4*\i], %[l16], s[46:47] offset:1024*\i sc0 sc1
  .endr
    s_add_u32 s46, s60, s49
    s_addc_u32 s47, s61, 0
    global_load_ushort v[\r+14], v63, s[46:47] sc0 sc1
.endm
.macro c_flag n
    s_mov_b32 s68, 0
Lc6\@:
    s_cmp_lt_u32 s74, \n
    s_cbranch_scc0 Lc7\@
    s_lshl_b64 exec, 1, s74
  .irp i, 0, 1, 2, 3
   .if \i < )" STR(REP) R"(
    global_store_dword %[fo], %[fv], s[60:61] offset:128*\i sc0 sc1
   .endif
  .endr
    global_atomic_add v153, %[pace], v152, s[60:61] sc0 sc1
    s_add_u32 s74, s74, 1
    s_add_u32 s68, s68, 1+)" STR(REP) R"(
    s_cmp_eq_u32 s74, %[ur]
    s_cbranch_scc0 Lc6\@
    s_mov_b32 exec_lo, %[twins]
    global_store_dword %[rfa], %[thr], off sc0 sc1
    s_branch Lc6\@
Lc7\@:
    s_mov_b64 exec, -1
.endm
.macro c_take
    v_cmp_lt_u32 vcc, v112, %[thr]
    s_ff1_i32_b64 s48, vcc
    v_readlane_b32 s42, %[gate], s48
.endm
.macro c_poll
    global_load_dword v112, %[poff], s[60:61] sc0 sc1
.endm
.macro c_stamp n
)"
R"(
.endm
.macro c_near skip
    s_add_u32 s48, s41, %[win]
    s_min_u32 s48, s48, %[nu]
    s_cmp_lt_u32 s42, s48
    s_cbranch_scc0 \skip
    c_poll
    s_add_u32 s68, s68, 1
.endm
.macro c_step r, o
    s_sub_u32 s48, s41, s40
    s_cmp_eq_u32 s48, 2
    s_cbranch_scc0 Lcd\r\()_%=
    s_cmp_gt_u32 s41, %[uy]
    s_cbranch_scc1 Lck\r\()_%=
    s_cmp_lg_u32 s73, 0
    s_cbranch_scc1 Lc3\r\()_%=
    s_waitcnt vmcnt(%[early])
    c_late \o
Lca\r\()_%=:
    s_sub_u32 s48, s40, s73
    c_flag s48
Lcl\r\()_%=:
    c_take
Lce\r\()_%=:
    v_readlane_b32 s49, %[so], s40
    v_mov_b32_dpp v113, v[\r+14] quad_perm:[1,2,3,3] row_mask:0xf bank_mask:0xf
    s_cmp_lt_i32 s40, %[ur]
    s_cbranch_scc1 Lcr\r\()_%=
    s_bitcmp1_b32 s49, 0
    s_cbranch_scc1 Lcv\r\()_%=
Lcf\r\()_%=:
    v_alignbit_b32 v62, v113, v[\r+14], v60
    v_alignbit_b32 v[\r+14], v113, v[\r+14], v60
    s_nop 1
    v_permlane32_swap_b32 v[\r+14], v62
    s_nop 0
Lcx\r\()_%=:
    v_readlane_b32 s56, v[\r+14], 31
    v_lshrrev_b32 v[\r+14], v61, v[\r+14]
    s_and_b32 s48, s56, 0x4000
    s_lshr_b32 s56, s56, 16
    v_and_b32 v113, 0x7ff, v[\r+14]
    v_lshl_or_b32 v[\r+14], v[\r+14], 2, s48
Lcx2\r\()_%=:
    s_bitcmp1_b32 s56, 15
    s_cbranch_scc1 Lcw\r\()_%=
    s_mul_i32 s56, s56, s69
    v_mad_u32_u24 v113, v113, s66, v117
    v_and_b32 v114, s54, v[\r]
    ds_write_b16 v113, v[\r+14]
    ds_read_b128 v[96:99], %[tl]
    ds_read_b128 v[100:103], %[tl] offset:1024
    ds_read_b128 v[104:107], %[tl] offset:2048
    ds_read_b128 v[108:111], %[tl] offset:3072
    ds_write_b16 v113, v116
    v_and_b32 v115, s54, v[\r+7]
  .irp i, 1, 2, 3, 4, 5, 6
    v_lshrrev_b32 v114, s66, v114
    v_lshrrev_b32 v115, s66, v115
    v_bfi_b32 v114, s54, v[\r+\i], v114
    v_bfi_b32 v115, s54, v[\r+7+\i], v115
  .endr
    v_lshlrev_b32 v[\r+14], 1, v114
    v_lshlrev_b32 v[\r+15], 1, v115
    v_lshrrev_b32 v114, s67, v114
    v_lshrrev_b32 v115, s67, v115
    v_bfi_b32 v[\r+14], s55, v[\r+14], v114
    v_bfi_b32 v[\r+15], s55, v[\r+15], v115
    s_waitcnt lgkmcnt(0)
  .irp i, 0, 1, 2, 3, 4, 5, 6
    v_bfi_b32 v[\r+\i], s54, v[96+\i], v[\r+\i]
    v_bfi_b32 v[\r+7+\i], s54, v[104+\i], v[\r+7+\i]
  .endr
    v_bfi_b32 v[\r+14], s54, v103, v[\r+14]
    v_bfi_b32 v[\r+15], s54, v111, v[\r+15]
  .irp i, 0, 1, 2, 3, 4, 5, 6, 7, 8, 9, 10, 11, 12, 13, 14, 15
    v_pk_add_u16 v[\r+\i], v[\r+\i], s56
  .endr
Lcs\r\()_%=:
    v_readlane_b32 s48, %[dst], s40
    s_cmp_ge_u32 s40, %[uy]
    s_cbranch_scc1 Lcy\r\()_%=
    s_add_u32 s46, s60, s48
    s_addc_u32 s47, s61, 0
    global_store_dwordx4 %[l16], v[\r:\r+3], s[46:47] sc0 sc1
    global_store_dwordx4 %[l16], v[\r+4:\r+7], s[46:47] offset:1024 sc0 sc1
    global_store_dwordx4 %[l16], v[\r+8:\r+11], s[46:47] offset:2048 sc0 sc1
    global_store_dwordx4 %[l16], v[\r+12:\r+15], s[46:47] offset:3072 sc0 sc1
    c_near Lcp\r\()_%=
Lcp\r\()_%=:
    s_add_u32 s40, s40, 1
    s_cmp_eq_u32 s40, %[uy]
    s_cbranch_scc1 Lcm\r\()_%=
Lco\r\()_%=:
    s_cmp_gt_u32 s41, s40
    s_cselect_b32 s48, s42, 0
    s_cmp_gt_u32 s48, s41
    s_cbranch_scc0 Lcq\r\()_%=
    c_early \r
    v_cmp_ne_u32 vcc, 0, v153
    s_flbit_i32_b64 s48, vcc
    v_cmp_ge_u32 vcc, v153, %[lim]
    s_not_b32 s48, s48
    s_bitcmp1_b64 vcc, s48
    s_cselect_b32 s73, 1, 0
    c_sum \r, Lc1\r\()_%=
Lc2\r\()_%=:
    c_late \r
Lcq\r\()_%=:
    c_stamp s40
.endm
.macro c_rare r, o
Lc1\r\()_%=:
    s_cmp_lg_u32 s73, 0
    s_cbranch_scc0 Lcq\r\()_%=
    s_bitcmp0_b32 s43, 0
    s_addc_u32 s68, s68, 8
    s_branch Lc2\r\()_%=
Lc3\r\()_%=:
  .irp n, 20, 19, 18, 17, 16, 15, 14, 13, 12, 11, 10, 9
   .if \n <= 12+2*)" STR(REP) R"(
    s_waitcnt vmcnt(\n)
    s_cmp_ge_u32 s68, \n
    s_cbranch_scc1 Lca\r\()_%=
   .endif
  .endr
    s_waitcnt vmcnt(8)
    s_branch Lca\r\()_%=
Lck\r\()_%=:
    s_waitcnt vmcnt(13)
    s_cmp_gt_u32 s40, %[uy]
    s_cbranch_scc1 Lcl\r\()_%=
    s_waitcnt vmcnt(9)
    s_mov_b32 s73, 0
    s_branch Lca\r\()_%=
Lcd\r\()_%=:
    s_waitcnt vmcnt(4)
    s_cmp_gt_u32 s40, %[uy]
    s_cbranch_scc1 Lcb\r\()_%=
    s_waitcnt vmcnt(0)
    c_flag s40
    c_stamp 58
Lcb\r\()_%=:
    c_take
    s_mov_b32 s68, 0
    s_cmp_gt_u32 s41, s40
    s_cbranch_scc0 Lcg\r\()_%=
    s_cmp_gt_u32 s42, s41
    s_cbranch_scc0 Lce\r\()_%=
    c_early \o
    c_late \o
    c_sum \o, Lce\r\()_%=
    s_branch Lce\r\()_%=
Lcg\r\()_%=:
    s_cmp_eq_u32 s40, %[nu]
    s_cbranch_scc1 Lcz_%=
    s_memrealtime s[50:51]
Lch\r\()_%=:
    s_cmp_gt_u32 s42, s40
    s_cbranch_scc1 Lci\r\()_%=
    s_sleep 2
    c_poll
    s_memrealtime s[52:53]
    s_waitcnt vmcnt(0) lgkmcnt(0)
    c_take
    s_sub_u32 s48, s52, s50
    s_cmp_gt_u32 s48, %[ticks]
    s_cbranch_scc0 Lch\r\()_%=
    s_mov_b32 %[dead], 1
    s_branch Lcz_%=
Lci\r\()_%=:
    c_early \r
    c_late \r
)"
R"(
    c_sum \r, Lct\r\()_%=
Lct\r\()_%=:
    s_cmp_gt_u32 s42, s41
    s_cbranch_scc1 Lcj\r\()_%=
    s_waitcnt vmcnt(0) lgkmcnt(0)
    s_branch Lce\r\()_%=
Lcj\r\()_%=:
    c_early \o
    c_sum \o, Lcu\r\()_%=
    c_late \o
    s_waitcnt vmcnt(9) lgkmcnt(0)
    s_branch Lce\r\()_%=
Lcu\r\()_%=:
    s_waitcnt vmcnt(%[early]) lgkmcnt(0)
    c_late \o
    s_branch Lce\r\()_%=
Lcy\r\()_%=:
    s_add_u32 s46, s62, s48
    s_addc_u32 s47, s63, 0
    c_near Lcn\r\()_%=
Lcn\r\()_%=:
  .irp i, 0, 2, 4, 6, 8, 10, 12, 14
    v_pk_add_f32 v[\r+\i:\r+\i+1], v[\r+\i:\r+\i+1], v[56+\r+\i:57+\r+\i]
  .endr
  .irp i, 0, 2, 4, 6, 8, 10, 12, 14
    v_pk_mul_f32 v[96+\i:97+\i], v[\r+\i:\r+\i+1], s[70:71]
  .endr
  .irp i, 0, 1, 2, 3, 4, 5, 6, 7, 8, 9, 10, 11, 12, 13, 14, 15
    v_med3_f32 v[96+\i], v[96+\i], -s72, s72
  .endr
  .irp i, 0, 1, 2, 3, 4, 5, 6, 7, 8, 9, 10, 11, 12, 13, 14, 15
    v_exp_f32 v[96+\i], v[96+\i]
  .endr
  .irp i, 0, 2, 4, 6, 8, 10, 12, 14
    v_pk_add_f32 v[96+\i:97+\i], v[96+\i:97+\i], v[118:119]
  .endr
  .irp i, 0, 4, 8, 12
    v_rcp_f32 v[96+\i], v[96+\i]
    v_rcp_f32 v[97+\i], v[97+\i]
    v_rcp_f32 v[98+\i], v[98+\i]
    v_rcp_f32 v[99+\i], v[99+\i]
    v_pk_mul_f32 v[\r+\i:\r+\i+1], v[\r+\i:\r+\i+1], v[96+\i:97+\i]
    v_pk_mul_f32 v[\r+\i+2:\r+\i+3], v[\r+\i+2:\r+\i+3], v[98+\i:99+\i]
    global_store_dwordx4 %[yo], v[\r+\i:\r+\i+3], s[46:47] sc0 sc1
    s_add_u32 s46, s46, 16384
    s_addc_u32 s47, s47, 0
  .endr
    s_branch Lcp\r\()_%=
Lcm\r\()_%=:
    s_mov_b32 s54, 0x78000000
    s_brev_b32 s55, 1
    s_mov_b32 s66, 4
    s_mov_b32 s67, 3
    s_mov_b32 s69, 0x10000
    v_add_u32 v117, 2, %[tab]
    s_branch Lco\r\()_%=
Lcr\r\()_%=:
    s_andn2_b32 s48, s49, 1
    s_add_u32 s48, s48, %[rly]
    s_add_u32 s64, s60, s48
    s_addc_u32 s65, s61, 0
    global_store_dwordx4 %[l16], v[\r:\r+3], s[64:65] sc0 sc1
    global_store_dwordx4 %[l16], v[\r+4:\r+7], s[64:65] offset:1024 sc0 sc1
    global_store_dwordx4 %[l16], v[\r+8:\r+11], s[64:65] offset:2048 sc0 sc1
    global_store_dwordx2 v58, v[\r+12:\r+13], s[64:65] offset:3072 sc0 sc1
    s_bitcmp1_b32 s49, 0
    s_cbranch_scc1 Lcv\r\()_%=
    global_store_dword v59, v[\r+14], s[64:65] offset:-128 sc0 sc1
    s_branch Lcf\r\()_%=
Lcv\r\()_%=:
    s_cmp_ge_u32 s40, %[uy]
    s_cbranch_scc1 Lcv2\r\()_%=
    v_mov_b32 v[\r+14], v62
    s_branch Lcx\r\()_%=
Lcv2\r\()_%=:
    v_readlane_b32 s56, v[\r+14], 63
    v_writelane_b32 v[\r+14], s57, 63
    v_and_b32 v113, 0x7ff, v[\r+14]
    s_branch Lcx2\r\()_%=
Lcw\r\()_%=:
    s_andn2_b32 s48, s49, 1
    s_add_u32 s44, s58, s48
    s_addc_u32 s45, s59, 0
    global_load_dwordx2 v[\r+14:\r+15], v58, s[44:45] offset:3584 sc0 sc1
    s_cmp_lt_i32 s40, %[ur]
    s_waitcnt vmcnt(0)
    s_cbranch_scc0 Lcs\r\()_%=
    global_store_dwordx2 v58, v[\r+14:\r+15], s[64:65] offset:3584 sc0 sc1
    s_branch Lcs\r\()_%=
.endm
    s_mov_b64 s[58:59], %[qb]
    s_mov_b64 s[60:61], %[mb]
    s_lshl_b64 exec, 1, 63
    global_atomic_umax v154, %[fo], %[fv], s[60:61] sc0 sc1
    s_mov_b64 exec, -1
    c_poll
    s_mov_b64 s[62:63], %[ob]
    s_mov_b32 s54, 0x60006000
    s_mov_b32 s55, 0x80008000
    s_movk_i32 s57, 0x400
    s_mov_b32 s66, 2
    s_mov_b32 s67, 1
    v_lshrrev_b32 v58, 1, %[l16]
    v_and_b32 v59, 0xff, %[lk]
    v_bfe_u32 v60, %[lk], 8, 5
    v_bfe_u32 v61, %[lk], 13, 5
    v_bfe_u32 v63, %[lk], 18, 6
    v_lshrrev_b32 v152, 24, %[lk]
    v_mov_b32 v153, 0
    s_mov_b32 s69, 0x10001
    s_mov_b32 s70, 0xbfb8aa3b
    s_mov_b32 s71, s70
    s_mov_b32 s72, 0x42ad1f97
    v_mov_b32 v117, %[tab]
    v_mov_b32 v118, 1.0
    v_mov_b32 v119, 1.0
    s_mov_b32 s40, 0
    s_mov_b32 s74, 0
    s_mov_b32 s41, 0
    v_mov_b32 v116, 0
    s_waitcnt vmcnt(0)
    v_cmp_ge_u32 vcc, v154, %[fv]
    s_lshr_b32 %[dead], vcc_hi, 30
    s_bitcmp1_b32 vcc_hi, 31
    s_cbranch_scc1 Lcz_%=
    s_mov_b32 %[dead], 0
    c_stamp 62
Lc0_%=:
    c_step 64, 80
    c_step 80, 64
    s_branch Lc0_%=
    c_rare 64, 80
    c_rare 80, 64
Lcz_%=:
    s_waitcnt vmcnt(0) lgkmcnt(0)
    c_stamp 59
.purgem c_load
.purgem c_early
.purgem c_late
.purgem c_flag
.purgem c_take
.purgem c_poll
.purgem c_stamp
.purgem c_near
.purgem c_sum
.purgem c_step
.purgem c_rare
)"
            : [dead] "=&s"(dead)
            : [so] "v"(so), [dst] "v"(dst), [fo] "v"(fo), [fv] "v"(fv), [poff] "v"(poff), [gate] "v"(gate), [thr] "v"(thr),
              [l16] "v"(lane * 16), [lk] "v"(lk), [lim] "v"(lim), [pace] "v"(OFF_PACE + link * 128), [twins] "s"((0x55u << rj) & ~(1u << rank)), [yo] "v"(yo), [rfa] "v"(rfa),
              [tl] "v"(tab + lane * 16), [tab] "s"(tab), [qb] "s"(ws_of(wsl, q)), [mb] "s"(myws), [ob] "s"(p.out),
              [nu] "s"(nu), [uy] "s"(same ? 64 : uy), [ur] "s"(slot1 >= 20 ? 16 - 2 * DIRECT : -1),
              [rly] "s"(P_RLY + q * PBLOCK - P_EXP - gb1 * PBLOCK), [early] "n"(EARLY), [win] "n"(POLLWIN), [ticks] "n"(DEADLINE_TICKS)
            : "memory", "vcc", "scc", "s40", "s41", "s42", "s43", "s44", "s45", "s46", "s47", "s48", "s49", "s50", "s51", "s52",
              "s53", "s54", "s55", "s56", "s57", "s58", "s59", "s60", "s61", "s62", "s63", "s64", "s65", "s66", "s67", "s68",
              "s69", "s70", "s71", "s72", "s73", "s74", CLOBBER_V);
        if (dead >= 2) continue;
        if (dead && !same && lane == nu - 1) *(float*)(p.out + dst) = __builtin_nanf("");
        STAMP_AT(11);
        STAMP_T0();
        return;
        }
    }

    lds16* const lst = (lds16*)&red[4 * 32 * 64 + 1] + wave * 64;
    if (wave < 2 * G1 - 28) start = start >> 8 & 63 | start & ~255u;
    const bool send = start & 64;
    const unsigned nap = send ? (wave < 2 ? DELAYS : 0) : turn;
    int g = start & 63;
    if (nap) {
        __builtin_amdgcn_s_barrier();
        while ((unsigned)__builtin_amdgcn_s_memrealtime() - t0 < nap) __builtin_amdgcn_s_sleep(2);
    }
    f32x4 v0[8];
    STAMP_V(0);
    conv_load(v0, px, pw, g, send ? kss : kso, vx, vw);
    __builtin_amdgcn_sched_barrier(0);
    if (!nap) __builtin_amdgcn_s_barrier();
    char* const wsl = ws_of_lane();
    const int gb0 = (start >> 8) & 63, to = (start >> 16) & (NR - 1), sslot = start >> 24;
    STAMP_V(1);
    for (int job = send ? -1 : 0;;) {
        const int sub = b & 3, part = (b >> 2) & 3, k = 16 * (job & 1) + 4 * wave + sub;
        const bool isx = g < 32, late = job >= 2;
        const int other = (g & 3) * 2 + 1 - ((g >> 2) & 1);
        const bool efw = lane < NR && (lane & 1) == ((g >> 4) & 1);
        const bool efl = isx ? lane == g >> 2 || (lane == other && !late) : efw || lane == rank;
        const bool efu = !late && first && lane != rank && (isx ? lane == other : efw && ((g & 15) < 12 || lane >> 1 == part));
        const unsigned efo = OFF_EF + (rank * 64 + g) * 256;
        const int mine = !isx ? (((g >> 4) & 1) == rj ? rank * NSLOT + 8 : NR * NSLOT) + (g & 15)
                              : late ? rank * NSLOT + 4 + part : other == rank ? rank * NSLOT + (g >> 3) : -1;
        u32x4 f[4];
        STAMP_V(6 + 5 * job);
        conv_half(v0, isx, f);
        GET_CALL();
        STAMP_V(7 + 5 * job);
        const unsigned par = OFF_DATA + (call & 1) * P_BYTES;
        if (job < 0) {
            const __amdgpu_buffer_rsrc_t dst = __builtin_amdgcn_make_buffer_rsrc(ws_of(wsl, to), 0, WS_USED, RSRC_W3);
#pragma unroll
            for (int e = 0; e < 4; ++e)
                __builtin_amdgcn_raw_buffer_store_b128(f[e], dst, par + P_STG + (rank * NSLOT + sslot) * BLOCK +
                                                       (26 + wave + (wave & 2)) * UNIT + e * 1024 + lane * 16, 0, AUX_SYS);
            STAMP_V(3);
            asm volatile("s_waitcnt vmcnt(0)" ::: "memory");
            STAMP_V(4);
            if (lane == to) post((gu32*)(wsl + efo + (36 + wave) * 4), call * 64 + 32);
            while ((unsigned)__builtin_amdgcn_s_memrealtime() - t0 < turn) __builtin_amdgcn_s_sleep(2);
        } else {
            if (mine >= 0) {
#pragma unroll
                for (int e = 0; e < 4; ++e)
                    __builtin_amdgcn_raw_buffer_store_b128(f[e], buf, par + P_STG + mine * BLOCK + k * UNIT +
                                                           e * 1024 + lane * 16, 0, AUX_SYS);
            }
            if (!(isx && late))
                pack_store<false>(v0, f, buf, par + P_EXP + g * PBLOCK + k * PUNIT,
                                  par + P_EXP + g * PBLOCK + (k & ~4) * PUNIT + (k & 4) * 16, lst, lane);
            STAMP_V(8 + 5 * job);
            asm volatile("s_waitcnt vmcnt(0)" ::: "memory");
            STAMP_V(9 + 5 * job);
            if (efu) post((gu32*)(wsl + efo + k * 4), call * 64 + 32);
            if (job & 1) {
                unsigned n = 0;
                if (lane == 0) n = __hip_atomic_fetch_add(cnt + 2, 1u, __ATOMIC_RELAXED, __HIP_MEMORY_SCOPE_WORKGROUP);
                if ((__builtin_amdgcn_readfirstlane(n) & 3) == 3 && efl) post((gu32*)(wsl + efo + (32 + sub) * 4), call * 64 + 32);
                if (late || idx < 24) break;
                __syncthreads();
            } else __builtin_amdgcn_s_sleep(DELAY2);
        }
        ++job;
        g = job >= 2 ? (b & 16 ? rank * 4 : 44 + rj * 16) + part : gb0;
        conv_load(v0, px, pw, g, kso + 4 * (job & 1), vx, vw);
    }

    GET_CALL();
    if (lane == 0) post(seq, call);
    const unsigned fb = call * 64;
    const unsigned par = OFF_DATA + (call & 1) * P_BYTES;
    const int rh = (xcd & 1) * 4 + (idx & 3);
    const int njob = idx < 4 || idx >= 12 ? 2 : 1;
    const u32x4 bufw = {(unsigned)(size_t)myws, (unsigned)((size_t)myws >> 32) & 0xffff, WS_USED, RSRC_W3};
    unsigned dead = 0;
    for (int job = 0; job < njob; ++job) {
        const bool own = job == 1 && idx < 24, slab = job == 0 && idx >= 12;
        const int cb = idx >= 24 || (idx < 4 && job == 1) ? 12 + (xcd >> 1) : (xcd >> 1) * 3 + (idx >> 2) % 3;
        const int a = rh >> 1, peer = 2 * a + 1 - rj;
        const int s = lane >> 3, jj = lane & 7;
        const int gx = own || slab ? rank * 4 + a : peer * 4 + ri, gw = 32 + (rj ^ slab) * 16 + cb;
        const int sx = own || slab ? 4 + a : a;
        const bool isw = jj == 1 || jj == 2, on = own || (slab ? s == rank : s != peer);
        const bool two = ((s ^ rank) & 1) && cb >= 12 && cb != 12 + ri, spare = FEW && jj > 1 + two;
        unsigned poff;
        if (!on || spare) poff = WS_USED;
        else if (s == rank) poff = OFF_EF + (rank * 64 + (isw ? gw : gx)) * 256 + (32 + wave) * 4;
        else if (!isw) poff = sf_of(s, sx, wave) + (xcd >> 1) * REP / 4 * 128;
        else poff = sf_of(s, 8 + cb, wave) + ((xcd & 1) * 2 + (idx & 1)) * REP / 4 * 128 + (two ? jj - 1 : 0) * 4;
        unsigned xoff = par + P_STG + (s * NSLOT + sx) * BLOCK + wave * UNIT + (rh & 1) * 2048;
        const unsigned wd = par + P_STG + ((slab ? NR * NSLOT : s * NSLOT + 8) + cb) * BLOCK + wave * UNIT - xoff;
        const unsigned step = jj == 0 && on ? 4 * UNIT : 0, xb0 = jj == 0 ? xoff : 0;
        const unsigned lds = (unsigned)(size_t)(__attribute__((address_space(3))) f32x4*)red + wave * 32768 + lane * 16;
        if (slab && idx < 24)
            while ((unsigned)__builtin_amdgcn_s_memrealtime() - t0 < MBASE + (idx - 12) * MSTAG) __builtin_amdgcn_s_sleep(8);
        if (job == 1) __builtin_amdgcn_s_barrier();
        asm volatile(R"(
.macro g_x r, o
    buffer_load_dwordx4 v[64+24*\r+4*\o:67+24*\r+4*\o], %[vo], %[buf], s40 offen offset:1024*\o )" DSC R"(
.endm
.macro g_w r, o
    buffer_load_dwordx4 v[72+24*\r+4*\o:75+24*\r+4*\o], %[vo], %[buf], s41 offen offset:1024*\o )" DSC R"(
.endm
.macro g_m c, m, e
    v_mfma_f32_32x32x16_f16 a[64*\m+16*\e:64*\m+16*\e+15], v[64+24*\c+4*\m:67+24*\c+4*\m], v[72+24*\c+4*\e:75+24*\c+4*\e], a[64*\m+16*\e:64*\m+16*\e+15]
.endm
.macro g_i z, n, i
  .if \i < 8*\n
    g_m (\z+\i/8), ((\i/4)&1), (\i&3)
  .endif
.endm
.macro g_poll
    buffer_load_dword v60, %[poff], %[buf], 0 offen )" PSC R"(
.endm
.macro g_take z, n, again
    g_i \z, \n, 0
    v_max_u32 v62, v62, v60
  .if \again
    g_poll
    s_nop 0
  .else
    s_nop 1
  .endif
    v_min_u32_dpp v63, v62, v62 quad_perm:[1,0,3,2] row_mask:0xf bank_mask:0xf
    v_cmp_gt_u32 vcc, %[fin], v62
    s_nop 0
    v_min_u32_dpp v63, v63, v63 quad_perm:[2,3,0,1] row_mask:0xf bank_mask:0xf
    g_i \z, \n, 1
    v_cndmask_b32 %[poff], v58, %[poff], vcc
    s_nop 0
    v_min_u32_dpp v63, v63, v63 row_half_mirror row_mask:0xf bank_mask:0xf
    v_sub_u32 v63, v63, %[base] clamp
    v_lshrrev_b32 v63, 2, v63
    v_mad_u32_u24 v59, v63, %[step], %[xb0]
    v_cmp_gt_u32 vcc, v59, %[xoff]
    g_i \z, \n, 2
.endm
.macro g_load r, z, n, i
    s_ff1_i32_b64 s42, vcc
    v_readlane_b32 s45, %[xoff], s42
    v_readlane_b32 s46, %[wd], s42
    v_cmp_eq_u32 vcc, s42, %[lane]
    s_mov_b32 s40, s45
    s_add_u32 s41, s45, s46
    g_x \r, 0
    g_i \z, \n, \i
    g_x \r, 1
    v_cndmask_b32 v63, 0, %[step], vcc
    g_w \r, 0
    v_add_u32 %[xoff], %[xoff], v63
    g_i \z, \n, (\i+1)
    g_w \r, 1
    v_cmp_gt_u32 vcc, v59, %[xoff]
    g_w \r, 2
    s_sub_u32 s44, s44, 1
    g_i \z, \n, (\i+2)
    g_w \r, 3
)"
R"(
.endm
.macro g_next y, z, n, i, nx
  .irp k, 3, 4, 5, 6, 7, 8, 9, 10, 11, 12, 13, 14, 15
   .if \k >= \i
    g_i \z, \n, \k
   .endif
  .endr
  .if \nx < 2
    s_cmp_le_u32 s44, %[tail]
    s_cbranch_scc1 Lz\@
    s_sleep %[pace]
Lz\@:
  .endif
    s_mov_b32 s45, s47
    s_mov_b32 s47, \nx
    s_cmp_eq_u32 s45, 1
    s_waitcnt vmcnt(6*\nx)
    s_cbranch_scc1 Lt\y\()1_%=
    s_cmp_eq_u32 s45, 2
    s_cbranch_scc1 Lt\y\()2_%=
  .if \nx
    s_branch Lt\y\()0_%=
  .else
    s_branch Lb_%=
  .endif
.endm
.macro g_round x, y, z, n
Lk\x\n\()_%=:
    s_cmp_le_u32 s44, %[tail]
    s_cbranch_scc1 Lj\x\n\()_%=
    g_i \z, \n, 0
    g_i \z, \n, 1
    g_i \z, \n, 2
    s_branch Ll\x\n\()_%=
Lt\x\n\()_%=:
    s_cbranch_vccnz Lk\x\n\()_%=
Lj\x\n\()_%=:
    g_take \z, \n, 1
    s_cbranch_vccz Lu\x\n\()_%=
Ll\x\n\()_%=:
    g_load \x, \z, \n, 3
    s_cbranch_vccz Lv\x\n\()_%=
    g_load (\x+1), \z, \n, 6
    g_next \y, \z, \n, 9, 2
Lv\x\n\()_%=:
    g_next \y, \z, \n, 6, 1
Lu\x\n\()_%=:
    g_next \y, \z, \n, 3, 0
.endm
    s_nop 4
    g_poll
    v_mov_b32 v62, %[seen]
    v_mov_b32 v58, %[oob]
    s_mov_b32 s47, 0
    s_mov_b32 s44, %[nst]
  .set g_k, 0
  .rept 128
    v_accvgpr_write_b32 a[g_k], 0
  .set g_k, g_k+1
  .endr
    s_waitcnt vmcnt(0)
Lb_%=:
    s_cmp_eq_u32 s44, 0
    s_cbranch_scc1 Ld_%=
    s_memrealtime s[48:49]
Lp_%=:
    v_max_u32 v60, %[dead], v60
    g_take 0, 0, 0
    s_cbranch_vccnz Lq_%=
    s_sleep %[slp]
    s_cmp_le_u32 s44, %[tail]
    s_cbranch_scc1 Lr_%=
    s_sleep %[idle]
Lr_%=:
    g_poll
    s_memrealtime s[50:51]
    s_waitcnt vmcnt(0) lgkmcnt(0)
    s_sub_u32 s45, s50, s48
    s_cmp_gt_u32 s45, %[ticks]
    s_cselect_b32 %[dead], %[last], %[dead]
    s_branch Lp_%=
Lq_%=:
    g_poll
    s_branch Ll00_%=
  .irp n, 0, 1, 2
    g_round 0, 2, 2, \n
    g_round 2, 4, 4, \n
    g_round 4, 0, 0, \n
  .endr
Ld_%=:
)"
R"(
    s_nop 7
  .set g_k, 0
  .rept 32
    ds_write_b128 %[lds], a[64*(g_k/16)+16*(g_k&3)+(g_k&12):64*(g_k/16)+16*(g_k&3)+(g_k&12)+3] offset:1024*g_k
  .set g_k, g_k+1
  .endr
    s_waitcnt lgkmcnt(0)
.purgem g_x
.purgem g_w
.purgem g_m
.purgem g_i
.purgem g_poll
.purgem g_take
.purgem g_load
.purgem g_next
.purgem g_round
)"
            : [poff] "+v"(poff), [xoff] "+v"(xoff), [dead] "+s"(dead)
            : [vo] "v"(lane * 16), [wd] "v"(wd), [step] "v"(step), [xb0] "v"(xb0), [lane] "v"(lane), [lds] "v"(lds), [seen] "v"(spare ? ~0u : 0u),
              [buf] "s"(bufw), [fin] "s"(fb + wave + 29), [base] "s"(fb + wave - 3), [last] "s"(fb + 32), [nst] "s"(own ? 64 : slab ? 8 : 56),
              [oob] "n"(WS_USED), [slp] "n"(POLL_SLEEP), [tail] "n"(TAIL), [pace] "n"(PACE), [idle] "n"(IDLE), [ticks] "n"(DEADLINE_TICKS)
            : "memory", "vcc", "scc", "s40", "s41", "s42", "s43", "s44", "s45", "s46", "s47", "s48", "s49", "s50", "s51",
              CLOBBER_V, CLOBBER_A);
        if (dead) cnt[3] = 1;
        STAMP_V(10);
        asm volatile("s_waitcnt lgkmcnt(0)\n\ts_barrier" ::: "memory");
        STAMP_V(8);
        const bool bad = cnt[3] != 0;
        const __amdgpu_buffer_rsrc_t od = own ? __builtin_amdgcn_make_buffer_rsrc((void*)p.out, 0, IN_BYTES, RSRC_W3) : buf;
        const unsigned pitch = own ? NN * 4 : 1024, hstep = own ? 8 * NN * 4 : UNIT, yu = (rh * 16 + cb) * 8 + wave * 2;
        const unsigned ooff = own ? ((rh * 64 + wave * 16 + (lane >> 5) * 4) * NN + rj * 2048 + cb * 128) * 4 + (lane & 31) * 16
                                  : par + P_M + yu * UNIT + lane * 16;
        u64 yl[2] = {0, 0};
#pragma unroll
        for (int h = 0; h < 2; ++h) {
            f32x4 y[4];
            u32x4 o[4];
#pragma unroll
            for (int e = 0; e < 4; ++e) {
                const int i = ((wave * 2 + h) * 4 + e) * 64 + lane;
                y[e] = (red[i] + red[i + 32 * 64]) + (red[i + 2 * 32 * 64] + red[i + 3 * 32 * 64]);
            }
            if (bad) y[0].x = __builtin_nanf("");
            if (own) {
#pragma unroll
                for (int e = 0; e < 4; ++e) {
                    f32x4 z = y[e] * -1.44269504f;
#pragma unroll
                    for (int t = 0; t < 4; ++t) z[t] = __builtin_amdgcn_exp2f(__builtin_amdgcn_fmed3f(z[t], -86.5617025f, 86.5617025f));
                    z += 1.f;
#pragma unroll
                    for (int t = 0; t < 4; ++t) z[t] = __builtin_amdgcn_rcpf(z[t]);
                    y[e] *= z;
                }
            }
#pragma unroll
            for (int t = 0; t < 4; ++t) o[t] = __builtin_bit_cast(u32x4, f32x4{y[0][t], y[1][t], y[2][t], y[3][t]});
            if (own || slab) {
#pragma unroll
                for (int t = 0; t < 4; ++t) __builtin_amdgcn_raw_buffer_store_b128(o[t], od, ooff + t * pitch + h * hstep, 0, AUX_SYS);
            } else yl[h] = pack_store<true>(y, o, buf, par + P_Y + (yu + h) * PUNIT, 0, lst, lane);
        }
        if (!own) {
            asm volatile("s_waitcnt vmcnt(0)" ::: "memory");
            if (lane == 0) {
                if (slab) __hip_atomic_fetch_add((gu32*)(myws + OFF_MC), 1u, __ATOMIC_RELAXED, __HIP_MEMORY_SCOPE_SYSTEM);
                else __builtin_amdgcn_raw_buffer_store_b128(
                    u32x4{fb + 32, (unsigned)yl[0], (unsigned)(yl[0] >> 32 | yl[1] << 16), (unsigned)(yl[1] >> 16)},
                    __builtin_amdgcn_make_buffer_rsrc(ws_of(wsl, peer), 0, WS_USED, RSRC_W3),
                    OFF_YF + ((rank * 2 + (rh & 1)) * 4 + wave) * 256 + cb * 16, 0, AUX_SYS);
            }
        }
        STAMP_V(11);
    }
}

extern "C" void kernel_launch(int rank, int nranks, ncclComm_t comm,
                              hipStream_t stream,
                              void* const* d_in, const int* in_sizes, int n_in,
                              void* d_out, int out_size,
                              const int* in_dtypes, int out_dtype,
                              void* const* d_out_all,
                              void* const* d_ws_all, size_t ws_bytes) {
    if (nranks != NR || (unsigned)rank >= (unsigned)NR || ws_bytes < WS_USED || n_in != 2 ||
        in_sizes[0] != 4096 * 512 || in_sizes[1] != 4096 * 512 || out_size != 4096 * 512) {
        fprintf(stderr, "unexpected problem shape\n");
        return;
    }
    Params p;
    p.x = (const float*)d_in[0];
    p.w = (const float*)d_in[1];
    for (int r = 0; r < NR; ++r) p.ws[r] = (char*)d_ws_all[r];
    p.mine = p.ws[rank];
    p.out = (char*)d_out;

    p.rank_links = rank;
    for (int q = 0, lo = 1, hi = 4; q < NR; ++q)
        if (q != rank) p.rank_links |= (u64)(rank ^ q) << (32 + 4 * (((GROUP >> q) ^ (GROUP >> rank)) & 1 ? hi++ : lo++));
    for (int b = 0; b < NWG; ++b) {
        const int g = b >> 4 == rank ? 60 - (rank & 1) * 16 + ((b >> 2) & 3) : ((b >> 2) + ((b + 80) >> 8) * 4) & 63;
        const int to = rank ^ ((p.rank_links >> (32 + 4 * (b & 7))) & 7), slot = slot_of((g & 16) / 4 + (g & 15) / 3, rank, to);
        const bool send = to != rank && !((0xf249f249ffffffffull >> g) & 1);
        p.start[b] = (send ? 64 + block_of(slot, to) : g) | g << 8 | to << 16 | slot << 24;
    }
    fused_gemm_silu<<<NWG, 256, 0, stream>>>(p);
    HIP_CHECK(hipGetLastError());
}
```

```cpp
#include <hip/hip_runtime.h>
#include <rccl/rccl.h>
#include <cstddef>
#include <cstdio>

#define HIP_CHECK(expr) do { hipError_t _e = (expr); if (_e != hipSuccess) { \
    fprintf(stderr, "HIP error %s at %s:%d\n", hipGetErrorString(_e), __FILE__, __LINE__); \
    return; } } while (0)

#define STR_(x) #x
#define STR(x) STR_(x)
typedef unsigned u32x4 __attribute__((ext_vector_type(4)));
typedef unsigned u32x2 __attribute__((ext_vector_type(2)));
typedef unsigned short u16x2 __attribute__((ext_vector_type(2)));
typedef float f32x4 __attribute__((ext_vector_type(4)));
typedef float f32x16 __attribute__((ext_vector_type(16)));
typedef _Float16 f16x8 __attribute__((ext_vector_type(8)));
typedef __attribute__((address_space(1))) unsigned gu32;
typedef volatile __attribute__((address_space(3))) unsigned short lds16;
typedef unsigned long long u64;
typedef u64 u64x4 __attribute__((ext_vector_type(4)));

#define POLL_SLEEP 4
#define TAIL 8
#define PACE 20
#define IDLE 30
#define PSC "sc0 sc1"
#define DSC "sc0 sc1"
#define REP 4
#define FEW 1
#define EARLY 2
#define POLLWIN 3
#define DIRECT 2
#define MSTAG 80
#define MBASE 2300
#define BOOST 6
#define G1 14
#define DELAYS 60
#define DELAY3 4
#define DELAY1 85
#define DELAY2 40

static_assert((G1 == 15 || G1 == 14) && DIRECT % 2 == 0 && EARLY < 5, "pairs of units");
constexpr int NR = 8;
constexpr int NN = 4096;
constexpr int KS = 512;
constexpr unsigned UNIT = 4096;
constexpr unsigned BLOCK = 32 * UNIT;
constexpr int NSLOT = 24;
constexpr unsigned IN_BYTES = 4096u * 512u * 4u;
constexpr int NWG = 256;
constexpr unsigned OFF_SEQ = 0;
constexpr unsigned OFF_EF = 64u << 10;
constexpr unsigned OFF_SF = 1024u << 10;
constexpr unsigned OFF_RF = 224u << 10;
constexpr unsigned OFF_MC = 236u << 10;
constexpr unsigned OFF_NONE = 240u << 10;
constexpr unsigned OFF_PACE = 244u << 10;
constexpr unsigned OFF_ROLE = 208u << 10;
constexpr unsigned OFF_YF = 256u << 10;
constexpr unsigned OFF_DBG = 512u << 10;
constexpr unsigned OFF_DATA = 1536u << 10;
constexpr unsigned PUNIT = 128 + UNIT;
constexpr unsigned PBLOCK = 32 * PUNIT;
constexpr unsigned P_EXP = 0;
constexpr unsigned P_RLY = 64 * PBLOCK;
constexpr unsigned P_STG = P_RLY + NR * PBLOCK;
constexpr unsigned P_OW = P_STG + NR * NSLOT * BLOCK;
constexpr unsigned P_Y = P_OW + 16 * BLOCK;
constexpr unsigned P_M = P_Y + 128 * 8 * PUNIT;
constexpr unsigned P_BYTES = P_M + 128 * 8 * UNIT;
constexpr unsigned WS_USED = OFF_DATA + 2 * P_BYTES;
constexpr u64 DEADLINE_TICKS = 50000000ull;
constexpr int AUX_SYS = 17;
#define GROUP_ 0xb2
constexpr unsigned GROUP = GROUP_;
constexpr int RSRC_W3 = 0x00020000;

struct Params {
    const float* x;
    const float* w;
    char* mine;
    u64 rank_links;
    char* out;
    char* ws[NR];
    unsigned start[NWG];
};

#define STAMP_V(idx) do {} while (0)
#define STAMP_AT(idx) do {} while (0)
#define STAMP_T0() do {} while (0)

__device__ __forceinline__ u32x4 pack8(float a0, float a1, float a2, float a3,
                                       float a4, float a5, float a6, float a7) {
    f16x8 h = {(_Float16)a0, (_Float16)a1, (_Float16)a2, (_Float16)a3,
               (_Float16)a4, (_Float16)a5, (_Float16)a6, (_Float16)a7};
    return __builtin_bit_cast(u32x4, h);
}
__device__ __forceinline__ unsigned sf_of(unsigned src, unsigned slot, unsigned sub) {
    return OFF_SF + ((src * NSLOT + slot) * 4 + sub) * 128 * REP;
}
__device__ __forceinline__ unsigned poll(gu32* a) {
    return __hip_atomic_load(a, __ATOMIC_RELAXED, __HIP_MEMORY_SCOPE_SYSTEM);
}
__device__ __forceinline__ void post(gu32* a, unsigned v) {
    __hip_atomic_store(a, v, __ATOMIC_RELAXED, __HIP_MEMORY_SCOPE_SYSTEM);
}

__device__ __forceinline__ char* ws_of_lane() {
    unsigned lo = 0, hi = 0;
    asm volatile(R"(
    s_load_dwordx16 s[40:55], %2, %3
    s_waitcnt lgkmcnt(0)
  .irp r, 0, 1, 2, 3, 4, 5, 6, 7
    v_writelane_b32 %0, s[40+2*\r], \r
    v_writelane_b32 %1, s[41+2*\r], \r
  .endr
)"
        : "+v"(lo), "+v"(hi) : "s"(__builtin_amdgcn_kernarg_segment_ptr()), "n"(offsetof(Params, ws))
        : "s40", "s41", "s42", "s43", "s44", "s45", "s46", "s47", "s48", "s49", "s50", "s51", "s52", "s53", "s54", "s55");
    return (char*)((u64)hi << 32 | lo);
}
__device__ __forceinline__ char* ws_of(char* wsl, unsigned r) {
    return (char*)((u64)(unsigned)__builtin_amdgcn_readlane((u64)wsl >> 32, r & (NR - 1)) << 32 |
                   (unsigned)__builtin_amdgcn_readlane((unsigned)(u64)wsl, r & (NR - 1)));
}

__device__ __forceinline__ void conv_load(f32x4 (&v)[8], u64 x, u64 w, unsigned gb, unsigned ks, unsigned vx, unsigned vw) {
    const bool isw = gb & 32;
    const __amdgpu_buffer_rsrc_t src = __builtin_amdgcn_make_buffer_rsrc((void*)(isw ? w : x), 0, IN_BYTES, RSRC_W3);
    const unsigned base = (isw ? ks : gb) << 18 | (isw ? (gb & 31) * 8 : ks) << 6;
    const unsigned odd = isw ? 16384 : 16, two = isw ? 32768 : 65536;
#pragma unroll
    for (int t = 0; t < 8; ++t)
        v[t] = __builtin_bit_cast(f32x4, __builtin_amdgcn_raw_buffer_load_b128(src, isw ? vw : vx, base + (t >> 1) * two + (t & 1) * odd, 0));
}
__host__ __device__ __forceinline__ int slot_of(int v, int q, int t) {
    return ((q ^ t) & 1) && (v & 7) == (q >> 1) ? 20 + (t >> 1) : (v & 7) < 4 ? v & 7 : 3 * (v & 7) - 4;
}
__host__ __device__ __forceinline__ int block_of(int slot, int t) {
    return slot < 4 ? (2 * slot + 1 - (t & 1)) * 4 + (t >> 1) : 24 + (t & 1) * 16 + slot;
}
__device__ __forceinline__ void conv_half(const f32x4 (&v)[8], bool isx, u32x4 (&f)[4]) {
    if (isx) {
#pragma unroll
        for (int e = 0; e < 4; ++e) {
            const f32x4 a = v[2 * e], b = v[2 * e + 1];
            f[e] = pack8(a.x, a.y, a.z, a.w, b.x, b.y, b.z, b.w);
        }
    } else {
#pragma unroll
        for (int e = 0; e < 4; ++e)
            f[e] = pack8(v[0][e], v[1][e], v[2][e], v[3][e], v[4][e], v[5][e], v[6][e], v[7][e]);
    }
}
__device__ __forceinline__ unsigned bfi(unsigned m, unsigned a, unsigned b) { return (a & m) | (b & ~m); }

template <bool Y, int N>
__device__ __forceinline__ u64 pack_store(const f32x4 (&v)[N], const u32x4 (&f)[4], __amdgpu_buffer_rsrc_t dst,
                                          unsigned off, unsigned loff, lds16* lst, int lane) {
    constexpr unsigned HI = Y ? 0x78000000u : 0x60006000u, W = Y ? 4 : 2, CAP = Y ? 2 : 41;
    float mx = 0.f;
#pragma unroll
    for (int t = 0; t < N; t += 4) {
        asm("v_max3_f32 %0, abs(%1), abs(%2), %0\n\tv_max3_f32 %0, abs(%3), abs(%4), %0\n\t"
            "v_max3_f32 %0, abs(%5), abs(%6), %0\n\tv_max3_f32 %0, abs(%7), abs(%8), %0\n\t"
            "v_max3_f32 %0, abs(%9), abs(%10), %0\n\tv_max3_f32 %0, abs(%11), abs(%12), %0\n\t"
            "v_max3_f32 %0, abs(%13), abs(%14), %0\n\tv_max3_f32 %0, abs(%15), abs(%16), %0"
            : "+v"(mx) : "v"(v[t].x), "v"(v[t].y), "v"(v[t].z), "v"(v[t].w), "v"(v[t + 1].x), "v"(v[t + 1].y), "v"(v[t + 1].z),
              "v"(v[t + 1].w), "v"(v[t + 2].x), "v"(v[t + 2].y), "v"(v[t + 2].z), "v"(v[t + 2].w), "v"(v[t + 3].x),
              "v"(v[t + 3].y), "v"(v[t + 3].z), "v"(v[t + 3].w));
    }
    lst[lane] = Y ? 0x400 : 0;
    unsigned c = Y ? __builtin_bit_cast(unsigned, mx) >> 16 : __builtin_bit_cast(unsigned short, (_Float16)mx);
    c = max(c, (unsigned)__builtin_amdgcn_update_dpp(0, c, 0xb1, 0xf, 0xf, true));
    c = max(c, (unsigned)__builtin_amdgcn_update_dpp(0, c, 0x4e, 0xf, 0xf, true));
    c = max(c, (unsigned)__builtin_amdgcn_update_dpp(0, c, 0x141, 0xf, 0xf, true));
    c = max(c, (unsigned)__builtin_amdgcn_update_dpp(0, c, 0x140, 0xf, 0xf, true));
    const unsigned top = max(max((unsigned)__builtin_amdgcn_readlane(c, 0), (unsigned)__builtin_amdgcn_readlane(c, 16)),
                             max((unsigned)__builtin_amdgcn_readlane(c, 32), (unsigned)__builtin_amdgcn_readlane(c, 48)));
    const unsigned B = max(top & 0x7fff, Y ? 2047u : 8191u) - (Y ? 2047u : 8191u);
    const u16x2 B2 = {(unsigned short)(Y ? 0 : B), (unsigned short)B};
    unsigned d[16], e[2];
#pragma unroll
    for (int i = 0; i < 16; ++i) {
        const unsigned t = f[i >> 2][i & 3];
        const u16x2 r = __builtin_bit_cast(u16x2, t) - B2;
        d[i] = __builtin_bit_cast(unsigned, r);
    }
#pragma unroll
    for (int a = 0; a < 2; ++a) {
        unsigned x = d[7 * a] & HI, g = bfi(Y ? 0x40000000u : 0x40004000u, d[14 + a] >> 1, d[14 + a] << (W - 1));
#pragma unroll
        for (int i = 1; i < 7; ++i) x = bfi(HI, d[7 * a + i], x >> W);
        e[a] = bfi(HI << 1, d[14 + a] << 1, x >> (W - 1));
#pragma unroll
        for (int i = 6; i >= 0; --i) { d[7 * a + i] = bfi(HI, g, d[7 * a + i]); g <<= W; }
    }
    auto put = [&]() {
#pragma unroll
        for (int i = 0; i < 3; ++i)
            __builtin_amdgcn_raw_buffer_store_b128(u32x4{d[4 * i], d[4 * i + 1], d[4 * i + 2], d[4 * i + 3]}, dst,
                                                   off + 128 + i * 1024 + lane * 16, 0, AUX_SYS);
        __builtin_amdgcn_raw_buffer_store_b64(u32x2{d[12], d[13]}, dst, off + 128 + 3072 + lane * 8, 0, AUX_SYS);
    };
    put();
    unsigned n = Y || __ballot((((e[0] & ~(e[0] >> 1)) | (e[1] & ~(e[1] >> 1))) & 0x55555555u) != 0) == 0 ? 0 : 64;
    for (;;) {
        const bool two = e[0] == 0;
        const unsigned x = two ? e[1] : e[0];
        const u64 m = __ballot(x != 0);
        const unsigned slot = n + __builtin_amdgcn_mbcnt_hi(m >> 32, __builtin_amdgcn_mbcnt_lo((unsigned)m, 0));
        n += __builtin_popcountll(m);
        if (m == 0 || n > CAP) break;
        const unsigned bit = __builtin_ctz(x | 0x80000000u) & (32 - W), y = x & ~(((1u << W) - 1) << bit);
        lst[x ? slot : 63] = Y ? ((x >> bit) & 15) << 11 | (two ? 512 : 0) | (bit & 16) << 4 | lane << 2 | ((bit >> 2) & 3)
                               : ((x >> bit) & 1) << 11 | (two ? 1024 : 0) | (bit & 8) << 6 | lane << 3 | (bit & 6) | bit >> 4;
        e[1] = two ? y : e[1];
        e[0] = two ? 0 : y;
    }
    const unsigned i2 = Y ? 0 : 2 * (lane & 31), e0 = lst[i2 < n ? i2 : 0], e1 = lst[i2 + 1 < n ? i2 + 1 : 0];
    const unsigned head = n > CAP ? 0x8000 : B;
    if (n > CAP) {
        asm volatile("s_waitcnt vmcnt(0)" ::: "memory");
#pragma unroll
        for (int i = 0; i < 14; ++i) d[i] = f[i >> 2][i & 3];
        put();
        __builtin_amdgcn_raw_buffer_store_b64(u32x2{f[3][2], f[3][3]}, dst, off + 128 + 3584 + lane * 8, 0, AUX_SYS);
    }
    if (!Y) {
        const unsigned two = e0 | (lane == 20 ? head << 16 | (n ? 0x4000 : 0) : e1 << 12), r8 = (lane & 3) * 8;
        const unsigned next = __builtin_amdgcn_update_dpp(0, two, 0x101, 0xf, 0xf, true);
        if (lane == 19 || (lane < 19 && r8 < 24))
            __builtin_amdgcn_raw_buffer_store_b32(two >> r8 | next << (24 - r8), dst, loff + (lane - (lane >> 2)) * 4, 0, AUX_SYS);
    }
    return head | e0 << 16 | (u64)e1 << 32;
}

#define CLOBBER_V \
    "v58", "v59", "v60", "v61", "v62", "v63", "v64", "v65", "v66", "v67", "v68", "v69", "v70", \
    "v71", "v72", "v73", "v74", "v75", "v76", "v77", "v78", "v79", "v80", "v81", "v82", "v83", \
    "v84", "v85", "v86", "v87", "v88", "v89", "v90", "v91", "v92", "v93", "v94", "v95", "v96", \
    "v97", "v98", "v99", "v100", "v101", "v102", "v103", "v104", "v105", "v106", "v107", "v108", "v109", \
    "v110", "v111", "v112", "v113", "v114", "v115", "v116", "v117", "v118", "v119", "v120", "v121", "v122", \
    "v123", "v124", "v125", "v126", "v127", "v128", "v129", "v130", "v131", "v132", "v133", "v134", "v135", \
    "v136", "v137", "v138", "v139", "v140", "v141", "v142", "v143", "v144", "v145", "v146", "v147", "v148", \
    "v149", "v150", "v151", "v152", "v153", "v154", "v155", "v156", "v157", "v158", "v159", "v160", "v161", \
    "v162", "v163", "v164", "v165", "v166", "v167", "v168", "v169", "v170", "v171", "v172", "v173", "v174", \
    "v175", "v176", "v177", "v178", "v179", "v180", "v181", "v182", "v183", "v184", "v185", "v186", "v187", \
    "v188", "v189", "v190", "v191", "v192", "v193", "v194", "v195", "v196", "v197", "v198", "v199", "v200", \
    "v201", "v202", "v203", "v204", "v205", "v206", "v207"
#define CLOBBER_A \
    "a0", "a1", "a2", "a3", "a4", "a5", "a6", "a7", "a8", "a9", "a10", "a11", "a12", "a13", "a14", "a15", "a16", \
    "a17", "a18", "a19", "a20", "a21", "a22", "a23", "a24", "a25", "a26", "a27", "a28", "a29", "a30", "a31", \
    "a32", "a33", "a34", "a35", "a36", "a37", "a38", "a39", "a40", "a41", "a42", "a43", "a44", "a45", "a46", \
    "a47", "a48", "a49", "a50", "a51", "a52", "a53", "a54", "a55", "a56", "a57", "a58", "a59", "a60", "a61", \
    "a62", "a63", "a64", "a65", "a66", "a67", "a68", "a69", "a70", "a71", "a72", "a73", "a74", "a75", "a76", \
    "a77", "a78", "a79", "a80", "a81", "a82", "a83", "a84", "a85", "a86", "a87", "a88", "a89", "a90", "a91", \
    "a92", "a93", "a94", "a95", "a96", "a97", "a98", "a99", "a100", "a101", "a102", "a103", "a104", "a105", \
    "a106", "a107", "a108", "a109", "a110", "a111", "a112", "a113", "a114", "a115", "a116", "a117", "a118", \
    "a119", "a120", "a121", "a122", "a123", "a124", "a125", "a126", "a127"

#define SILU(r, l, var) asm volatile("\n.set g_r, " #r R"(
  .irp i, 0, 2, 4, 6, 8, 10, 12, 14
    v_pk_mul_f32 v[64+\i:65+\i], v[g_r+\i:g_r+\i+1], %1
  .endr
  .irp i, 0, 1, 2, 3, 4, 5, 6, 7, 8, 9, 10, 11, 12, 13, 14, 15
    v_med3_f32 v[64+\i], v[64+\i], -%2, %2
  .endr
  .irp i, 0, 1, 2, 3, 4, 5, 6, 7, 8, 9, 10, 11, 12, 13, 14, 15
    v_exp_f32 v[64+\i], v[64+\i]
  .endr
  .irp i, 0, 2, 4, 6, 8, 10, 12, 14
    v_pk_add_f32 v[64+\i:65+\i], v[64+\i:65+\i], %3
  .endr
  .irp i, 0, 1, 2, 3, 4, 5, 6, 7, 8, 9, 10, 11, 12, 13, 14, 15
    v_rcp_f32 v[64+\i], v[64+\i]
  .endr
  .irp i, 0, 2, 4, 6, 8, 10, 12, 14
    v_pk_mul_f32 v[g_r+\i:g_r+\i+1], v[g_r+\i:g_r+\i+1], v[64+\i:65+\i]
  .endr
)" : "+{v[" #r ":" #l "]}"(var) : "s"(0xbfb8aa3bbfb8aa3bull), "s"(0x42ad1f97), "s"(0x3f8000003f800000ull) \
    : "v64", "v65", "v66", "v67", "v68", "v69", "v70", "v71", "v72", "v73", "v74", "v75", "v76", "v77", "v78", "v79")

__global__ void __launch_bounds__(256, 1) fused_gemm_silu(Params p) {
    __shared__ f32x4 red[4 * 32 * 64 + 33];
    const int b = blockIdx.x;
    u64x4 arg;
    u64 now;
    unsigned start, more;
    asm volatile("s_load_dwordx8 %0, %4, 0\n\ts_load_dword %1, %4, %5 offset:%6\n\ts_load_dword %2, %4, 64\n\ts_memrealtime %3"
                 : "=&s"(arg), "=&s"(start), "=&s"(more), "=&s"(now)
                 : "s"(__builtin_amdgcn_kernarg_segment_ptr()), "s"(b * 4), "n"(offsetof(Params, start)));
    const int lane = threadIdx.x & 63;
    const int wave = __builtin_amdgcn_readfirstlane(threadIdx.x >> 6);
    const int xcd = b & 7, idx = b >> 3;
    if (threadIdx.x == 0) red[4 * 32 * 64] = f32x4{0.f, 0.f, 0.f, 0.f};
    const unsigned first = (0x249f249ffffffffull >> ((b >> 2) + ((b + 80) >> 8) * 4)) & 1;
    unsigned turn = (((wave & 2) + ((b >> 1) & 1)) * first + (wave + DELAY3) * (1 - first)) * DELAY1;
    unsigned kso = (b & 3) * 8 + wave, kss = (wave | 2) * 8 + 6 + (wave >> 1), sqo = OFF_SEQ + (b * 4 + wave) * 64;
    unsigned vx = (lane & 31) << 11 | (lane >> 5) << 5, vw = (lane & 31) << 4 | (lane >> 5) << 17;
    asm volatile("" : "+v"(sqo), "+v"(vx), "+v"(vw));
    asm volatile("s_waitcnt lgkmcnt(0)" : "+s"(arg), "+s"(start), "+s"(more), "+s"(now), "+s"(turn), "+s"(kso), "+s"(kss) :: "memory");
    const u64 px = arg[0], pw = arg[1], pm = arg[2], rl = arg[3];
    const unsigned t0 = now;
    const int rank = rl & (NR - 1), ri = rank >> 1, rj = rank & 1;
    const unsigned links = rl >> 32;
    char* const myws = (char*)pm;
    unsigned* const cnt = (unsigned*)&red[4 * 32 * 64];

    const bool copier = idx >= 28;

    gu32* const seq = (gu32*)(myws + sqo);
    const unsigned last = poll(seq);

#define GET_CALL() const unsigned call = __builtin_amdgcn_readfirstlane(last) + 1u
    const __amdgpu_buffer_rsrc_t buf = __builtin_amdgcn_make_buffer_rsrc((void*)myws, 0, WS_USED, RSRC_W3);

    if (copier) {
        unsigned role = ((links >> 4 * (__builtin_amdgcn_s_getreg(20 | 3 << 11) & 7)) & 7) * 16 + (idx & 3) * 4 + wave;
        char* const wsl = ws_of_lane();
        if (role < 16) __builtin_amdgcn_s_sleep(127);
        for (unsigned dead = __builtin_amdgcn_readfirstlane(role < 16);;) {
        asm volatile("" : "+s"(dead));
        if (dead) {
            GET_CALL();
            if (lane == 0) post(seq, call);
            gu32* const taken = (gu32*)(myws + OFF_ROLE);
            const u64 m0 = __ballot(lane >= 16 && poll(taken + lane * 32) < call);
            const u64 m1 = __ballot(poll(taken + (64 + lane) * 32) < call);
            if ((m0 | m1) == 0) return;
            role = m0 ? __builtin_ctzll(m0) : 64 + __builtin_ctzll(m1);
        }
        const int link = role >> 4, v = role & 15;
        const unsigned q = (unsigned)(rank ^ link) & (NR - 1);
        const bool same = (link & 1) == 0;
        const int sub = v & 3, a = v >> 2;
        auto k1 = [&](int u) { return 8 * (u >> 2) + 4 * (u & 1) + (u & 2) + (v >> 3); };
        const int slot1 = slot_of(v, q, rank), gb1 = block_of(slot1, rank);
        const int sp = 2 * a + 1 - rj, cq = 12 + (q >> 1);
        const int uw = G1 + (same ? 24 - DIRECT : a == ri ? 8 : 8 + DIRECT), ux = uw + 8, uy = uw + 16, nu = same ? uy : uy + 16;
        auto ycol = [&](int n) { return n < 8 ? 3 * (n & 3) + (n >> 2) : n < 12 ? 12 + ((n + (q >> 1)) & 3) : 3 * n - 34; };
        unsigned poff = OFF_NONE, gate = 0;
        bool used = true;
        const unsigned efw = OFF_EF + (q * 64 + 32 + rj * 16) * 256 + (32 + sub) * 4;
        const unsigned yf = OFF_YF + (q * 8 + (v >> 1)) * 256;
        if (lane < 16) {
            const int u = G1 == 15 ? lane - (lane > 12) : lane - 2 * (lane > 11);
            gate = u & ~1;
            poff = OFF_EF + (q * 64 + gb1) * 256 + (G1 == 15 ? (lane == 12 ? 38 + (v >> 3) : k1(u))
                                                             : (lane >> 1 == 5 ? 16 + 2 * lane + (v >> 3) : k1(u))) * 4;
        } else if (lane == 16) { poff = efw + (3 * a + 1) * 256; gate = G1; }
        else if (lane == 17) { poff = efw + (12 + a) * 256; gate = G1 + 8; }
        else if (lane == 18 && same) { poff = OFF_RF + (q * 8 + sp) * 128 + (sub & 1) * 4; gate = G1 + 16; }
        else if (lane == 19) { poff = efw + (3 * a + 2) * 256; gate = uw; }
        else if (lane == 20) { poff = OFF_EF + (q * 64 + rank * 4 + a) * 256 + (32 + sub) * 4; gate = ux; }
        else if (lane == 21 && !same) { poff = OFF_MC; gate = uy; }
        else if (lane >= 32 && lane < 48 && !same) {
            poff = yf + ycol(lane - 32) * 16; gate = uy + lane - 32;
        } else used = false;
        if (lane == 63) gate = nu;
        char* const rfa = wsl + OFF_RF + (rank * 8 + q) * 128 + (v >> 3) * 4;
        const unsigned yo = (lane >> 5) * 4 * NN * 4 + (lane & 31) * 16;
        const unsigned tab = (unsigned)(size_t)(__attribute__((address_space(3))) f32x4*)red + wave * 32768;
#pragma unroll
        for (int i = 0; i < 5; ++i) red[wave * 2048 + i * 64 + lane] = f32x4{0.f, 0.f, 0.f, 0.f};

        unsigned so, dst, fo, fv;
        {
            const int u = lane;
            unsigned slot, src = q, k = 4 * ((u - (u < (same ? uw : G1 + 8) ? G1 : uw)) & 7) + sub;
            if (u < G1) { so = P_EXP + gb1 * PBLOCK; slot = slot1; k = k1(u); }
            else if (u >= ux) { so = P_EXP + (rank * 4 + a) * PBLOCK; slot = 4 + a; }
            else if (u >= G1 + 16 && u < uw) { src = sp; slot = 8 + cq; so = P_RLY + sp * PBLOCK; }
            else {
                slot = u < G1 + 8 ? 9 + 3 * a : u >= uw ? 10 + 3 * a : 20 + a;
                so = P_EXP + (24 + rj * 16 + slot) * PBLOCK;
            }
            so += k * PUNIT + 128 + ((k >> 2) & 1);
            dst = P_STG + (src * NSLOT + slot) * BLOCK + k * UNIT;
            fo = sf_of(src, slot, k & 3);
            fv = u == 13 || u == (G1 == 15 ? 14 : 11) || u == uw - 1 ? 32 : k + 1;
            if (!same && u >= G1 + 8 && u < uw) { fo += 4; fv = u == uw - 1 ? 32 : 0; }
            if (u >= uy) {
                const int n = ycol((u - uy) & 15);
                so = P_Y + (((ri * 2 + (v >> 3)) * 16 + n) * 8 + (v & 7)) * PUNIT + 129;
                dst = (((q >> 1) * 128 + v * 8) * NN + (q & 1) * 2048 + n * 128) * 4;
                fo = u < nu ? yf + n * 16 + 4 + (v & 1) * 6 : OFF_ROLE + role * 128;
                fv = P_M + ((((q >> 1) * 2 + (v >> 3)) * 16 + n) * 8 + (v & 7)) * UNIT;
            }
        }
        const int li = lane & 31;
        const int skip = !same && a == ri ? DIRECT : 0, done = lane + 1 + (lane >= G1 + 7 ? skip : 0);
        const unsigned inc = lane >= uy ? 0 : lane == uy - 1 ? 65 - done : lane == G1 + 7 ? 1 + skip : 1;
        unsigned lim = 16 * done + BOOST - 1024, unit = !used ? 0 : lane == 21 ? 512 : 64;
        unsigned lk = ((lane & 32) * 2 + (li < 20 ? 3 * li & ~3 : li == 31 ? 60 : 0)) | (li < 20 ? (24 * li & 31) << 8 : 0) |
                            (lane >= 32 && lane < 63 ? 12 << 13 : 0) | (lane == 63 ? 0 : 2 + 2 * (lane & 1)) << 18 | inc << 24;
        asm volatile("" : "+v"(so), "+v"(dst), "+v"(fo), "+v"(fv), "+v"(poff), "+v"(gate), "+v"(lk), "+v"(lim), "+v"(unit));
        GET_CALL();
        if (lane == 0) post(seq, call);
        const unsigned par = OFF_DATA + (call & 1) * P_BYTES;
        const unsigned thr = lane == 63 ? ~0u : call * unit + (unit & 64) / 2;
        so += par;
        dst += lane < uy ? par : 0;
        fv = lane == 63 ? call : fv + (lane < uy ? call * 64 : par);
        lim += call * 1024;
        asm volatile(R"(
.macro c_load r, i
  .if \i < 3
    global_load_dwordx4 v[\r+4*\i:\r+4*\i+3], %[l16], s[44:45] offset:1024*\i sc0 sc1
  .elseif \i == 3
    global_load_dwordx2 v[\r+12:\r+13], v58, s[44:45] offset:3072 sc0 sc1
  .else
    s_bitcmp1_b32 s43, 0
    s_cbranch_scc1 Lcc\@
    global_load_dword v[\r+14], v59, s[44:45] offset:-128 sc0 sc1
Lcc\@:
  .endif
.endm
.macro c_early r
    v_readlane_b32 s43, %[so], s41
    s_add_u32 s41, s41, 1
    s_andn2_b32 s48, s43, 1
    s_add_u32 s44, s58, s48
    s_addc_u32 s45, s59, 0
  .irp i, 0, 1, 2, 3, 4
   .if \i < %[early]
    c_load \r, \i
   .endif
  .endr
.endm
.macro c_late r
  .irp i, 0, 1, 2, 3, 4
   .if \i >= %[early]
    c_load \r, \i
   .endif
  .endr
.endm
.macro c_sum r, skip
    s_cmp_gt_u32 s41, %[uy]
    s_cbranch_scc0 \skip
    s_sub_u32 s48, s41, 1
    v_readlane_b32 s49, %[fo], s48
    v_readlane_b32 s48, %[fv], s48
    s_add_u32 s46, s60, s48
    s_addc_u32 s47, s61, 0
  .irp i, 0, 1, 2, 3
    global_load_dwordx4 v[56+\r+4*\i:59+\r+4*\i], %[l16], s[46:47] offset:1024*\i sc0 sc1
  .endr
    s_add_u32 s46, s60, s49
    s_addc_u32 s47, s61, 0
    global_load_ushort v[\r+14], v63, s[46:47] sc0 sc1
.endm
.macro c_flag n
    s_mov_b32 s68, 0
Lc6\@:
    s_cmp_lt_u32 s74, \n
    s_cbranch_scc0 Lc7\@
    s_lshl_b64 exec, 1, s74
  .irp i, 0, 1, 2, 3
   .if \i < )" STR(REP) R"(
    global_store_dword %[fo], %[fv], s[60:61] offset:128*\i sc0 sc1
   .endif
  .endr
    global_atomic_add v153, %[pace], v152, s[60:61] sc0 sc1
    s_add_u32 s74, s74, 1
    s_add_u32 s68, s68, 1+)" STR(REP) R"(
    s_cmp_eq_u32 s74, %[ur]
    s_cbranch_scc0 Lc6\@
    s_mov_b32 exec_lo, %[twins]
    global_store_dword %[rfa], %[thr], off sc0 sc1
    s_branch Lc6\@
Lc7\@:
    s_mov_b64 exec, -1
.endm
.macro c_take
    v_cmp_lt_u32 vcc, v112, %[thr]
    s_ff1_i32_b64 s48, vcc
    v_readlane_b32 s42, %[gate], s48
.endm
.macro c_poll
    global_load_dword v112, %[poff], s[60:61] sc0 sc1
.endm
.macro c_stamp n
)"
R"(
.endm
.macro c_near skip
    s_add_u32 s48, s41, %[win]
    s_min_u32 s48, s48, %[nu]
    s_cmp_lt_u32 s42, s48
    s_cbranch_scc0 \skip
    c_poll
    s_add_u32 s68, s68, 1
.endm
.macro c_step r, o
    s_sub_u32 s48, s41, s40
    s_cmp_eq_u32 s48, 2
    s_cbranch_scc0 Lcd\r\()_%=
    s_cmp_gt_u32 s41, %[uy]
    s_cbranch_scc1 Lck\r\()_%=
    s_cmp_lg_u32 s73, 0
    s_cbranch_scc1 Lc3\r\()_%=
    s_waitcnt vmcnt(%[early])
    c_late \o
Lca\r\()_%=:
    s_sub_u32 s48, s40, s73
    c_flag s48
Lcl\r\()_%=:
    c_take
Lce\r\()_%=:
    v_readlane_b32 s49, %[so], s40
    v_mov_b32_dpp v113, v[\r+14] quad_perm:[1,2,3,3] row_mask:0xf bank_mask:0xf
    s_cmp_lt_i32 s40, %[ur]
    s_cbranch_scc1 Lcr\r\()_%=
    s_bitcmp1_b32 s49, 0
    s_cbranch_scc1 Lcv\r\()_%=
Lcf\r\()_%=:
    v_alignbit_b32 v62, v113, v[\r+14], v60
    v_alignbit_b32 v[\r+14], v113, v[\r+14], v60
    s_nop 1
    v_permlane32_swap_b32 v[\r+14], v62
    s_nop 0
Lcx\r\()_%=:
    v_readlane_b32 s56, v[\r+14], 31
    v_lshrrev_b32 v[\r+14], v61, v[\r+14]
    s_and_b32 s48, s56, 0x4000
    s_lshr_b32 s56, s56, 16
    v_and_b32 v113, 0x7ff, v[\r+14]
    v_lshl_or_b32 v[\r+14], v[\r+14], 2, s48
Lcx2\r\()_%=:
    s_bitcmp1_b32 s56, 15
    s_cbranch_scc1 Lcw\r\()_%=
    s_mul_i32 s56, s56, s69
    v_mad_u32_u24 v113, v113, s66, v117
    v_and_b32 v114, s54, v[\r]
    ds_write_b16 v113, v[\r+14]
    ds_read_b128 v[96:99], %[tl]
    ds_read_b128 v[100:103], %[tl] offset:1024
    ds_read_b128 v[104:107], %[tl] offset:2048
    ds_read_b128 v[108:111], %[tl] offset:3072
    ds_write_b16 v113, v116
    v_and_b32 v115, s54, v[\r+7]
  .irp i, 1, 2, 3, 4, 5, 6
    v_lshrrev_b32 v114, s66, v114
    v_lshrrev_b32 v115, s66, v115
    v_bfi_b32 v114, s54, v[\r+\i], v114
    v_bfi_b32 v115, s54, v[\r+7+\i], v115
  .endr
    v_lshlrev_b32 v[\r+14], 1, v114
    v_lshlrev_b32 v[\r+15], 1, v115
    v_lshrrev_b32 v114, s67, v114
    v_lshrrev_b32 v115, s67, v115
    v_bfi_b32 v[\r+14], s55, v[\r+14], v114
    v_bfi_b32 v[\r+15], s55, v[\r+15], v115
    s_waitcnt lgkmcnt(0)
  .irp i, 0, 1, 2, 3, 4, 5, 6
    v_bfi_b32 v[\r+\i], s54, v[96+\i], v[\r+\i]
    v_bfi_b32 v[\r+7+\i], s54, v[104+\i], v[\r+7+\i]
  .endr
    v_bfi_b32 v[\r+14], s54, v103, v[\r+14]
    v_bfi_b32 v[\r+15], s54, v111, v[\r+15]
  .irp i, 0, 1, 2, 3, 4, 5, 6, 7, 8, 9, 10, 11, 12, 13, 14, 15
    v_pk_add_u16 v[\r+\i], v[\r+\i], s56
  .endr
Lcs\r\()_%=:
    v_readlane_b32 s48, %[dst], s40
    s_cmp_ge_u32 s40, %[uy]
    s_cbranch_scc1 Lcy\r\()_%=
    s_add_u32 s46, s60, s48
    s_addc_u32 s47, s61, 0
    global_store_dwordx4 %[l16], v[\r:\r+3], s[46:47] sc0 sc1
    global_store_dwordx4 %[l16], v[\r+4:\r+7], s[46:47] offset:1024 sc0 sc1
    global_store_dwordx4 %[l16], v[\r+8:\r+11], s[46:47] offset:2048 sc0 sc1
    global_store_dwordx4 %[l16], v[\r+12:\r+15], s[46:47] offset:3072 sc0 sc1
    c_near Lcp\r\()_%=
Lcp\r\()_%=:
    s_add_u32 s40, s40, 1
    s_cmp_eq_u32 s40, %[uy]
    s_cbranch_scc1 Lcm\r\()_%=
Lco\r\()_%=:
    s_cmp_gt_u32 s41, s40
    s_cselect_b32 s48, s42, 0
    s_cmp_gt_u32 s48, s41
    s_cbranch_scc0 Lcq\r\()_%=
    c_early \r
    v_cmp_ne_u32 vcc, 0, v153
    s_flbit_i32_b64 s48, vcc
    v_cmp_ge_u32 vcc, v153, %[lim]
    s_not_b32 s48, s48
    s_bitcmp1_b64 vcc, s48
    s_cselect_b32 s73, 1, 0
    c_sum \r, Lc1\r\()_%=
Lc2\r\()_%=:
    c_late \r
Lcq\r\()_%=:
    c_stamp s40
.endm
.macro c_rare r, o
Lc1\r\()_%=:
    s_cmp_lg_u32 s73, 0
    s_cbranch_scc0 Lcq\r\()_%=
    s_bitcmp0_b32 s43, 0
    s_addc_u32 s68, s68, 8
    s_branch Lc2\r\()_%=
Lc3\r\()_%=:
  .irp n, 20, 19, 18, 17, 16, 15, 14, 13, 12, 11, 10, 9
   .if \n <= 12+2*)" STR(REP) R"(
    s_waitcnt vmcnt(\n)
    s_cmp_ge_u32 s68, \n
    s_cbranch_scc1 Lca\r\()_%=
   .endif
  .endr
    s_waitcnt vmcnt(8)
    s_branch Lca\r\()_%=
Lck\r\()_%=:
    s_waitcnt vmcnt(13)
    s_cmp_gt_u32 s40, %[uy]
    s_cbranch_scc1 Lcl\r\()_%=
    s_waitcnt vmcnt(9)
    s_mov_b32 s73, 0
    s_branch Lca\r\()_%=
Lcd\r\()_%=:
    s_waitcnt vmcnt(4)
    s_cmp_gt_u32 s40, %[uy]
    s_cbranch_scc1 Lcb\r\()_%=
    s_waitcnt vmcnt(0)
    c_flag s40
    c_stamp 58
Lcb\r\()_%=:
    c_take
    s_mov_b32 s68, 0
    s_cmp_gt_u32 s41, s40
    s_cbranch_scc0 Lcg\r\()_%=
    s_cmp_gt_u32 s42, s41
    s_cbranch_scc0 Lce\r\()_%=
    c_early \o
    c_late \o
    c_sum \o, Lce\r\()_%=
    s_branch Lce\r\()_%=
Lcg\r\()_%=:
    s_cmp_eq_u32 s40, %[nu]
    s_cbranch_scc1 Lcz_%=
    s_memrealtime s[50:51]
Lch\r\()_%=:
    s_cmp_gt_u32 s42, s40
    s_cbranch_scc1 Lci\r\()_%=
    s_sleep 2
    c_poll
    s_memrealtime s[52:53]
    s_waitcnt vmcnt(0) lgkmcnt(0)
    c_take
    s_sub_u32 s48, s52, s50
    s_cmp_gt_u32 s48, %[ticks]
    s_cbranch_scc0 Lch\r\()_%=
    s_mov_b32 %[dead], 1
    s_branch Lcz_%=
Lci\r\()_%=:
    c_early \r
    c_late \r
)"
R"(
    c_sum \r, Lct\r\()_%=
Lct\r\()_%=:
    s_cmp_gt_u32 s42, s41
    s_cbranch_scc1 Lcj\r\()_%=
    s_waitcnt vmcnt(0) lgkmcnt(0)
    s_branch Lce\r\()_%=
Lcj\r\()_%=:
    c_early \o
    c_sum \o, Lcu\r\()_%=
    c_late \o
    s_waitcnt vmcnt(9) lgkmcnt(0)
    s_branch Lce\r\()_%=
Lcu\r\()_%=:
    s_waitcnt vmcnt(%[early]) lgkmcnt(0)
    c_late \o
    s_branch Lce\r\()_%=
Lcy\r\()_%=:
    s_add_u32 s46, s62, s48
    s_addc_u32 s47, s63, 0
    c_near Lcn\r\()_%=
Lcn\r\()_%=:
  .irp i, 0, 2, 4, 6, 8, 10, 12, 14
    v_pk_add_f32 v[\r+\i:\r+\i+1], v[\r+\i:\r+\i+1], v[56+\r+\i:57+\r+\i]
  .endr
  .irp i, 0, 2, 4, 6, 8, 10, 12, 14
    v_pk_mul_f32 v[96+\i:97+\i], v[\r+\i:\r+\i+1], s[70:71]
  .endr
  .irp i, 0, 1, 2, 3, 4, 5, 6, 7, 8, 9, 10, 11, 12, 13, 14, 15
    v_med3_f32 v[96+\i], v[96+\i], -s72, s72
  .endr
  .irp i, 0, 1, 2, 3, 4, 5, 6, 7, 8, 9, 10, 11, 12, 13, 14, 15
    v_exp_f32 v[96+\i], v[96+\i]
  .endr
  .irp i, 0, 2, 4, 6, 8, 10, 12, 14
    v_pk_add_f32 v[96+\i:97+\i], v[96+\i:97+\i], v[118:119]
  .endr
  .irp i, 0, 4, 8, 12
    v_rcp_f32 v[96+\i], v[96+\i]
    v_rcp_f32 v[97+\i], v[97+\i]
    v_rcp_f32 v[98+\i], v[98+\i]
    v_rcp_f32 v[99+\i], v[99+\i]
    v_pk_mul_f32 v[\r+\i:\r+\i+1], v[\r+\i:\r+\i+1], v[96+\i:97+\i]
    v_pk_mul_f32 v[\r+\i+2:\r+\i+3], v[\r+\i+2:\r+\i+3], v[98+\i:99+\i]
    global_store_dwordx4 %[yo], v[\r+\i:\r+\i+3], s[46:47] sc0 sc1
    s_add_u32 s46, s46, 16384
    s_addc_u32 s47, s47, 0
  .endr
    s_branch Lcp\r\()_%=
Lcm\r\()_%=:
    s_mov_b32 s54, 0x78000000
    s_brev_b32 s55, 1
    s_mov_b32 s66, 4
    s_mov_b32 s67, 3
    s_mov_b32 s69, 0x10000
    v_add_u32 v117, 2, %[tab]
    s_branch Lco\r\()_%=
Lcr\r\()_%=:
    s_andn2_b32 s48, s49, 1
    s_add_u32 s48, s48, %[rly]
    s_add_u32 s64, s60, s48
    s_addc_u32 s65, s61, 0
    global_store_dwordx4 %[l16], v[\r:\r+3], s[64:65] sc0 sc1
    global_store_dwordx4 %[l16], v[\r+4:\r+7], s[64:65] offset:1024 sc0 sc1
    global_store_dwordx4 %[l16], v[\r+8:\r+11], s[64:65] offset:2048 sc0 sc1
    global_store_dwordx2 v58, v[\r+12:\r+13], s[64:65] offset:3072 sc0 sc1
    s_bitcmp1_b32 s49, 0
    s_cbranch_scc1 Lcv\r\()_%=
    global_store_dword v59, v[\r+14], s[64:65] offset:-128 sc0 sc1
    s_branch Lcf\r\()_%=
Lcv\r\()_%=:
    s_cmp_ge_u32 s40, %[uy]
    s_cbranch_scc1 Lcv2\r\()_%=
    v_mov_b32 v[\r+14], v62
    s_branch Lcx\r\()_%=
Lcv2\r\()_%=:
    v_readlane_b32 s56, v[\r+14], 63
    v_writelane_b32 v[\r+14], s57, 63
    v_and_b32 v113, 0x7ff, v[\r+14]
    s_branch Lcx2\r\()_%=
Lcw\r\()_%=:
    s_andn2_b32 s48, s49, 1
    s_add_u32 s44, s58, s48
    s_addc_u32 s45, s59, 0
    global_load_dwordx2 v[\r+14:\r+15], v58, s[44:45] offset:3584 sc0 sc1
    s_cmp_lt_i32 s40, %[ur]
    s_waitcnt vmcnt(0)
    s_cbranch_scc0 Lcs\r\()_%=
    global_store_dwordx2 v58, v[\r+14:\r+15], s[64:65] offset:3584 sc0 sc1
    s_branch Lcs\r\()_%=
.endm
    s_mov_b64 s[58:59], %[qb]
    s_mov_b64 s[60:61], %[mb]
    s_lshl_b64 exec, 1, 63
    global_atomic_umax v154, %[fo], %[fv], s[60:61] sc0 sc1
    s_mov_b64 exec, -1
    c_poll
    s_mov_b64 s[62:63], %[ob]
    s_mov_b32 s54, 0x60006000
    s_mov_b32 s55, 0x80008000
    s_movk_i32 s57, 0x400
    s_mov_b32 s66, 2
    s_mov_b32 s67, 1
    v_lshrrev_b32 v58, 1, %[l16]
    v_and_b32 v59, 0xff, %[lk]
    v_bfe_u32 v60, %[lk], 8, 5
    v_bfe_u32 v61, %[lk], 13, 5
    v_bfe_u32 v63, %[lk], 18, 6
    v_lshrrev_b32 v152, 24, %[lk]
    v_mov_b32 v153, 0
    s_mov_b32 s69, 0x10001
    s_mov_b32 s70, 0xbfb8aa3b
    s_mov_b32 s71, s70
    s_mov_b32 s72, 0x42ad1f97
    v_mov_b32 v117, %[tab]
    v_mov_b32 v118, 1.0
    v_mov_b32 v119, 1.0
    s_mov_b32 s40, 0
    s_mov_b32 s74, 0
    s_mov_b32 s41, 0
    v_mov_b32 v116, 0
    s_waitcnt vmcnt(0)
    v_cmp_ge_u32 vcc, v154, %[fv]
    s_lshr_b32 %[dead], vcc_hi, 30
    s_bitcmp1_b32 vcc_hi, 31
    s_cbranch_scc1 Lcz_%=
    s_mov_b32 %[dead], 0
    c_stamp 62
Lc0_%=:
    c_step 64, 80
    c_step 80, 64
    s_branch Lc0_%=
    c_rare 64, 80
    c_rare 80, 64
Lcz_%=:
    s_waitcnt vmcnt(0) lgkmcnt(0)
    c_stamp 59
.purgem c_load
.purgem c_early
.purgem c_late
.purgem c_flag
.purgem c_take
.purgem c_poll
.purgem c_stamp
.purgem c_near
.purgem c_sum
.purgem c_step
.purgem c_rare
)"
            : [dead] "=&s"(dead)
            : [so] "v"(so), [dst] "v"(dst), [fo] "v"(fo), [fv] "v"(fv), [poff] "v"(poff), [gate] "v"(gate), [thr] "v"(thr),
              [l16] "v"(lane * 16), [lk] "v"(lk), [lim] "v"(lim), [pace] "v"(OFF_PACE + link * 128), [twins] "s"((0x55u << rj) & ~(1u << rank)), [yo] "v"(yo), [rfa] "v"(rfa),
              [tl] "v"(tab + lane * 16), [tab] "s"(tab), [qb] "s"(ws_of(wsl, q)), [mb] "s"(myws), [ob] "s"(p.out),
              [nu] "s"(nu), [uy] "s"(same ? 64 : uy), [ur] "s"(slot1 >= 20 ? 16 - 2 * DIRECT : -1),
              [rly] "s"(P_RLY + q * PBLOCK - P_EXP - gb1 * PBLOCK), [early] "n"(EARLY), [win] "n"(POLLWIN), [ticks] "n"(DEADLINE_TICKS)
            : "memory", "vcc", "scc", "s40", "s41", "s42", "s43", "s44", "s45", "s46", "s47", "s48", "s49", "s50", "s51", "s52",
              "s53", "s54", "s55", "s56", "s57", "s58", "s59", "s60", "s61", "s62", "s63", "s64", "s65", "s66", "s67", "s68",
              "s69", "s70", "s71", "s72", "s73", "s74", CLOBBER_V);
        if (dead >= 2) continue;
        if (dead && !same && lane == nu - 1) *(float*)(p.out + dst) = __builtin_nanf("");
        STAMP_AT(11);
        STAMP_T0();
        return;
        }
    }

    lds16* const lst = (lds16*)&red[4 * 32 * 64 + 1] + wave * 64;
    if (wave < 2 * G1 - 28) start = start >> 8 & 63 | start & ~255u;
    const bool send = start & 64;
    const unsigned nap = send ? (wave < 2 ? DELAYS : 0) : turn;
    int g = start & 63;
    if (nap) {
        __builtin_amdgcn_s_barrier();
        while ((unsigned)__builtin_amdgcn_s_memrealtime() - t0 < nap) __builtin_amdgcn_s_sleep(2);
    }
    f32x4 v0[8];
    STAMP_V(0);
    conv_load(v0, px, pw, g, send ? kss : kso, vx, vw);
    __builtin_amdgcn_sched_barrier(0);
    if (!nap) __builtin_amdgcn_s_barrier();
    char* const wsl = ws_of_lane();
    const int gb0 = (start >> 8) & 63, to = (start >> 16) & (NR - 1), sslot = start >> 24;
    STAMP_V(1);
    for (int job = send ? -1 : 0;;) {
        const int sub = b & 3, part = (b >> 2) & 3, k = 16 * (job & 1) + 4 * wave + sub;
        const bool isx = g < 32, late = job >= 2;
        const int other = (g & 3) * 2 + 1 - ((g >> 2) & 1);
        const bool efw = lane < NR && (lane & 1) == ((g >> 4) & 1);
        const bool efl = isx ? lane == g >> 2 || (lane == other && !late) : efw || lane == rank;
        const bool efu = !late && first && lane != rank && (isx ? lane == other : efw && ((g & 15) < 12 || lane >> 1 == part));
        const unsigned efo = OFF_EF + (rank * 64 + g) * 256;
        const int mine = !isx ? (((g >> 4) & 1) == rj ? rank * NSLOT + 8 : NR * NSLOT) + (g & 15)
                              : late ? rank * NSLOT + 4 + part : other == rank ? rank * NSLOT + (g >> 3) : -1;
        u32x4 f[4];
        STAMP_V(6 + 5 * job);
        conv_half(v0, isx, f);
        GET_CALL();
        STAMP_V(7 + 5 * job);
        const unsigned par = OFF_DATA + (call & 1) * P_BYTES;
        if (job < 0) {
            const __amdgpu_buffer_rsrc_t dst = __builtin_amdgcn_make_buffer_rsrc(ws_of(wsl, to), 0, WS_USED, RSRC_W3);
#pragma unroll
            for (int e = 0; e < 4; ++e)
                __builtin_amdgcn_raw_buffer_store_b128(f[e], dst, par + P_STG + (rank * NSLOT + sslot) * BLOCK +
                                                       (26 + wave + (wave & 2)) * UNIT + e * 1024 + lane * 16, 0, AUX_SYS);
            STAMP_V(3);
            asm volatile("s_waitcnt vmcnt(0)" ::: "memory");
            STAMP_V(4);
            if (lane == to) post((gu32*)(wsl + efo + (36 + wave) * 4), call * 64 + 32);
            while ((unsigned)__builtin_amdgcn_s_memrealtime() - t0 < turn) __builtin_amdgcn_s_sleep(2);
        } else {
            if (mine >= 0) {
#pragma unroll
                for (int e = 0; e < 4; ++e)
                    __builtin_amdgcn_raw_buffer_store_b128(f[e], buf, par + P_STG + mine * BLOCK + k * UNIT +
                                                           e * 1024 + lane * 16, 0, AUX_SYS);
            }
            if (!(isx && late))
                pack_store<false>(v0, f, buf, par + P_EXP + g * PBLOCK + k * PUNIT,
                                  par + P_EXP + g * PBLOCK + (k & ~4) * PUNIT + (k & 4) * 16, lst, lane);
            STAMP_V(8 + 5 * job);
            asm volatile("s_waitcnt vmcnt(0)" ::: "memory");
            STAMP_V(9 + 5 * job);
            if (efu) post((gu32*)(wsl + efo + k * 4), call * 64 + 32);
            if (job & 1) {
                unsigned n = 0;
                if (lane == 0) n = __hip_atomic_fetch_add(cnt + 2, 1u, __ATOMIC_RELAXED, __HIP_MEMORY_SCOPE_WORKGROUP);
                if ((__builtin_amdgcn_readfirstlane(n) & 3) == 3 && efl) post((gu32*)(wsl + efo + (32 + sub) * 4), call * 64 + 32);
                if (late || idx < 24) break;
                __syncthreads();
            } else __builtin_amdgcn_s_sleep(DELAY2);
        }
        ++job;
        g = job >= 2 ? (b & 16 ? rank * 4 : 44 + rj * 16) + part : gb0;
        conv_load(v0, px, pw, g, kso + 4 * (job & 1), vx, vw);
    }

    GET_CALL();
    if (lane == 0) post(seq, call);
    const unsigned fb = call * 64;
    const unsigned par = OFF_DATA + (call & 1) * P_BYTES;
    const int rh = (xcd & 1) * 4 + (idx & 3);
    const int njob = idx < 4 || idx >= 12 ? 2 : 1;
    const u32x4 bufw = {(unsigned)(size_t)myws, (unsigned)((size_t)myws >> 32) & 0xffff, WS_USED, RSRC_W3};
    unsigned dead = 0;
    for (int job = 0; job < njob; ++job) {
        const bool own = job == 1 && idx < 24, slab = job == 0 && idx >= 12;
        const int cb = idx >= 24 || (idx < 4 && job == 1) ? 12 + (xcd >> 1) : (xcd >> 1) * 3 + (idx >> 2) % 3;
        const int a = rh >> 1, peer = 2 * a + 1 - rj;
        const int s = lane >> 3, jj = lane & 7;
        const int gx = own || slab ? rank * 4 + a : peer * 4 + ri, gw = 32 + (rj ^ slab) * 16 + cb;
        const int sx = own || slab ? 4 + a : a;
        const bool isw = jj == 1 || jj == 2, on = own || (slab ? s == rank : s != peer);
        const bool two = ((s ^ rank) & 1) && cb >= 12 && cb != 12 + ri, spare = FEW && jj > 1 + two;
        unsigned poff;
        if (!on || spare) poff = WS_USED;
        else if (s == rank) poff = OFF_EF + (rank * 64 + (isw ? gw : gx)) * 256 + (32 + wave) * 4;
        else if (!isw) poff = sf_of(s, sx, wave) + (xcd >> 1) * REP / 4 * 128;
        else poff = sf_of(s, 8 + cb, wave) + ((xcd & 1) * 2 + (idx & 1)) * REP / 4 * 128 + (two ? jj - 1 : 0) * 4;
        unsigned xoff = par + P_STG + (s * NSLOT + sx) * BLOCK + wave * UNIT + (rh & 1) * 2048;
        const unsigned wd = par + P_STG + ((slab ? NR * NSLOT : s * NSLOT + 8) + cb) * BLOCK + wave * UNIT - xoff;
        const unsigned step = jj == 0 && on ? 4 * UNIT : 0, xb0 = jj == 0 ? xoff : 0;
        const unsigned lds = (unsigned)(size_t)(__attribute__((address_space(3))) f32x4*)red;
        if (slab && idx < 24)
            while ((unsigned)__builtin_amdgcn_s_memrealtime() - t0 < MBASE + (idx - 12) * MSTAG) __builtin_amdgcn_s_sleep(8);
        if (job == 1) __builtin_amdgcn_s_barrier();
        f32x16 y[2];
        asm volatile(R"(
.macro g_x r, o
    buffer_load_dwordx4 v[64+24*\r+4*\o:67+24*\r+4*\o], %[vo], %[buf], s40 offen offset:1024*\o )" DSC R"(
.endm
.macro g_w r, o
    buffer_load_dwordx4 v[72+24*\r+4*\o:75+24*\r+4*\o], %[vo], %[buf], s41 offen offset:1024*\o )" DSC R"(
.endm
.macro g_m c, m, e
    v_mfma_f32_32x32x16_f16 a[64*\m+16*\e:64*\m+16*\e+15], v[64+24*\c+4*\m:67+24*\c+4*\m], v[72+24*\c+4*\e:75+24*\c+4*\e], a[64*\m+16*\e:64*\m+16*\e+15]
.endm
.macro g_i z, n, i
  .if \i < 8*\n
    g_m (\z+\i/8), ((\i/4)&1), (\i&3)
  .endif
.endm
.macro g_poll
    buffer_load_dword v60, %[poff], %[buf], 0 offen )" PSC R"(
.endm
.macro g_take z, n, again
    g_i \z, \n, 0
    v_max_u32 v62, v62, v60
  .if \again
    g_poll
    s_nop 0
  .else
    s_nop 1
  .endif
    v_min_u32_dpp v63, v62, v62 quad_perm:[1,0,3,2] row_mask:0xf bank_mask:0xf
    v_cmp_gt_u32 vcc, %[fin], v62
    s_nop 0
    v_min_u32_dpp v63, v63, v63 quad_perm:[2,3,0,1] row_mask:0xf bank_mask:0xf
    g_i \z, \n, 1
    v_cndmask_b32 %[poff], v58, %[poff], vcc
    s_nop 0
    v_min_u32_dpp v63, v63, v63 row_half_mirror row_mask:0xf bank_mask:0xf
    v_sub_u32 v63, v63, %[base] clamp
    v_lshrrev_b32 v63, 2, v63
    v_mad_u32_u24 v59, v63, %[step], %[xb0]
    v_cmp_gt_u32 vcc, v59, %[xoff]
    g_i \z, \n, 2
.endm
.macro g_load r, z, n, i
    s_ff1_i32_b64 s42, vcc
    v_readlane_b32 s45, %[xoff], s42
    v_readlane_b32 s46, %[wd], s42
    v_cmp_eq_u32 vcc, s42, %[lane]
    s_mov_b32 s40, s45
    s_add_u32 s41, s45, s46
    g_x \r, 0
    g_i \z, \n, \i
    g_x \r, 1
    v_cndmask_b32 v63, 0, %[step], vcc
    g_w \r, 0
    v_add_u32 %[xoff], %[xoff], v63
    g_i \z, \n, (\i+1)
    g_w \r, 1
    v_cmp_gt_u32 vcc, v59, %[xoff]
    g_w \r, 2
    s_sub_u32 s44, s44, 1
    g_i \z, \n, (\i+2)
    g_w \r, 3
)"
R"(
.endm
.macro g_next y, z, n, i, nx
  .irp k, 3, 4, 5, 6, 7, 8, 9, 10, 11, 12, 13, 14, 15
   .if \k >= \i
    g_i \z, \n, \k
   .endif
  .endr
  .if \nx < 2
    s_cmp_le_u32 s44, )" STR(TAIL) R"(
    s_cbranch_scc1 Lz\@
    s_sleep )" STR(PACE) R"(
Lz\@:
  .endif
    s_mov_b32 s45, s47
    s_mov_b32 s47, \nx
    s_cmp_eq_u32 s45, 1
    s_waitcnt vmcnt(6*\nx)
    s_cbranch_scc1 Lt\y\()1_%=
    s_cmp_eq_u32 s45, 2
    s_cbranch_scc1 Lt\y\()2_%=
  .if \nx
    s_branch Lt\y\()0_%=
  .else
    s_branch Lb_%=
  .endif
.endm
.macro g_round x, y, z, n
Lk\x\n\()_%=:
    s_cmp_le_u32 s44, )" STR(TAIL) R"(
    s_cbranch_scc1 Lj\x\n\()_%=
    g_i \z, \n, 0
    g_i \z, \n, 1
    g_i \z, \n, 2
    s_branch Ll\x\n\()_%=
Lt\x\n\()_%=:
    s_cbranch_vccnz Lk\x\n\()_%=
Lj\x\n\()_%=:
    g_take \z, \n, 1
    s_cbranch_vccz Lu\x\n\()_%=
Ll\x\n\()_%=:
    g_load \x, \z, \n, 3
    s_cbranch_vccz Lv\x\n\()_%=
    g_load (\x+1), \z, \n, 6
    g_next \y, \z, \n, 9, 2
Lv\x\n\()_%=:
    g_next \y, \z, \n, 6, 1
Lu\x\n\()_%=:
    g_next \y, \z, \n, 3, 0
.endm
    s_nop 4
    g_poll
    v_mov_b32 v62, %[seen]
    v_mov_b32 v58, %[oob]
    s_mov_b32 s47, 0
    s_mov_b32 s44, %[nst]
  .set g_k, 0
  .rept 128
    v_accvgpr_write_b32 a[g_k], 0
  .set g_k, g_k+1
  .endr
    s_waitcnt vmcnt(0)
Lb_%=:
    s_cmp_eq_u32 s44, 0
    s_cbranch_scc1 Ld_%=
    s_memrealtime s[48:49]
Lp_%=:
    v_max_u32 v60, %[dead], v60
    g_take 0, 0, 0
    s_cbranch_vccnz Lq_%=
    s_sleep )" STR(POLL_SLEEP) R"(
    s_cmp_le_u32 s44, )" STR(TAIL) R"(
    s_cbranch_scc1 Lr_%=
    s_sleep )" STR(IDLE) R"(
Lr_%=:
    g_poll
    s_memrealtime s[50:51]
    s_waitcnt vmcnt(0) lgkmcnt(0)
    s_sub_u32 s45, s50, s48
    s_cmp_gt_u32 s45, %[ticks]
    s_cselect_b32 %[dead], %[last], %[dead]
    s_branch Lp_%=
Lq_%=:
    g_poll
    s_branch Ll00_%=
  .irp n, 0, 1, 2
    g_round 0, 2, 2, \n
    g_round 2, 4, 4, \n
    g_round 4, 0, 0, \n
  .endr
.macro g_stamp n
)"
R"(
.endm
Ld_%=:
    g_stamp 12
    s_cmp_eq_u32 %[dead], 0
    s_cbranch_scc1 Le_%=
    s_nop 15
  .irp i, 0, 8, 64, 72
    v_accvgpr_write_b32 a\i, -1
  .endr
Le_%=:
.macro g_out w
  .set g_k, 0
  .rept 24
   .set g_d, (\w+1+g_k/8)&3
   .set g_a, 64*(g_d/2)+16*(g_k&3)+8*(g_d&1)+(g_k&4)
    ds_write_b128 v[58+g_d/2], a[g_a:g_a+3] offset:1024*(24*(g_d&1)+g_k)
   .if g_k >= 16
    .set g_a, 64*(\w/2)+16*(g_k&3)+8*(\w&1)+(g_k&4)
    .irp t, 0, 1, 2, 3
    v_accvgpr_read_b32 v[4*g_k+\t], a[g_a+\t]
    .endr
   .endif
   .set g_k, g_k+1
  .endr
.endm
    v_lshl_add_u32 v58, %[lane], 4, %[lds]
    s_mul_i32 s45, %[wave], 24576
    v_add_u32 v59, 49152, v58
    v_add_u32 v60, s45, v58
    s_cmp_gt_u32 %[wave], 1
    s_cbranch_scc1 Lo2_%=
    s_cmp_eq_u32 %[wave], 1
    s_cbranch_scc1 Lo1_%=
    g_out 0
    s_branch Lo4_%=
Lo1_%=:
    g_out 1
    s_branch Lo4_%=
Lo2_%=:
    s_cmp_eq_u32 %[wave], 3
    s_cbranch_scc1 Lo3_%=
    g_out 2
    s_branch Lo4_%=
Lo3_%=:
    g_out 3
Lo4_%=:
    s_waitcnt lgkmcnt(0)
    g_stamp 10
    s_barrier
    g_stamp 8
  .irp i, 0, 1, 2, 3, 4, 5, 6, 7
   .irp j, 0, 1, 2
    ds_read_b128 v[96+12*\i+4*\j:99+12*\i+4*\j], v60 offset:1024*(8*\j+\i)
   .endr
  .endr
  .irp i, 0, 1, 2, 3, 4, 5, 6, 7
   .if \i == 0
    s_waitcnt lgkmcnt(15)
   .elseif \i > 2
    s_waitcnt lgkmcnt(21-3*\i)
   .endif
   .irp t, 0, 2
    v_pk_add_f32 v[64+4*\i+\t:65+4*\i+\t], v[64+4*\i+\t:65+4*\i+\t], v[96+12*\i+\t:97+12*\i+\t]
    v_pk_add_f32 v[100+12*\i+\t:101+12*\i+\t], v[100+12*\i+\t:101+12*\i+\t], v[104+12*\i+\t:105+12*\i+\t]
   .endr
   .irp t, 0, 1, 2, 3
    v_add_f32 v[208+4*(\i&4)+4*\t+(\i&3)], v[64+4*\i+\t], v[100+12*\i+\t]
   .endr
  .endr
.purgem g_out
.purgem g_stamp
.purgem g_x
.purgem g_w
.purgem g_m
.purgem g_i
.purgem g_poll
.purgem g_take
.purgem g_load
.purgem g_next
.purgem g_round
)"
            : [poff] "+v"(poff), [xoff] "+v"(xoff), [dead] "+s"(dead), "=&{v[208:223]}"(y[0]), "=&{v[224:239]}"(y[1])
            : [vo] "v"(lane * 16), [wd] "v"(wd), [step] "v"(step), [xb0] "v"(xb0), [lane] "v"(lane), [lds] "s"(lds), [seen] "v"(spare ? ~0u : 0u),
              [buf] "s"(bufw), [fin] "s"(fb + wave + 29), [base] "s"(fb + wave - 3), [last] "s"(fb + 32), [nst] "s"(own ? 64 : slab ? 8 : 56),
              [oob] "n"(WS_USED), [wave] "s"(wave), [ticks] "n"(DEADLINE_TICKS)
            : "memory", "vcc", "scc", "s40", "s41", "s42", "s43", "s44", "s45", "s46", "s47", "s48", "s49", "s50", "s51",
              CLOBBER_V, CLOBBER_A);
        const __amdgpu_buffer_rsrc_t od = own ? __builtin_amdgcn_make_buffer_rsrc((void*)p.out, 0, IN_BYTES, RSRC_W3) : buf;
        const unsigned pitch = own ? NN * 4 : 1024, hstep = own ? 8 * NN * 4 : UNIT, yu = (rh * 16 + cb) * 8 + wave * 2;
        const unsigned ooff = own ? ((rh * 64 + wave * 16 + (lane >> 5) * 4) * NN + rj * 2048 + cb * 128) * 4 + (lane & 31) * 16
                                  : par + P_M + yu * UNIT + lane * 16;
        u64 yl[2] = {0, 0};
#pragma unroll
        for (int h = 0; h < 2; ++h) {
            if (own) {
                if (h) SILU(224, 239, y[1]);
                else SILU(208, 223, y[0]);
            }
            f32x4 z[4];
            u32x4 o[4];
#pragma unroll
            for (int t = 0; t < 4; ++t) {
                z[t] = f32x4{y[h][4 * t], y[h][4 * t + 1], y[h][4 * t + 2], y[h][4 * t + 3]};
                o[t] = __builtin_bit_cast(u32x4, z[t]);
            }
            if (own || slab) {
#pragma unroll
                for (int t = 0; t < 4; ++t) __builtin_amdgcn_raw_buffer_store_b128(o[t], od, ooff + t * pitch + h * hstep, 0, AUX_SYS);
            } else yl[h] = pack_store<true>(z, o, buf, par + P_Y + (yu + h) * PUNIT, 0, lst, lane);
        }
        if (!own) {
            asm volatile("s_waitcnt vmcnt(0)" ::: "memory");
            if (lane == 0) {
                if (slab) __hip_atomic_fetch_add((gu32*)(myws + OFF_MC), 1u, __ATOMIC_RELAXED, __HIP_MEMORY_SCOPE_SYSTEM);
                else __builtin_amdgcn_raw_buffer_store_b128(
                    u32x4{fb + 32, (unsigned)yl[0], (unsigned)(yl[0] >> 32 | yl[1] << 16), (unsigned)(yl[1] >> 16)},
                    __builtin_amdgcn_make_buffer_rsrc(ws_of(wsl, peer), 0, WS_USED, RSRC_W3),
                    OFF_YF + ((rank * 2 + (rh & 1)) * 4 + wave) * 256 + cb * 16, 0, AUX_SYS);
            }
        }
        STAMP_V(11);
    }
}

extern "C" void kernel_launch(int rank, int nranks, ncclComm_t comm,
                              hipStream_t stream,
                              void* const* d_in, const int* in_sizes, int n_in,
                              void* d_out, int out_size,
                              const int* in_dtypes, int out_dtype,
                              void* const* d_out_all,
                              void* const* d_ws_all, size_t ws_bytes) {
    if (nranks != NR || (unsigned)rank >= (unsigned)NR || ws_bytes < WS_USED || n_in != 2 ||
        in_sizes[0] != 4096 * 512 || in_sizes[1] != 4096 * 512 || out_size != 4096 * 512) {
        fprintf(stderr, "unexpected problem shape\n");
        return;
    }
    Params p;
    p.x = (const float*)d_in[0];
    p.w = (const float*)d_in[1];
    for (int r = 0; r < NR; ++r) p.ws[r] = (char*)d_ws_all[r];
    p.mine = p.ws[rank];
    p.out = (char*)d_out;

    p.rank_links = rank;
    for (int q = 0, lo = 1, hi = 4; q < NR; ++q)
        if (q != rank) p.rank_links |= (u64)(rank ^ q) << (32 + 4 * (((GROUP >> q) ^ (GROUP >> rank)) & 1 ? hi++ : lo++));
    for (int b = 0; b < NWG; ++b) {
        const int g = b >> 4 == rank ? 60 - (rank & 1) * 16 + ((b >> 2) & 3) : ((b >> 2) + ((b + 80) >> 8) * 4) & 63;
        const int to = rank ^ ((p.rank_links >> (32 + 4 * (b & 7))) & 7), slot = slot_of((g & 16) / 4 + (g & 15) / 3, rank, to);
        const bool send = to != rank && !((0xf249f249ffffffffull >> g) & 1);
        p.start[b] = (send ? 64 + block_of(slot, to) : g) | g << 8 | to << 16 | slot << 24;
    }
    fused_gemm_silu<<<NWG, 256, 0, stream>>>(p);
    HIP_CHECK(hipGetLastError());
}
```
